```python
import numpy as np

kernel_name = "gpu_gemm_ar_m2048_k2048_n2048_bf16_silu"


ABSMAX_RTOL = 0.008

COMPUTE_DTYPE = "bfloat16"
EPILOGUE = "silu"
IN_QUANT = None
OUT_QUANT = None
IN_DTYPES = {"x": "bfloat16", "w_mat": "bfloat16"}
WORLD_SIZE = 8

def _to_bf16(x):
    u = np.ascontiguousarray(x, dtype=np.float32).view(np.uint32)
    return ((u + 0x7FFF + ((u >> 16) & 1)) & 0xFFFF0000).view(np.float32)

def setup_inputs():

    x = _to_bf16(np.random.randn(2048, 2048).astype(np.float32))
    w_mat = _to_bf16((np.random.randn(2048, 2048) / np.sqrt(2048)).astype(np.float32))
    return {"x": x, "w_mat": w_mat}

def _epilogue(y):
    return (y / (1.0 + np.exp(-np.clip(y, -60.0, 60.0)))).astype(np.float32)

def reference(x, w_mat):
    y = (x @ w_mat).astype(np.float32)
    return _epilogue(y)
```

```cpp
#include <hip/hip_runtime.h>
#include <rccl/rccl.h>
#include <cstdio>

#define GATE_OUT 1
#define ACQ_MODE 0
#define DIAG 0

typedef unsigned u32x4 __attribute__((ext_vector_type(4)));
typedef unsigned u32x2 __attribute__((ext_vector_type(2)));
typedef short bf16x8 __attribute__((ext_vector_type(8)));
typedef float f32x16 __attribute__((ext_vector_type(16)));
#define LAS __attribute__((address_space(3)))
#define GAS __attribute__((address_space(1)))

constexpr int NR = 8;
constexpr int M_ = 2048, N_ = 2048, KS = 256;
constexpr int NGEMM = 128, NPUSH = 32, NWG = NGEMM + NPUSH;
constexpr int NTHR = 256;
constexpr int NS = 8, DEPTH = 7, SLOT = 16384, NT = 32;
constexpr int LDS_BYTES = NS * SLOT;
constexpr unsigned long long DEADLINE = 20000000ull;

constexpr size_t OFF_CNT = 0;
constexpr size_t OFF_TMO = 64 * 1024;
constexpr size_t OFF_PD = OFF_TMO + 128;
constexpr size_t OFF_FA = OFF_TMO + 1024;
constexpr size_t OFF_FB = OFF_FA + 16 * 128;
constexpr size_t OFF_FO = OFF_FB + 8 * 128;
constexpr size_t OFF_DIAG = 128 * 1024;
constexpr size_t OFF_A = 1 << 20;
constexpr size_t OFF_B = OFF_A + (4u << 20);
constexpr size_t WS_NEED = OFF_B + (2u << 20);

struct Args { char* ws[NR]; char* out[NR]; const char* x; const char* w; int rank; int pad; };

__device__ __forceinline__ void st16(char* base, unsigned voff, u32x4 v) {
  asm volatile("global_store_dwordx4 %0, %1, %2 sc0 sc1\n\ts_nop 1" ::"v"(voff), "v"(v), "s"(base) : "memory");
}
__device__ __forceinline__ unsigned ld_sys(const char* p) { return __hip_atomic_load((const GAS unsigned*)p, __ATOMIC_RELAXED, __HIP_MEMORY_SCOPE_SYSTEM); }
__device__ __forceinline__ unsigned ld_agt(const char* p) { return __hip_atomic_load((const GAS unsigned*)p, __ATOMIC_RELAXED, __HIP_MEMORY_SCOPE_AGENT); }
__device__ __forceinline__ void st_sys(char* p, unsigned v) { __hip_atomic_store((GAS unsigned*)p, v, __ATOMIC_RELAXED, __HIP_MEMORY_SCOPE_SYSTEM); }
__device__ __forceinline__ void st_agt(char* p, unsigned v) { __hip_atomic_store((GAS unsigned*)p, v, __ATOMIC_RELAXED, __HIP_MEMORY_SCOPE_AGENT); }

template <bool SYS>
__device__ __forceinline__ bool wave_wait(const char* flag, unsigned seq, char* tmo) {
  const unsigned long long t0 = __builtin_amdgcn_s_memrealtime();
  for (;;) {
    unsigned v = seq;
    if (flag) v = SYS ? ld_sys(flag) : ld_agt(flag);
    if (__all(v >= seq)) return true;
    if (ld_agt(tmo) == seq || __builtin_amdgcn_s_memrealtime() - t0 > DEADLINE) { st_agt(tmo, seq); return false; }
    __builtin_amdgcn_s_sleep(2);
  }
}
#define WAITV(n) asm volatile("s_waitcnt vmcnt(" #n ")" ::: "memory")
#define RAWBAR() do { asm volatile("" ::: "memory"); __builtin_amdgcn_s_barrier(); asm volatile("" ::: "memory"); } while (0)
#define STAMP(i) do {} while (0)

__device__ __forceinline__ float silu(float y) {
  float c = fminf(fmaxf(y, -60.f), 60.f);
  return y * __builtin_amdgcn_rcpf(1.f + __expf(-c));
}
__device__ __forceinline__ unsigned pack_bf16(float lo, float hi) {
  typedef __bf16 bf2 __attribute__((ext_vector_type(2)));
  bf2 r; r.x = (__bf16)lo; r.y = (__bf16)hi; return __builtin_bit_cast(unsigned, r);
}

__global__ __launch_bounds__(NTHR, 1) void fused_gemm_ar(Args a) {
  extern __shared__ __attribute__((aligned(16))) char smem[];
  const int wg = blockIdx.x, tid = threadIdx.x, lane = tid & 63;
  const int wave = __builtin_amdgcn_readfirstlane(tid >> 6);
  const int rank = a.rank;
  char* myws = a.ws[rank];
  char* tmo = myws + OFF_TMO;
  STAMP(0);
  const int xcd = wg & 7, slot = wg >> 3;
  if (slot >= 16) {
    const int g = (slot - 16) * 8 + xcd;
    unsigned call = 0;
    const int ci = g >> 3, tn = g & 7;
    const int cg = lane & 7, kc = lane >> 3;
    const char* wsrc = a.w + (size_t)(wave * 64 + kc * 8) * 4096 + (size_t)(ci * 512 + tn * 64 + cg * 8) * 2;
    u32x4 wv[8];
#pragma unroll
    for (int i = 0; i < 8; ++i) wv[i] = *(const u32x4*)(wsrc + (size_t)i * 4096);
    const int half = g & 1, idx = g >> 1, u = idx & 3, rh = wave & 1, k16x = 4 * u + 2 * (wave >> 1);
    u32x4 xv[4][2];
#pragma unroll
    for (int q = 0; q < 4; ++q) {
      const int tm = 4 * q + (idx >> 2);
      const char* xs = a.x + (size_t)(half * 1024 + tm * 64 + rh * 32 + (lane & 31)) * 512 + k16x * 32 + (lane >> 5) * 16;
      xv[q][0] = *(const u32x4*)(xs); xv[q][1] = *(const u32x4*)(xs + 32);
    }
    call = ld_agt(myws + OFF_CNT + wg * 128) + 1;
#pragma unroll
    for (int i = 0; i < 8; ++i) asm volatile("" : "+v"(wv[i]));
#pragma unroll
    for (int q = 0; q < 4; ++q) { asm volatile("" : "+v"(xv[q][0])); asm volatile("" : "+v"(xv[q][1])); }
    asm volatile("" : "+v"(call));
    STAMP(1);
    {
      char* d0 = a.ws[ci] + OFF_B; char* d1 = a.ws[ci + 4] + OFF_B;
      const unsigned k16g = rank * 16 + wave * 4 + (kc >> 1);
      const unsigned base = tn * (256u << 10) + k16g * 2048 + ((kc & 1) * 32 + cg) * 16;
#pragma unroll
      for (int j = 0; j < 8; ++j) {
        u32x4 c;
#pragma unroll
        for (int e = 0; e < 4; ++e) c[e] = __builtin_amdgcn_perm(wv[2 * e + 1][j >> 1], wv[2 * e][j >> 1], (j & 1) ? 0x07060302u : 0x05040100u);
        const unsigned off = base + (j >> 2) * 1024 + (j & 3) * 128;
        st16(d0, off, c); st16(d1, off, c);
      }
    }
    char* xd[4];
#pragma unroll
    for (int p = 0; p < 4; ++p) xd[p] = a.ws[half * 4 + ((p + rank + idx) & 3)];
#define XSTORES(q) do { const unsigned off = (4 * (q) + (idx >> 2)) * (256u << 10) + (rank * 16 + k16x) * 2048 + rh * 1024 + lane * 16 + (unsigned)OFF_A; \
      _Pragma("unroll") for (int p = 0; p < 4; ++p) { st16(xd[p], off, xv[q][0]); st16(xd[p], off + 2048, xv[q][1]); } } while (0)
#define XFLAG(q) do { if (tid < 4) st_sys(a.ws[half * 4 + tid] + OFF_FA + (4 * (q) + (idx >> 2)) * 128 + (rank * 4 + u) * 4, call); } while (0)
    XSTORES(0);
    WAITV(8); RAWBAR();
    if (tid < 2) st_sys(a.ws[ci + 4 * tid] + OFF_FB + tn * 128 + rank * 4, call);
    STAMP(2);
    XSTORES(1);
    WAITV(8); RAWBAR(); XFLAG(0);
    XSTORES(2);
    WAITV(8); RAWBAR(); XFLAG(1);
    XSTORES(3);
    WAITV(8); RAWBAR(); XFLAG(2);
    STAMP(3);
    WAITV(0); RAWBAR(); XFLAG(3);
    if (tid == 0) { st_agt(myws + OFF_PD + g * 4, call); st_agt(myws + OFF_CNT + wg * 128, call); }
    STAMP(4);
    if (g == 0) {
      bool ok = true;
      for (int s = 0; s < 2; ++s) {
        const int src = wave * 2 + s;
        if (src == rank) continue;
        for (int r = 0; r < 2; ++r) ok &= wave_wait<true>(myws + OFF_FO + src * 512 + (r * 64 + lane) * 4, call, tmo);
      }
      if (!ok || ld_agt(tmo) == call) {
        if (lane == 0) *(GAS unsigned short*)(a.out[rank] + ((size_t)(rank >> 2) * 1024 * N_ + (rank & 3) * 512) * 2) = 0x7FC0;
      }
      STAMP(5);
    }
    return;
  }
  const int tm = xcd * 2 + (slot >> 3), tn = slot & 7;
  const int tile = tm * 8 + tn;
  unsigned call = ld_agt(myws + OFF_CNT + wg * 128) + 1;
  bool ok = true;
  if (wave == 0) {
    const char* f = nullptr;
    if (lane < 32) f = myws + OFF_FA + tm * 128 + lane * 4;
    else if (lane < 40) f = myws + OFF_FB + tn * 128 + (lane - 32) * 4;
    ok = wave_wait<true>(f, call, tmo);
    __builtin_amdgcn_fence(__ATOMIC_ACQUIRE, "");
    WAITV(0);
  }
  RAWBAR();
  STAMP(1);
  const char* asrc = myws + OFF_A + (size_t)tm * (256u << 10) + wave * 2048 + lane * 16;
  const char* bsrc = myws + OFF_B + (size_t)tn * (256u << 10) + wave * 2048 + lane * 16;
  constexpr int AUX = 0;
#define ISSUE(t) do { LAS char* l_ = (LAS char*)smem + ((t) & (NS - 1)) * SLOT + wave * 2048; \
    __builtin_amdgcn_global_load_lds((const GAS void*)(asrc + (size_t)(t) * 8192), (LAS void*)l_, 16, 0, AUX); \
    __builtin_amdgcn_global_load_lds((const GAS void*)(asrc + (size_t)(t) * 8192), (LAS void*)l_, 16, 1024, AUX); \
    __builtin_amdgcn_global_load_lds((const GAS void*)(bsrc + (size_t)(t) * 8192), (LAS void*)(l_ + 8192), 16, 0, AUX); \
    __builtin_amdgcn_global_load_lds((const GAS void*)(bsrc + (size_t)(t) * 8192), (LAS void*)(l_ + 8192), 16, 1024, AUX); } while (0)
  const int wr = wave >> 1, wc = wave & 1;
  f32x16 acc = {};
#define COMPUTE(t) do { const LAS char* l_ = (const LAS char*)smem + ((t) & (NS - 1)) * SLOT + lane * 16; \
    bf16x8 xf[4], wf[4]; \
    _Pragma("unroll") for (int kk = 0; kk < 4; ++kk) { \
      xf[kk] = *(const LAS bf16x8*)(l_ + (kk * 2 + wr) * 1024); \
      wf[kk] = *(const LAS bf16x8*)(l_ + 8192 + (kk * 2 + wc) * 1024); } \
    __builtin_amdgcn_sched_barrier(0); \
    _Pragma("unroll") for (int kk = 0; kk < 4; ++kk) acc = __builtin_amdgcn_mfma_f32_32x32x16_bf16(wf[kk], xf[kk], acc, 0, 0, 0); \
    __builtin_amdgcn_sched_barrier(0); } while (0)
#pragma unroll
  for (int s = 0; s < DEPTH; ++s) ISSUE(s);
#pragma unroll 1
  for (int t = 0; t < NT - DEPTH; ++t) {
    asm volatile("s_waitcnt vmcnt(24) lgkmcnt(0)" ::: "memory"); RAWBAR();
    ISSUE(t + DEPTH);
    COMPUTE(t);
  }
#define TAIL(t, n) do { asm volatile("s_waitcnt vmcnt(" #n ") lgkmcnt(0)" ::: "memory"); RAWBAR(); COMPUTE(t); } while (0)
  TAIL(25, 24); TAIL(26, 20); TAIL(27, 16); TAIL(28, 12); TAIL(29, 8); TAIL(30, 4); TAIL(31, 0);
  STAMP(2);
  asm volatile("s_waitcnt lgkmcnt(0)" ::: "memory"); RAWBAR();
  {
    const int h = lane >> 5, m = wr * 32 + (lane & 31);
#pragma unroll
    for (int q = 0; q < 4; ++q) {
      u32x2 v; v.x = pack_bf16(silu(acc[q]), silu(acc[q + 4])); v.y = pack_bf16(silu(acc[q + 8]), silu(acc[q + 12]));
      *(LAS u32x2*)((LAS char*)smem + m * 144 + 16 * (q + 4 * h) + 8 * wc) = v;
    }
  }
  bool bad = !ok;
  if (wave == 0) {
    const char* f = lane < 32 ? myws + OFF_PD + lane * 4 : nullptr;
    (void)wave_wait<false>(f, call, tmo);
  }
  asm volatile("s_waitcnt lgkmcnt(0)" ::: "memory"); RAWBAR();
  bad = ld_agt(tmo) == call;
  u32x4 o0 = *(const LAS u32x4*)((LAS char*)smem + (tid >> 3) * 144 + (tid & 7) * 16);
  u32x4 o1 = *(const LAS u32x4*)((LAS char*)smem + ((tid >> 3) + 32) * 144 + (tid & 7) * 16);
  if (bad) { o0 = (u32x4){0x7FC07FC0u, 0x7FC07FC0u, 0x7FC07FC0u, 0x7FC07FC0u}; o1 = o0; }
  asm volatile("" : "+v"(o0), "+v"(o1));
  STAMP(3);
  {
    const unsigned off = ((rank >> 2) * 1024 + tm * 64 + (tid >> 3)) * (N_ * 2) + ((rank & 3) * 512 + tn * 64) * 2 + (tid & 7) * 16;
#pragma unroll
    for (int p = 0; p < NR; ++p) { char* d = a.out[(p + rank + 1 + tile) & 7]; st16(d, off, o0); st16(d, off + 32 * N_ * 2, o1); }
  }
  WAITV(0); RAWBAR();
  if (tid < NR && tid != rank) st_sys(a.ws[tid] + OFF_FO + rank * 512 + tile * 4, call);
  if (tid == 0) st_agt(myws + OFF_CNT + wg * 128, call);
  STAMP(4);
}

extern "C" void kernel_launch(int rank, int nranks, ncclComm_t comm, hipStream_t stream, void* const* d_in, const int* in_sizes, int n_in,
                              void* d_out, int out_size, const int* in_dtypes, int out_dtype, void* const* d_out_all, void* const* d_ws_all, size_t ws_bytes) {
  if (nranks != NR || ws_bytes < WS_NEED || (unsigned)rank >= (unsigned)NR) { fprintf(stderr, "bad config\n"); return; }
  (void)hipFuncSetAttribute((const void*)fused_gemm_ar, hipFuncAttributeMaxDynamicSharedMemorySize, LDS_BYTES);
  Args a; for (int p = 0; p < NR; ++p) { a.ws[p] = (char*)d_ws_all[p]; a.out[p] = (char*)d_out_all[p]; }
  a.x = (const char*)d_in[0]; a.w = (const char*)d_in[1]; a.rank = rank; a.pad = 0;
  fused_gemm_ar<<<NWG, NTHR, LDS_BYTES, stream>>>(a);
}
```

```cpp
#include <hip/hip_runtime.h>
#include <rccl/rccl.h>
#include <cstdio>

#define DIAG 0
#define WIN 1
#define FORCE_RAW 0

typedef unsigned u32;
typedef unsigned long long u64;
typedef unsigned u32x4 __attribute__((ext_vector_type(4)));
typedef unsigned short u16x2 __attribute__((ext_vector_type(2)));
typedef short bf16x8 __attribute__((ext_vector_type(8)));
typedef float f32x16 __attribute__((ext_vector_type(16)));
typedef float f32x4 __attribute__((ext_vector_type(4)));
#define LAS __attribute__((address_space(3)))
#define GAS __attribute__((address_space(1)))

constexpr int NR = 8;
constexpr int N_ = 2048;
constexpr int NWG = 256, NTHR = 512;
constexpr u64 DEADLINE = 20000000ull;
constexpr u32 UNIT = 4096;

constexpr u32 OFF_CNT = 0;
constexpr u32 OFF_TMO = 64 * 1024;
constexpr u32 OFF_FA = 80 * 1024;
constexpr u32 OFF_FB = OFF_FA + 16 * 8 * 8 * 8;
constexpr u32 OFF_FO = 96 * 1024;
constexpr u32 OFF_FL = 128 * 1024;
constexpr u32 OFF_DIAG = 256 * 1024;
constexpr u32 OFF_A = 1u << 20;
constexpr u32 OFF_B = OFF_A + 16 * 8 * 8 * UNIT;
constexpr u32 OFF_IN = OFF_B + 8 * 8 * 8 * UNIT;
constexpr u32 OFF_OB = OFF_IN + 8 * 128 * 2 * UNIT;
constexpr u32 WS_NEED = OFF_OB + 128 * 2 * UNIT;

constexpr u32 L_IMG = 64 * 1024;
constexpr u32 L_HDR = 28 * UNIT;
constexpr u32 LDS_BYTES = L_HDR + 128;

struct Args { char* ws[NR]; char* out; const char* x; const char* w; int rank; int pad; };

__device__ __forceinline__ void st16(char* base, u32 voff, u32x4 v) {
  asm volatile("global_store_dwordx4 %0, %1, %2 sc0 sc1\n\ts_nop 1" ::"v"(voff), "v"(v), "s"(base) : "memory");
}
__device__ __forceinline__ u64 ld_sys64(const char* p) { return __hip_atomic_load((const GAS u64*)p, __ATOMIC_RELAXED, __HIP_MEMORY_SCOPE_SYSTEM); }
__device__ __forceinline__ u32 ld_agt(const char* p) { return __hip_atomic_load((const GAS u32*)p, __ATOMIC_RELAXED, __HIP_MEMORY_SCOPE_AGENT); }
__device__ __forceinline__ void st_sys64(char* p, u64 v) { __hip_atomic_store((GAS u64*)p, v, __ATOMIC_RELAXED, __HIP_MEMORY_SCOPE_SYSTEM); }
__device__ __forceinline__ void st_agt(char* p, u32 v) { __hip_atomic_store((GAS u32*)p, v, __ATOMIC_RELAXED, __HIP_MEMORY_SCOPE_AGENT); }
#define WAITV(n) asm volatile("s_waitcnt vmcnt(" #n ")" ::: "memory")
#define WAITL() asm volatile("s_waitcnt lgkmcnt(0)" ::: "memory")
#define RAWBAR() do { asm volatile("" ::: "memory"); __builtin_amdgcn_s_barrier(); asm volatile("" ::: "memory"); } while (0)
__device__ __forceinline__ u32 rfl(u32 v) { return __builtin_amdgcn_readfirstlane(v); }
__device__ __forceinline__ u32 rl(u32 v, int l) { return __builtin_amdgcn_readlane(v, l); }
#define STAMP(i) do {} while (0)

__device__ __forceinline__ u32 wave_wait64(const char* flag, u32 call, char* tmo, bool& ok) {
  const u64 t0 = __builtin_amdgcn_s_memrealtime();
  u32 data = 0;
  for (;;) {
    u64 v = (u64)call << 32;
    if (flag) v = ld_sys64(flag);
    const u32 tv = ld_agt(tmo);
    data = (u32)v;
    if (__all((u32)(v >> 32) >= call)) { ok = true; break; }
    if (tv == call || __builtin_amdgcn_s_memrealtime() - t0 > DEADLINE) { st_agt(tmo, call); ok = false; break; }
    __builtin_amdgcn_s_sleep(1);
  }
  asm volatile("" ::: "memory");
  return data;
}

__device__ __forceinline__ u32 perm(u32 hi, u32 lo, u32 sel) { return __builtin_amdgcn_perm(hi, lo, sel); }
__device__ __forceinline__ u32 rotr(u32 x, u32 n) { return __builtin_amdgcn_alignbit(x, x, n); }
__device__ __forceinline__ u32 bfi(u32 m, u32 a, u32 b) { return (a & m) | (b & ~m); }
__device__ __forceinline__ u32 pkmax(u32 a, u32 b) { return __builtin_bit_cast(u32, __builtin_elementwise_max(__builtin_bit_cast(u16x2, a), __builtin_bit_cast(u16x2, b))); }
template <int CTRL, int RM> __device__ __forceinline__ u32 dppmax(u32 m) { u32 t = __builtin_amdgcn_update_dpp(0u, m, CTRL, RM, 0xf, false); return t > m ? t : m; }

constexpr u32 HDR_RAW = 127u | (63u << 13) | (63u << 25);

__device__ __forceinline__ u32 pack_unit(const u32 (&O)[16], u32 (&P)[12], int lane) {
  u32 H[8], U[8];
#pragma unroll
  for (int m = 0; m < 8; ++m) { P[m] = perm(O[2 * m + 1], O[2 * m], 0x06040200u); H[m] = perm(O[2 * m + 1], O[2 * m], 0x07050301u); }
  u32 e = 0, o = 0;
#pragma unroll
  for (int m = 0; m < 8; ++m) { const u32 h7 = H[m] & 0x7F7F7F7Fu; e = pkmax(e, h7); o = pkmax(o, h7 << 8); }
  u32 mx = pkmax(e, o);
  mx = pkmax(mx, mx >> 16) & 0xFF00u;
  mx = dppmax<0x111, 0xf>(mx); mx = dppmax<0x112, 0xf>(mx); mx = dppmax<0x114, 0xf>(mx); mx = dppmax<0x118, 0xf>(mx);
  mx = dppmax<0x142, 0xa>(mx); mx = dppmax<0x143, 0xc>(mx);
  const u32 mc = rl(mx, 63) >> 8;
  const u32 base = mc > 7 ? mc - 7 : 0;
  const u32 baseB = base * 0x01010101u;
  u32 orr = 0;
#pragma unroll
  for (int m = 0; m < 8; ++m) { U[m] = (H[m] | 0x80808080u) - baseB; orr |= U[m]; }
  const bool bad = (orr & 0x78787878u) != 0;
  const u64 mask = __ballot(bad);
  u32 hdr = base | (63u << 13) | (63u << 25);
  if (mask) {
    u32 eo = 0, ea = 0xFFFFFFFFu, z = 0;
#pragma unroll
    for (int m = 0; m < 8; ++m) {
      const u32 bl = ~U[m] & 0x80808080u;
      const u32 mk = bl | (bl - (bl >> 7));
      eo |= U[m] & mk; ea &= U[m] | ~mk;
      const u32 t = U[m] ^ 0x80808080u;
      z |= (t - 0x01010101u) & ~t & 0x80808080u;
      U[m] = bfi(mk, 0x80808080u, U[m]);
    }
    eo |= eo >> 16; eo |= eo >> 8; eo &= 0xFFu;
    ea &= ea >> 16; ea &= ea >> 8; ea &= 0xFFu;
    const bool lane_ok = z == 0 && eo == ea && eo >= 0x41u;
    const u64 fail = __ballot(bad && !lane_ok);
    if (fail || __popcll(mask) > 2) hdr = HDR_RAW;
    else {
      const int l0 = __ffsll((long long)mask) - 1, l1 = 63 - __clzll((long long)mask);
      const u32 d0 = 0x7Fu - rl(eo, l0), d1 = 0x7Fu - rl(eo, l1);
      hdr = base | (l0 << 7) | (d0 << 13) | (l1 << 19) | (d1 << 25);
    }
  }
#pragma unroll
  for (int t = 0; t < 4; ++t) {
    const u32 ya = bfi(0x80808080u, H[2 * t], U[2 * t]), yb = bfi(0x80808080u, H[2 * t + 1], U[2 * t + 1]);
    P[8 + t] = ya | rotr(yb, 28);
  }
  return hdr;
}

__device__ __forceinline__ void merge_lo_hi(u32x4 p0, u32x4 p1, int t, u32 h0, u32 h1, u32 (&O)[16]) {
  const u32 l0 = t < 2 ? p0[2 * t] : p1[2 * t - 4], l1 = t < 2 ? p0[2 * t + 1] : p1[2 * t - 3];
  O[4 * t + 0] = perm(h0, l0, 0x05010400u); O[4 * t + 1] = perm(h0, l0, 0x07030602u);
  O[4 * t + 2] = perm(h1, l1, 0x05010400u); O[4 * t + 3] = perm(h1, l1, 0x07030602u);
}
__device__ __forceinline__ void unpack_unit(u32x4 p0, u32x4 p1, u32x4 p2, u32 hdr, int lane, u32 (&O)[16]) {
  const u32 base = hdr & 127u;
  const u32 baseB = base * 0x01010101u;
  const u32 d0 = (hdr >> 13) & 63u;
  if (d0 == 63u) {
#pragma unroll
    for (int t = 0; t < 4; ++t) {
      const u32 n = p2[t];
      merge_lo_hi(p0, p1, t, (n & 0x87878787u) + baseB, (rotr(n, 4) & 0x87878787u) + baseB, O);
    }
  } else {
    const u32 T0 = baseB + 0x03020100u, T1 = T0 + 0x04040404u;
    const u32 d1 = (hdr >> 25) & 63u;
    const u32 Ta = (T0 & ~0xFFu) | (base - 1u - d0), Tb = (T0 & ~0xFFu) | (base - 1u - d1);
    u32 Tv = lane == (int)((hdr >> 19) & 63u) ? Tb : T0;
    Tv = lane == (int)((hdr >> 7) & 63u) ? Ta : Tv;
#pragma unroll
    for (int t = 0; t < 4; ++t) {
      const u32 n = p2[t], r = rotr(n, 4);
      merge_lo_hi(p0, p1, t, perm(T1, Tv, n & 0x07070707u) | (n & 0x80808080u), perm(T1, Tv, r & 0x07070707u) | (r & 0x80808080u), O);
    }
  }
}

__device__ __forceinline__ float silu(float y) {
  const float c = fminf(fmaxf(y, -60.f), 60.f);
  return y * __builtin_amdgcn_rcpf(1.f + __expf(-c));
}
__device__ __forceinline__ u32 pack_bf16(float lo, float hi) {
  typedef __bf16 bf2 __attribute__((ext_vector_type(2)));
  bf2 r; r.x = (__bf16)lo; r.y = (__bf16)hi; return __builtin_bit_cast(u32, r);
}
__device__ __forceinline__ u32x4 mk4(const u32* p) { u32x4 v = {p[0], p[1], p[2], p[3]}; return v; }

struct Unit { u32x4 p[3]; };
__device__ __forceinline__ void load_unit(Unit& u, __amdgpu_buffer_rsrc_t rs, u32 voff, u32 soff) {
  u.p[0] = __builtin_amdgcn_raw_buffer_load_b128(rs, voff, soff, 17);
  u.p[1] = __builtin_amdgcn_raw_buffer_load_b128(rs, voff + 1024, soff, 17);
  u.p[2] = __builtin_amdgcn_raw_buffer_load_b128(rs, voff + 2048, soff, 17);
}
__device__ __forceinline__ void decode_unit(const Unit& u, const char* ws, u32 voff, u32 soff, u32 hdr, int lane, u32 (&O)[16]) {
  if ((hdr & 127u) == 127u) {
    u32x4 p3;
    asm volatile("global_load_dwordx4 %0, %1, %2 offset:3072 sc0 sc1\n\ts_waitcnt vmcnt(0)" : "=&v"(p3) : "v"(voff), "s"(ws + soff) : "memory");
#pragma unroll
    for (int i = 0; i < 12; ++i) O[i] = u.p[i >> 2][i & 3];
#pragma unroll
    for (int i = 0; i < 4; ++i) O[12 + i] = p3[i];
  } else unpack_unit(u.p[0], u.p[1], u.p[2], hdr, lane, O);
}

__global__ __launch_bounds__(NTHR, 2) void fused_gemm_ar(Args a) {
  __shared__ __attribute__((aligned(16))) char smem_[LDS_BYTES];
  LAS char* smem = (LAS char*)smem_;
  const int wg = blockIdx.x, tid = threadIdx.x, lane = tid & 63;
  const int wave = rfl(tid >> 6);
  const int rank = a.rank;
  char* myws = a.ws[rank];
  char* tmo = myws + OFF_TMO;
  STAMP(0);
  const char* cntp = myws + OFF_CNT + wg * 128;

  if (wg < 128) {
    const int p = wg & 7, j = wg >> 3;
    const int q = j & 3, blk = j >> 2;
    const bool self = p == rank;
    LAS u32* hdrs = (LAS u32*)(smem + L_HDR);
    if (wave == 0) {
      const u32 callv = ld_agt(cntp);
      if (lane < 28) hdrs[lane] = 0;
      WAITL(); RAWBAR();
      const u32 call = rfl(callv) + 1;
      char* dws = a.ws[p];
      const int nr = self ? 6 : 14;
      char* pend_flag = nullptr; u32 ph = 0;
      for (int r = 0; r < nr; ++r) {
        u32 h0, h1;
        {
          u64 hv;
          asm volatile("ds_read_b64 %0, %1\n\ts_waitcnt lgkmcnt(0)" : "=v"(hv) : "v"((u32)(uintptr_t)(hdrs + 2 * r)) : "memory");
          if (!(rfl((u32)hv) >> 31 & rfl((u32)(hv >> 32)) >> 31)) {
            if (pend_flag) { WAITV(0); if (lane < 2) st_sys64(pend_flag + lane * 8, ((u64)call << 32) | ph); pend_flag = nullptr; }
            do {
              __builtin_amdgcn_s_sleep(1);
              asm volatile("ds_read_b64 %0, %1\n\ts_waitcnt lgkmcnt(0)" : "=v"(hv) : "v"((u32)(uintptr_t)(hdrs + 2 * r)) : "memory");
            } while (!(rfl((u32)hv) >> 31 & rfl((u32)(hv >> 32)) >> 31));
          }
          h0 = rfl((u32)hv) & 0x7FFFFFFFu; h1 = rfl((u32)(hv >> 32)) & 0x7FFFFFFFu;
        }
        u32 dslot, fslot;
        if (r >= 6) { const u32 u = (rank * 128 + (r - 6) * 16 + j) * 2; dslot = OFF_IN + u * UNIT; fslot = OFF_FO + u * 8; }
        else if (r == 0 || r == 2) { const u32 u = ((r * 2 + blk) * 8 + rank) * 8 + q * 2; dslot = OFF_B + u * UNIT; fslot = OFF_FB + u * 8; }
        else { const u32 xq = r == 1 ? 0 : r - 2; const u32 u = ((xq * 4 + blk) * 8 + rank) * 8 + q * 2; dslot = OFF_A + u * UNIT; fslot = OFF_FA + u * 8; }
        const LAS char* src = smem + r * 2 * UNIT + lane * 16;
        char* dst = dws + dslot;
        const u32 vo = lane * 16;
        const u32x4 v0 = *(const LAS u32x4*)(src), v1 = *(const LAS u32x4*)(src + 1024), v2 = *(const LAS u32x4*)(src + 2048);
        const u32x4 v4 = *(const LAS u32x4*)(src + UNIT), v5 = *(const LAS u32x4*)(src + UNIT + 1024), v6 = *(const LAS u32x4*)(src + UNIT + 2048);
        if ((h0 & 127u) == 127u) st16(dst, vo + 3072, *(const LAS u32x4*)(src + 3072));
        if ((h1 & 127u) == 127u) st16(dst, vo + UNIT + 3072, *(const LAS u32x4*)(src + UNIT + 3072));
        st16(dst, vo, v0); st16(dst, vo + 1024, v1); st16(dst, vo + 2048, v2);
        st16(dst, vo + UNIT, v4); st16(dst, vo + UNIT + 1024, v5); st16(dst, vo + UNIT + 2048, v6);
        WAITV(0);
        if (lane < 2) st_sys64(dws + fslot + lane * 8, ((u64)call << 32) | (lane == 0 ? h0 : h1));
      }
      if (lane == 0) st_agt(myws + OFF_CNT + wg * 128, call);
      STAMP(6);
      return;
    }
    const int k = wave - 1;
    const int r1 = k >> 1, i1 = k & 1;
    const bool w1 = r1 == 0 || r1 == 2;
    const int c = lane & 31, h = lane >> 5;
    const char* xrow = a.x + (size_t)((p >> 2) * 1024 + blk * 64 + c) * 512 + q * 128 + h * 32;
    u32 DW[16]; u32x4 X1[4], X2[4];
    if (w1) {
      const char* ws_ = a.w + (size_t)(q * 64 + i1 * 32 + h * 16) * 4096 + ((p & 3) * 512 + (r1 * 2 + blk) * 64 + 2 * c) * 2;
#pragma unroll
      for (int rr = 0; rr < 16; ++rr) DW[rr] = *(const GAS u32*)(ws_ + (size_t)rr * 4096);
    } else {
      const int xq = r1 == 1 ? 0 : 1;
      const char* xs = xrow + (size_t)xq * 256 * 512 + i1 * 32 * 512;
#pragma unroll
      for (int cc = 0; cc < 4; ++cc) X1[cc] = *(const GAS u32x4*)(xs + (cc >> 1) * 64 + (cc & 1) * 16);
    }
    const int g2 = k + 7, r2 = g2 >> 1, i2 = g2 & 1;
    if (k < 5) {
      const char* xs = xrow + (size_t)(r2 - 2) * 256 * 512 + i2 * 32 * 512;
#pragma unroll
      for (int cc = 0; cc < 4; ++cc) X2[cc] = *(const GAS u32x4*)(xs + (cc >> 1) * 64 + (cc & 1) * 16);
    }
    const u32 callv = ld_agt(cntp);
    STAMP(4);
    RAWBAR();
    STAMP(5);
#pragma unroll 1
    for (int job = 0; job < (k < 5 ? 2 : 1); ++job) {
      u32 O[16];
      if (job == 1) {
#pragma unroll
        for (int i = 0; i < 16; ++i) O[i] = X2[i >> 2][i & 3];
      } else if (w1) {
#pragma unroll
        for (int cb = 0; cb < 4; ++cb)
#pragma unroll
          for (int e = 0; e < 4; ++e) O[4 * cb + e] = perm(DW[8 * (cb & 1) + 2 * e + 1], DW[8 * (cb & 1) + 2 * e], (cb >> 1) ? 0x07060302u : 0x05040100u);
      } else {
#pragma unroll
        for (int i = 0; i < 16; ++i) O[i] = X1[i >> 2][i & 3];
      }
      const int slot = job == 0 ? k : g2;
      u32 P[12];
      const u32 hdr = pack_unit(O, P, lane);
      LAS char* d = smem + slot * UNIT + lane * 16;
      if ((hdr & 127u) == 127u) {
#pragma unroll
        for (int pl = 0; pl < 4; ++pl) *(LAS u32x4*)(d + pl * 1024) = mk4(O + 4 * pl);
      } else {
#pragma unroll
        for (int pl = 0; pl < 3; ++pl) *(LAS u32x4*)(d + pl * 1024) = mk4(P + 4 * pl);
      }
      WAITL();
      if (lane == 0) hdrs[slot] = hdr | 0x80000000u;
    }
    const u32 call = rfl(callv) + 1;
    STAMP(1);
    if (!self) {
      for (int o = k; o < 8; o += 7) {
        const int tile = o * 16 + j;
        bool ok;
        const u32 hv = wave_wait64(lane < 2 ? myws + OFF_FL + tile * 128 + lane * 8 : nullptr, call, tmo, ok);
        u32 h0 = rl(hv, 0), h1 = rl(hv, 1);
        if (!ok) { h0 = HDR_RAW; h1 = HDR_RAW; }
        const char* src = myws + OFF_OB + tile * 2 * UNIT + lane * 16;
        LAS char* d = smem + (12 + 2 * o) * UNIT;
#pragma unroll
        for (int pl = 0; pl < 3; ++pl) {
          __builtin_amdgcn_global_load_lds((const GAS void*)(src + pl * 1024), (LAS void*)(d + pl * 1024), 16, 0, 17);
          __builtin_amdgcn_global_load_lds((const GAS void*)(src + UNIT + pl * 1024), (LAS void*)(d + UNIT + pl * 1024), 16, 0, 17);
        }
        if ((h0 & 127u) == 127u) __builtin_amdgcn_global_load_lds((const GAS void*)(src + 3072), (LAS void*)(d + 3072), 16, 0, 17);
        if ((h1 & 127u) == 127u) __builtin_amdgcn_global_load_lds((const GAS void*)(src + UNIT + 3072), (LAS void*)(d + UNIT + 3072), 16, 0, 17);
        WAITV(0);
        if (lane < 2) hdrs[12 + 2 * o + lane] = (lane == 0 ? h0 : h1) | 0x80000000u;
        STAMP(2 + (o >= 7));
      }
    }
    return;
  }

  const int gw = wg - 128;
  const int x = gw & 7, slot = gw >> 3, grp = slot >> 2, m = slot & 3;
  const int rh = m & 1, ul = (x >> 1) * 4 + (x & 1) * 2 + (m >> 1);
  const __amdgpu_buffer_rsrc_t rs = __builtin_amdgcn_make_buffer_rsrc((void*)myws, 0, (int)WS_NEED, 0x00020000);
  const u32 voff = lane * 16;
  const u32 call = rfl(ld_agt(cntp)) + 1;
  char* outp = a.out;
  const u32 ovo = (lane >> 3) * (N_ * 2) + (lane & 7) * 16;
#pragma unroll 1
  for (int job = 0; job < 2; ++job) {
    const int o = grp + 4 * job;
    const int tm = 4 * (o >> 1) + (ul >> 2), tn = 4 * (o & 1) + (ul & 3);
    const int ht = (o * 16 + ul) * 2 + rh;
    bool ok;
    u32 hv;
    {
      const char* f = nullptr;
      if (lane < 4) f = myws + OFF_FA + ((tm * 8 + wave) * 8 + lane * 2 + rh) * 8;
      else if (lane < 12) f = myws + OFF_FB + ((tn * 8 + wave) * 8 + lane - 4) * 8;
      hv = wave_wait64(f, call, tmo, ok);
    }
    STAMP(1 + 4 * job);
    f32x16 acc[2] = {};
    if (ok) {
      const u32 sa = OFF_A + ((tm * 8 + wave) * 8 + rh) * UNIT, sb = OFF_B + (tn * 8 + wave) * 8 * UNIT;
      Unit ua[4], ub[8];
#pragma unroll
      for (int q = 0; q < 4; ++q) {
        load_unit(ua[q], rs, voff, sa + 2 * q * UNIT); __builtin_amdgcn_sched_barrier(0);
        load_unit(ub[2 * q], rs, voff, sb + 2 * q * UNIT); __builtin_amdgcn_sched_barrier(0);
        load_unit(ub[2 * q + 1], rs, voff, sb + (2 * q + 1) * UNIT); __builtin_amdgcn_sched_barrier(0);
      }
#pragma unroll
      for (int q = 0; q < 4; ++q) {
        u32 OA[16];
        decode_unit(ua[q], myws, voff, sa + 2 * q * UNIT, rl(hv, q), lane, OA);
#pragma unroll
        for (int i = 0; i < 2; ++i) {
          u32 OB[16];
          decode_unit(ub[2 * q + i], myws, voff, sb + (2 * q + i) * UNIT, rl(hv, 4 + 2 * q + i), lane, OB);
#pragma unroll
          for (int b = 0; b < 2; ++b)
#pragma unroll
            for (int ch = 0; ch < 2; ++ch)
              acc[ch] = __builtin_amdgcn_mfma_f32_32x32x16_bf16(__builtin_bit_cast(bf16x8, mk4(OB + 4 * (2 * ch + b))), __builtin_bit_cast(bf16x8, mk4(OA + 4 * (2 * i + b))), acc[ch], 0, 0, 0);
        }
      }
    }
    STAMP(2 + 4 * job);
    {
      LAS char* pw = smem + wave * 8192 + lane * 16;
#pragma unroll
      for (int ch = 0; ch < 2; ++ch)
#pragma unroll
        for (int g = 0; g < 4; ++g) {
          const f32x4 v = {acc[ch][4 * g], acc[ch][4 * g + 1], acc[ch][4 * g + 2], acc[ch][4 * g + 3]};
          *(LAS f32x4*)(pw + (ch * 4 + g) * 1024) = v;
        }
    }
    WAITL(); RAWBAR();
    {
      typedef float f32x2 __attribute__((ext_vector_type(2)));
      typedef unsigned u32x2 __attribute__((ext_vector_type(2)));
      const int g = wave & 3, hf = wave >> 2;
      const LAS char* pr = smem + g * 1024 + lane * 16 + hf * 8;
      f32x2 r0[8], r1[8];
#pragma unroll
      for (int s = 0; s < 8; ++s) { r0[s] = *(const LAS f32x2*)(pr + s * 8192); r1[s] = *(const LAS f32x2*)(pr + s * 8192 + 4096); }
      __builtin_amdgcn_sched_barrier(0);
      const f32x2 a0 = ((r0[0] + r0[1]) + (r0[2] + r0[3])) + ((r0[4] + r0[5]) + (r0[6] + r0[7]));
      const f32x2 a1 = ((r1[0] + r1[1]) + (r1[2] + r1[3])) + ((r1[4] + r1[5]) + (r1[6] + r1[7]));
      u32x2 y;
      y[0] = pack_bf16(silu(a0[0]), silu(a1[0])); y[1] = pack_bf16(silu(a0[1]), silu(a1[1]));
      *(LAS u32x2*)(smem + L_IMG + (lane & 31) * 144 + (2 * g + (lane >> 5)) * 16 + hf * 8) = y;
    }
    const u32 tv = ld_agt(tmo);
    WAITL(); RAWBAR();
    STAMP(3 + 4 * job);
    if (wave < 2) {
      u32 O[16];
      const bool bad = rfl(tv) == call;
#pragma unroll
      for (int cc = 0; cc < 4; ++cc) {
        const u32x4 t = *(const LAS u32x4*)(smem + L_IMG + (cc * 8 + (lane >> 3)) * 144 + (lane & 7) * 16);
#pragma unroll
        for (int e = 0; e < 4; ++e) O[4 * cc + e] = bad ? 0x7FC07FC0u : t[e];
      }
      if (wave == 0) {
        u32 P[12];
        const u32 hdr = pack_unit(O, P, lane);
        char* d = myws + OFF_OB + ht * UNIT;
        if ((hdr & 127u) == 127u) {
#pragma unroll
          for (int pl = 0; pl < 4; ++pl) st16(d, voff + pl * 1024, mk4(O + 4 * pl));
        } else {
#pragma unroll
          for (int pl = 0; pl < 3; ++pl) st16(d, voff + pl * 1024, mk4(P + 4 * pl));
        }
        WAITV(0);
        if (lane == 0) st_sys64(myws + OFF_FL + (ht >> 1) * 128 + rh * 8, ((u64)call << 32) | hdr);
      } else {
        char* d = outp + ((size_t)((rank >> 2) * 1024 + tm * 64 + rh * 32) * N_ + (rank & 3) * 512 + tn * 64) * 2;
#pragma unroll
        for (int cc = 0; cc < 4; ++cc) st16(d + cc * 8 * N_ * 2, ovo, mk4(O + 4 * cc));
      }
    }
    STAMP(4 + 4 * job);
  }
  if (wave != rank) {
#pragma unroll 1
    for (int job = 0; job < 2; ++job) {
      const int o = grp + 4 * job;
      const int tm = 4 * (o >> 1) + (ul >> 2), tn = 4 * (o & 1) + (ul & 3);
      const int ht = (o * 16 + ul) * 2 + rh;
      const char* f = lane == 0 ? myws + OFF_FO + (wave * 256 + ht) * 8 : nullptr;
      bool ok2;
      const u32 h0 = rl(wave_wait64(f, call, tmo, ok2), 0);
      STAMP(9 + job);
      const u32 tv = ld_agt(tmo);
      u32 O[16];
      if (ok2) {
        Unit u0;
        const u32 si = OFF_IN + (wave * 256 + ht) * UNIT;
        load_unit(u0, rs, voff, si);
        decode_unit(u0, myws, voff, si, h0, lane, O);
      }
      if (!ok2 || rfl(tv) == call) {
#pragma unroll
        for (int i = 0; i < 16; ++i) O[i] = 0x7FC07FC0u;
      }
      char* d = outp + ((size_t)((wave >> 2) * 1024 + tm * 64 + rh * 32) * N_ + (wave & 3) * 512 + tn * 64) * 2;
#pragma unroll
      for (int cc = 0; cc < 4; ++cc) st16(d + cc * 8 * N_ * 2, ovo, mk4(O + 4 * cc));
    }
  }
  if (tid == 0) st_agt(myws + OFF_CNT + wg * 128, call);
  STAMP(11);
}

extern "C" void kernel_launch(int rank, int nranks, ncclComm_t comm, hipStream_t stream, void* const* d_in, const int* in_sizes, int n_in,
                              void* d_out, int out_size, const int* in_dtypes, int out_dtype, void* const* d_out_all, void* const* d_ws_all, size_t ws_bytes) {
  if (nranks != NR || ws_bytes < WS_NEED || (unsigned)rank >= (unsigned)NR) { fprintf(stderr, "bad config\n"); return; }
  Args a;
  for (int p = 0; p < NR; ++p) a.ws[p] = (char*)d_ws_all[p];
  a.out = (char*)d_out; a.x = (const char*)d_in[0]; a.w = (const char*)d_in[1]; a.rank = rank; a.pad = 0;
  fused_gemm_ar<<<NWG, NTHR, 0, stream>>>(a);
}
```

```cpp
#include <hip/hip_runtime.h>
#include <rccl/rccl.h>
#include <cstdio>

#define DIAG 0
#define WIN 1
#define FORCE_RAW 0
#define ZDELAY 0
#define EARLY 1
#define UNR 0
#define ORD 1
#define HFIRST 1
#define DEFER2 0
#define STAG 0
#define FIRSTD 1
#define NOZERO 0
#define PATH1 0
#define BARC 1

typedef unsigned u32;
typedef unsigned long long u64;
typedef unsigned u32x4 __attribute__((ext_vector_type(4)));
typedef unsigned short u16x2 __attribute__((ext_vector_type(2)));
typedef short bf16x8 __attribute__((ext_vector_type(8)));
typedef float f32x16 __attribute__((ext_vector_type(16)));
typedef float f32x4 __attribute__((ext_vector_type(4)));
#define LAS __attribute__((address_space(3)))
#define GAS __attribute__((address_space(1)))

constexpr int NR = 8;
constexpr int N_ = 2048;
constexpr int NWG = 256, NTHR = 512;
constexpr u64 DEADLINE = 20000000ull;
constexpr u32 UNIT = 4096;

constexpr u32 OFF_CNT = 0;
constexpr u32 OFF_TMO = 64 * 1024;
constexpr u32 OFF_FA = 80 * 1024;
constexpr u32 OFF_FB = OFF_FA + 16 * 8 * 8 * 8;
constexpr u32 OFF_FO = 96 * 1024;
constexpr u32 OFF_FL = 128 * 1024;
constexpr u32 OFF_DIAG = 256 * 1024;
constexpr u32 OFF_A = 1u << 20;
constexpr u32 OFF_B = OFF_A + 16 * 8 * 8 * UNIT;
constexpr u32 AB_BYTES = (16 + 8) * 8 * 8 * UNIT;
constexpr u32 OFF_IN = OFF_A + 2 * AB_BYTES;
constexpr u32 OFF_OB = OFF_IN + 8 * 128 * 2 * UNIT;
constexpr u32 WS_NEED = OFF_OB + 128 * 2 * UNIT;

constexpr u32 L_IMG = 64 * 1024;
constexpr u32 L_HDR = 28 * UNIT;
constexpr u32 LDS_BYTES = L_HDR + 128;

struct Args { const char* x; const char* w; char* out; char* myws; int rank; int pad; char* ws[NR]; };

__device__ __forceinline__ void st16(char* base, u32 voff, u32x4 v) {
  asm volatile("global_store_dwordx4 %0, %1, %2 sc0 sc1\n\ts_nop 1" ::"v"(voff), "v"(v), "s"(base) : "memory");
}
__device__ __forceinline__ u64 ld_sys64(const char* p) { return __hip_atomic_load((const GAS u64*)p, __ATOMIC_RELAXED, __HIP_MEMORY_SCOPE_SYSTEM); }
__device__ __forceinline__ u32 ld_agt(const char* p) { return __hip_atomic_load((const GAS u32*)p, __ATOMIC_RELAXED, __HIP_MEMORY_SCOPE_AGENT); }
__device__ __forceinline__ void st_sys64(char* p, u64 v) { __hip_atomic_store((GAS u64*)p, v, __ATOMIC_RELAXED, __HIP_MEMORY_SCOPE_SYSTEM); }
__device__ __forceinline__ void st_agt(char* p, u32 v) { __hip_atomic_store((GAS u32*)p, v, __ATOMIC_RELAXED, __HIP_MEMORY_SCOPE_AGENT); }
#define WAITV(n) asm volatile("s_waitcnt vmcnt(" #n ")" ::: "memory")
#define WAITL() asm volatile("s_waitcnt lgkmcnt(0)" ::: "memory")
#define RAWBAR() do { asm volatile("" ::: "memory"); __builtin_amdgcn_s_barrier(); asm volatile("" ::: "memory"); } while (0)
__device__ __forceinline__ u32 rfl(u32 v) { return __builtin_amdgcn_readfirstlane(v); }
__device__ __forceinline__ u32 rl(u32 v, int l) { return __builtin_amdgcn_readlane(v, l); }
#define STAMP(i) do {} while (0)
#define LSTAMP(i) do {} while (0)

__device__ __forceinline__ u32 wave_wait64(const char* flag, u32 call, char* tmo, bool& ok) {
  const u64 t0 = __builtin_amdgcn_s_memrealtime();
  u32 data = 0;
  for (;;) {
    u64 v = (u64)call << 32;
    if (flag) v = ld_sys64(flag);
    const u32 tv = ld_agt(tmo);
    data = (u32)v;
    if (__all((u32)(v >> 32) >= call)) { ok = true; break; }
    if (tv == call || __builtin_amdgcn_s_memrealtime() - t0 > DEADLINE) { st_agt(tmo, call); ok = false; break; }
    __builtin_amdgcn_s_sleep(1);
  }
  asm volatile("" ::: "memory");
  return data;
}

__device__ __forceinline__ u32 perm(u32 hi, u32 lo, u32 sel) { return __builtin_amdgcn_perm(hi, lo, sel); }
__device__ __forceinline__ u32 rotr(u32 x, u32 n) { return __builtin_amdgcn_alignbit(x, x, n); }
__device__ __forceinline__ u32 bfi(u32 m, u32 a, u32 b) { return (a & m) | (b & ~m); }
__device__ __forceinline__ u32 pkmax(u32 a, u32 b) { return __builtin_bit_cast(u32, __builtin_elementwise_max(__builtin_bit_cast(u16x2, a), __builtin_bit_cast(u16x2, b))); }
template <int CTRL, int RM> __device__ __forceinline__ u32 dppmax(u32 m) { u32 t = __builtin_amdgcn_update_dpp(0u, m, CTRL, RM, 0xf, false); return t > m ? t : m; }

constexpr u32 HDR_RAW = 127u | (63u << 13) | (63u << 25);
constexpr u32 KX = 0x80008080u;
__device__ __forceinline__ u32 min3u(u32 a, u32 b, u32 c) { const u32 m = a < b ? a : b; return m < c ? m : c; }

__device__ __forceinline__ u32 pack_unit(const u32 (&O)[16], u32 (&P)[12], int lane) {
  u32 H[8], U[8], L[8];
#pragma unroll
  for (int m = 0; m < 8; ++m) { L[m] = perm(O[2 * m + 1], O[2 * m], 0x06040200u); H[m] = perm(O[2 * m + 1], O[2 * m], 0x07050301u); }
  u32 e = 0, o = 0;
#pragma unroll
  for (int m = 0; m < 8; ++m) { const u32 h7 = H[m] & 0x7F7F7F7Fu; e = pkmax(e, h7); o = pkmax(o, h7 << 8); }
  u32 mx = pkmax(e, o);
  mx = pkmax(mx, mx >> 16) & 0xFF00u;
  mx = dppmax<0x111, 0xf>(mx); mx = dppmax<0x112, 0xf>(mx); mx = dppmax<0x114, 0xf>(mx); mx = dppmax<0x118, 0xf>(mx);
  mx = dppmax<0x142, 0xa>(mx); mx = dppmax<0x143, 0xc>(mx);
  const u32 mc = rl(mx, 63) >> 8;
  const u32 base = mc > 7 ? mc - 7 : 0;
  const u32 baseB = base * 0x01010101u;
  u32 orr = 0;
#pragma unroll
  for (int m = 0; m < 8; ++m) { U[m] = (H[m] | 0x80808080u) - baseB; orr |= U[m]; }
  const bool bad = (orr & 0x78787878u) != 0;
  const u64 mask = __ballot(bad);
  u32 hdr = base | (63u << 13) | (63u << 25);
  if (mask) {
    u32 eo = 0, ea = 0xFFFFFFFFu, z = 0;
#pragma unroll
    for (int m = 0; m < 8; ++m) {
      const u32 bl = ~U[m] & 0x80808080u;
      const u32 mk = bl | (bl - (bl >> 7));
      eo |= U[m] & mk; ea &= U[m] | ~mk;
      const u32 t = U[m] ^ 0x80808080u;
      z |= (t - 0x01010101u) & ~t & 0x80808080u;
      U[m] = bfi(mk, 0x80808080u, U[m]);
    }
    eo |= eo >> 16; eo |= eo >> 8; eo &= 0xFFu;
    ea &= ea >> 16; ea &= ea >> 8; ea &= 0xFFu;
    const bool lane_ok = z == 0 && eo == ea && eo >= 0x41u;
    const u64 fail = __ballot(bad && !lane_ok);
    if (fail || __popcll(mask) > 2) hdr = HDR_RAW;
    else {
      const int l0 = __ffsll((long long)mask) - 1, l1 = 63 - __clzll((long long)mask);
      const u32 d0 = 0x7Fu - rl(eo, l0), d1 = 0x7Fu - rl(eo, l1);
      hdr = base | (l0 << 7) | (d0 << 13) | (l1 << 19) | (d1 << 25);
    }
  }
  u32 C[4];
#pragma unroll
  for (int t = 0; t < 4; ++t) {
    const u32 ya = bfi(0x80808080u, H[2 * t], U[2 * t]), yb = bfi(0x80808080u, H[2 * t + 1], U[2 * t + 1]);
    C[t] = (ya ^ KX) + rotr(yb, 28);
  }
  if (__ballot(min3u(C[0], C[1], C[2]) == 0)) hdr = HDR_RAW;
  P[0] = L[0]; P[1] = L[1]; P[2] = L[2]; P[3] = C[0];
  P[4] = L[3]; P[5] = L[4]; P[6] = L[5]; P[7] = C[1];
  P[8] = L[6]; P[9] = L[7]; P[10] = C[2]; P[11] = C[3];
  return hdr;
}

__device__ __forceinline__ void merge_lo_hi(u32x4 p0, u32x4 p1, u32x4 p2, int t, u32 h0, u32 h1, u32 (&O)[16]) {
  const u32 l0 = t == 0 ? p0[0] : t == 1 ? p0[2] : t == 2 ? p1[1] : p2[0], l1 = t == 0 ? p0[1] : t == 1 ? p1[0] : t == 2 ? p1[2] : p2[1];
  O[4 * t + 0] = perm(h0, l0, 0x05010400u); O[4 * t + 1] = perm(h0, l0, 0x07030602u);
  O[4 * t + 2] = perm(h1, l1, 0x05010400u); O[4 * t + 3] = perm(h1, l1, 0x07030602u);
}
__device__ __forceinline__ void unpack_unit(u32x4 p0, u32x4 p1, u32x4 p2, u32 hdr, int lane, u32 (&O)[16]) {
  const u32 base = hdr & 127u;
  const u32 baseB = base * 0x01010101u;
  const u32 d0 = (hdr >> 13) & 63u;
  if (d0 == 63u) {
#pragma unroll
    for (int t = 0; t < 4; ++t) {
      const u32 n = t == 0 ? p0[3] : t == 1 ? p1[3] : t == 2 ? p2[2] : p2[3];
      merge_lo_hi(p0, p1, p2, t, ((n & 0x87878787u) ^ KX) + baseB, (rotr(n, 4) & 0x87878787u) + baseB, O);
    }
  } else {
    const u32 T0 = baseB + 0x03020100u, T1 = T0 + 0x04040404u;
    const u32 d1 = (hdr >> 25) & 63u;
    const u32 Ta = (T0 & ~0xFFu) | (base - 1u - d0), Tb = (T0 & ~0xFFu) | (base - 1u - d1);
    u32 Tv = lane == (int)((hdr >> 19) & 63u) ? Tb : T0;
    Tv = lane == (int)((hdr >> 7) & 63u) ? Ta : Tv;
#pragma unroll
    for (int t = 0; t < 4; ++t) {
      const u32 n = t == 0 ? p0[3] : t == 1 ? p1[3] : t == 2 ? p2[2] : p2[3], r = rotr(n, 4);
      merge_lo_hi(p0, p1, p2, t, perm(T1, Tv, n & 0x07070707u) | ((n & 0x80808080u) ^ KX), perm(T1, Tv, r & 0x07070707u) | (r & 0x80808080u), O);
    }
  }
}

__device__ __forceinline__ float silu(float y) {
  const float c = fminf(fmaxf(y, -60.f), 60.f);
  return y * __builtin_amdgcn_rcpf(1.f + __expf(-c));
}
__device__ __forceinline__ u32 pack_bf16(float lo, float hi) {
  typedef __bf16 bf2 __attribute__((ext_vector_type(2)));
  bf2 r; r.x = (__bf16)lo; r.y = (__bf16)hi; return __builtin_bit_cast(u32, r);
}
__device__ __forceinline__ u32x4 mk4(const u32* p) { u32x4 v = {p[0], p[1], p[2], p[3]}; return v; }

struct Unit { u32x4 p[3]; };
__device__ __forceinline__ void load_unit(Unit& u, __amdgpu_buffer_rsrc_t rs, u32 voff, u32 soff) {
  u.p[0] = __builtin_amdgcn_raw_buffer_load_b128(rs, voff, soff, 17);
  u.p[1] = __builtin_amdgcn_raw_buffer_load_b128(rs, voff + 1024, soff, 17);
  u.p[2] = __builtin_amdgcn_raw_buffer_load_b128(rs, voff + 2048, soff, 17);
}
__device__ __forceinline__ void decode_unit(Unit& u, __amdgpu_buffer_rsrc_t rs, const char* ws, u32 voff, u32 soff, u32 hdr, int lane, u32 call, char* tmo, u32 (&O)[16], u32& nretry) {
  if ((hdr & 127u) == 127u) {
    u32x4 p3;
    asm volatile("global_load_dwordx4 %0, %1, %2 offset:3072 sc0 sc1\n\ts_waitcnt vmcnt(0)" : "=&v"(p3) : "v"(voff), "s"(ws + soff) : "memory");
#pragma unroll
    for (int i = 0; i < 12; ++i) O[i] = u.p[i >> 2][i & 3];
#pragma unroll
    for (int i = 0; i < 4; ++i) O[12 + i] = p3[i];
  } else {
    if (__builtin_expect(__ballot(min3u(u.p[0][3], u.p[1][3], u.p[2][2]) == 0) != 0, 0)) {
      const u64 t0 = __builtin_amdgcn_s_memrealtime();
      for (;;) {
        load_unit(u, rs, voff, soff);
        const u32 tv = rfl(ld_agt(tmo));
        if (!__ballot(min3u(u.p[0][3], u.p[1][3], u.p[2][2]) == 0)) break;
        if (tv == call || __builtin_amdgcn_s_memrealtime() - t0 > DEADLINE) { st_agt(tmo, call); break; }
      }
    }
    unpack_unit(u.p[0], u.p[1], u.p[2], hdr, lane, O);
  }
}

__device__ __forceinline__ bool round_is_w(int r) { return r < 2; }
__device__ __forceinline__ int round_idx(int r) { return r < 2 ? r : r - 2; }
__global__ __launch_bounds__(NTHR, 2) void fused_gemm_ar(Args a) {
  __shared__ __attribute__((aligned(16))) char smem_[LDS_BYTES];
  LAS char* smem = (LAS char*)smem_;
  const int wg = blockIdx.x, tid = threadIdx.x, lane = tid & 63;
  const int wave = rfl(tid >> 6);
  const char* xin = a.x; const char* win = a.w; char* outp = a.out; char* myws = a.myws; int rank = a.rank; char* dws = a.ws[wg & 7];
  asm volatile("" : "+s"(xin), "+s"(win), "+s"(outp), "+s"(myws), "+s"(rank), "+s"(dws));
  char* tmo = myws + OFF_TMO;
  STAMP(0);
  const char* cntp = myws + OFF_CNT + wg * 128;

  if (wg < 128) {
    const int p = wg & 7, j = wg >> 3;
    const int q = j & 3, blk = j >> 2;
    const bool self = p == rank;
    LAS u32* hdrs = (LAS u32*)(smem + L_HDR);
    if (wave == 0) {
      const u32 callv = ld_agt(cntp);
      if (lane < 28) hdrs[lane] = 0;
      WAITL(); RAWBAR();
      const u32 call = rfl(callv) + 1;
      const int nr = self ? 6 : 14;
      const u32 par = (call & 1) * AB_BYTES;
#pragma unroll 1
      for (int r = FIRSTD ? 1 : 0; r < nr; ++r) {
        u32 h0, h1;
        {
          u64 hv;
          for (;;) {
            asm volatile("ds_read_b64 %0, %1\n\ts_waitcnt lgkmcnt(0)" : "=v"(hv) : "v"((u32)(uintptr_t)(hdrs + 2 * r)) : "memory");
            if (rfl((u32)hv) >> 31 & rfl((u32)(hv >> 32)) >> 31) break;
            __builtin_amdgcn_s_sleep(1);
          }
          h0 = rfl((u32)hv) & 0x7FFFFFFFu; h1 = rfl((u32)(hv >> 32)) & 0x7FFFFFFFu;
        }
        LSTAMP(2 * r);
        u32 dslot, fslot;
        if (r >= 6) { const u32 u = (rank * 128 + (r - 6) * 16 + j) * 2; dslot = OFF_IN + u * UNIT; fslot = OFF_FO + u * 8; }
        else if (round_is_w(r)) { const u32 u = ((round_idx(r) * 4 + blk) * 8 + rank) * 8 + q * 2; dslot = OFF_B + par + u * UNIT; fslot = OFF_FB + u * 8; }
        else { const u32 u = ((round_idx(r) * 4 + blk) * 8 + rank) * 8 + q * 2; dslot = OFF_A + par + u * UNIT; fslot = OFF_FA + u * 8; }
        const LAS char* src = smem + r * 2 * UNIT + lane * 16;
        char* dst = dws + dslot;
        const u32 vo = lane * 16;
        const u32x4 v0 = *(const LAS u32x4*)(src), v1 = *(const LAS u32x4*)(src + 1024), v2 = *(const LAS u32x4*)(src + 2048);
        const u32x4 v4 = *(const LAS u32x4*)(src + UNIT), v5 = *(const LAS u32x4*)(src + UNIT + 1024), v6 = *(const LAS u32x4*)(src + UNIT + 2048);
        const bool raw0 = (h0 & 127u) == 127u, raw1 = (h1 & 127u) == 127u;
        const bool hfirst = HFIRST && EARLY && (r >= 6 || HFIRST == 2) && !raw0 && !raw1;
        if (hfirst && lane < 2) st_sys64(dws + fslot + lane * 8, ((u64)call << 32) | (lane == 0 ? h0 : h1));
        if (raw0) st16(dst, vo + 3072, *(const LAS u32x4*)(src + 3072));
        if (raw1) st16(dst, vo + UNIT + 3072, *(const LAS u32x4*)(src + UNIT + 3072));
        st16(dst, vo, v0); st16(dst, vo + 1024, v1); st16(dst, vo + 2048, v2);
        st16(dst, vo + UNIT, v4); st16(dst, vo + UNIT + 1024, v5); st16(dst, vo + UNIT + 2048, v6);
        if (!EARLY || raw0 || raw1) WAITV(0);
        if (!hfirst && lane < 2) st_sys64(dws + fslot + lane * 8, ((u64)call << 32) | (lane == 0 ? h0 : h1));
        WAITV(0);
        LSTAMP(2 * r + 1);
      }
      if (lane == 0) st_agt(myws + OFF_CNT + wg * 128, call);
      STAMP(6);
      return;
    }
    const int k = wave - 1;
    const int r1 = k >> 1, i1 = k & 1;
    const bool w1 = round_is_w(r1);
    const int c = lane & 31, h = lane >> 5;
    const __amdgpu_buffer_rsrc_t rx = __builtin_amdgcn_make_buffer_rsrc((void*)xin, 0, 2048 * 512, 0x00020000);
    const __amdgpu_buffer_rsrc_t rw = __builtin_amdgcn_make_buffer_rsrc((void*)win, 0, 256 * 4096, 0x00020000);
    const u32 xv = c * 512 + h * 32, xs0 = ((p >> 2) * 1024 + blk * 64) * 512 + q * 128;
    const u32 wv = h * 16 * 4096 + c * 4, ws0 = q * 64 * 4096 + ((p & 3) * 512 + blk * 64) * 2;
    u32 D1[16], D2[16];
    auto loadw = [&](u32 (&D)[16], int r, int i) {
      const u32 so = ws0 + i * 32 * 4096 + round_idx(r) * 512;
#pragma unroll
      for (int rr = 0; rr < 16; ++rr) D[rr] = __builtin_amdgcn_raw_buffer_load_b32(rw, wv + (rr & 0) , so + rr * 4096, 0);
    };
    auto loadx = [&](u32 (&D)[16], int r, int i) {
      const u32 so = xs0 + round_idx(r) * 256 * 512 + i * 32 * 512;
#pragma unroll
      for (int cc = 0; cc < 4; ++cc) { const u32x4 t = __builtin_amdgcn_raw_buffer_load_b128(rx, xv + (cc >> 1) * 64 + (cc & 1) * 16, so, 0); D[4 * cc] = t[0]; D[4 * cc + 1] = t[1]; D[4 * cc + 2] = t[2]; D[4 * cc + 3] = t[3]; }
    };
    if (STAG && k >= 2) __builtin_amdgcn_s_sleep(STAG);
    if (w1) loadw(D1, r1, i1); else loadx(D1, r1, i1);
    const int g2 = k + 7, r2 = g2 >> 1, i2 = g2 & 1;
    const bool w2 = ORD == 2 && round_is_w(r2);
    auto load2 = [&]() { if (k < 5) { if (w2) loadw(D2, r2, i2); else loadx(D2, r2, i2); } };
    if (!DEFER2) load2();
    const u32 callv = ld_agt(cntp);
    STAMP(4);
    RAWBAR();
    STAMP(5);
#pragma unroll
    for (int job = 0; job < 2; ++job) {
      if (job == 1 && k >= 5) break;
      u32 O[16];
      if (job == 1 ? w2 : w1) {
        const u32* D = job == 1 ? D2 : D1;
#pragma unroll
        for (int cb = 0; cb < 4; ++cb)
#pragma unroll
          for (int e = 0; e < 4; ++e) O[4 * cb + e] = perm(D[8 * (cb & 1) + 2 * e + 1], D[8 * (cb & 1) + 2 * e], (cb >> 1) ? 0x07060302u : 0x05040100u);
      } else {
#pragma unroll
        for (int i = 0; i < 16; ++i) O[i] = job == 1 ? D2[i] : D1[i];
      }
      if (DEFER2 && job == 0) { asm volatile("" :: "v"(O[0]), "v"(O[15]) : "memory"); load2(); __builtin_amdgcn_sched_barrier(0); }
      const int slot = job == 0 ? k : g2;
      u32 P[12];
      const u32 hdr = pack_unit(O, P, lane);
      if (FIRSTD && job == 0 && k < 2) {
        const u32 u = ((round_idx(0) * 4 + blk) * 8 + rank) * 8 + q * 2 + k;
        const u32 call0 = rfl(callv) + 1;
        char* dst = dws + OFF_B + (call0 & 1) * AB_BYTES + u * UNIT;
        char* fl = dws + OFF_FB + u * 8;
        if ((hdr & 127u) == 127u) {
#pragma unroll
          for (int pl = 0; pl < 4; ++pl) st16(dst, lane * 16 + pl * 1024, mk4(O + 4 * pl));
          WAITV(0);
        } else {
#pragma unroll
          for (int pl = 0; pl < 3; ++pl) st16(dst, lane * 16 + pl * 1024, mk4(P + 4 * pl));
        }
        if (lane == 0) st_sys64(fl, ((u64)call0 << 32) | hdr);
        if (k == 0) { LSTAMP(0); LSTAMP(1); }
        continue;
      }
      LAS char* d = smem + slot * UNIT + lane * 16;
      if ((hdr & 127u) == 127u) {
#pragma unroll
        for (int pl = 0; pl < 4; ++pl) *(LAS u32x4*)(d + pl * 1024) = mk4(O + 4 * pl);
      } else {
#pragma unroll
        for (int pl = 0; pl < 3; ++pl) *(LAS u32x4*)(d + pl * 1024) = mk4(P + 4 * pl);
      }
      WAITL();
      if (lane == 0) hdrs[slot] = hdr | 0x80000000u;
    }
    const u32 call = rfl(callv) + 1;
    STAMP(1);
    if (!self) {
      for (int o = k; o < 8; o += 7) {
        const int tile = o * 16 + j;
        bool ok;
        const u32 hv = wave_wait64(lane < 2 ? myws + OFF_FL + tile * 128 + lane * 8 : nullptr, call, tmo, ok);
        u32 h0 = rl(hv, 0), h1 = rl(hv, 1);
        const char* src = myws + OFF_OB + tile * 2 * UNIT + lane * 16;
        LAS char* d = smem + (12 + 2 * o) * UNIT;
        if (ok) {
#pragma unroll
          for (int pl = 0; pl < 3; ++pl) {
            __builtin_amdgcn_global_load_lds((const GAS void*)(src + pl * 1024), (LAS void*)(d + pl * 1024), 16, 0, 17);
            __builtin_amdgcn_global_load_lds((const GAS void*)(src + UNIT + pl * 1024), (LAS void*)(d + UNIT + pl * 1024), 16, 0, 17);
          }
          if ((h0 & 127u) == 127u) __builtin_amdgcn_global_load_lds((const GAS void*)(src + 3072), (LAS void*)(d + 3072), 16, 0, 17);
          if ((h1 & 127u) == 127u) __builtin_amdgcn_global_load_lds((const GAS void*)(src + UNIT + 3072), (LAS void*)(d + UNIT + 3072), 16, 0, 17);
          WAITV(0);
        } else {
          h0 = HDR_RAW; h1 = HDR_RAW;
          const u32x4 nan4 = {0x7FC07FC0u, 0x7FC07FC0u, 0x7FC07FC0u, 0x7FC07FC0u};
#pragma unroll
          for (int pl = 0; pl < 8; ++pl) *(LAS u32x4*)(d + pl * 1024 + lane * 16) = nan4;
          WAITL();
        }
        if (lane < 2) hdrs[12 + 2 * o + lane] = (lane == 0 ? h0 : h1) | 0x80000000u;
        STAMP(2);
      }
    }
    return;
  }

  const int gw = wg - 128;
  const int x = gw & 7, slot = gw >> 3, grp = slot >> 2, m = slot & 3;
  const int rh = m & 1, ul = (x >> 1) * 4 + (x & 1) * 2 + (m >> 1);
  const __amdgpu_buffer_rsrc_t rs = __builtin_amdgcn_make_buffer_rsrc((void*)myws, 0, (int)WS_NEED, 0x00020000);
  const u32 voff = lane * 16;
  const u32 call = rfl(ld_agt(cntp)) + 1;
  const u32 par = (call & 1) * AB_BYTES;
  const u32 ovo = (lane >> 3) * (N_ * 2) + (lane & 7) * 16;
  {
    if (ZDELAY) { const u64 z0 = __builtin_amdgcn_s_memrealtime(); while (__builtin_amdgcn_s_memrealtime() - z0 < ZDELAY) __builtin_amdgcn_s_sleep(2); }
    const u32x4 z = {0, 0, 0, 0};
    char* zb = myws + OFF_A + (AB_BYTES - par) + gw * 12 * UNIT;
#pragma unroll
    for (int i = 0; i < 5; ++i) {
      const u32 pl = wave + 8 * i, un = (pl * 43) >> 7;
      if (!NOZERO && pl < 36) st16(zb + un * UNIT + (pl - 3 * un) * 1024, voff, z);
    }
    WAITV(0);
  }
  u32 OB[8][16];
  u32 nrB = 0, nrA = 0, nrO = 0;
#pragma unroll 1
  for (int job = 0; job < 2; ++job) {
    const int o = grp + 4 * job;
    const int tm = 4 * (o >> 1) + (ul >> 2), tn = 4 * (o & 1) + (ul & 3);
    const int ht = (o * 16 + ul) * 2 + rh;
    f32x16 acc[2] = {};
    {
      const u32 sa = OFF_A + par + ((tm * 8 + wave) * 8 + rh) * UNIT, sb = OFF_B + par + (tn * 8 + wave) * 8 * UNIT;
      bool ok;
      if (job == 0) {
        const u32 hb = wave_wait64(lane < 8 ? myws + OFF_FB + ((tn * 8 + wave) * 8 + lane) * 8 : nullptr, call, tmo, ok);
        STAMP(12);
        Unit ub[8];
#pragma unroll
        for (int u = 0; u < 8; ++u) { load_unit(ub[u], rs, voff, sb + u * UNIT); __builtin_amdgcn_sched_barrier(0); }
#pragma unroll
        for (int u = 0; u < 8; ++u) decode_unit(ub[u], rs, myws, voff, sb + u * UNIT, rl(hb, u), lane, call, tmo, OB[u], nrB);
      }
      const u32 ha = wave_wait64(lane < 4 ? myws + OFF_FA + ((tm * 8 + wave) * 8 + lane * 2 + rh) * 8 : nullptr, call, tmo, ok);
      STAMP(1 + 4 * job);
      Unit ua[4];
#pragma unroll
      for (int q = 0; q < 4; ++q) { load_unit(ua[q], rs, voff, sa + 2 * q * UNIT); __builtin_amdgcn_sched_barrier(0); }
#pragma unroll
      for (int q = 0; q < 4; ++q) {
        u32 OA[16];
        decode_unit(ua[q], rs, myws, voff, sa + 2 * q * UNIT, rl(ha, q), lane, call, tmo, OA, nrA);
#pragma unroll
        for (int i = 0; i < 2; ++i)
#pragma unroll
          for (int b = 0; b < 2; ++b)
#pragma unroll
            for (int ch = 0; ch < 2; ++ch)
              acc[ch] = __builtin_amdgcn_mfma_f32_32x32x16_bf16(__builtin_bit_cast(bf16x8, mk4(OB[2 * q + i] + 4 * (2 * ch + b))), __builtin_bit_cast(bf16x8, mk4(OA + 4 * (2 * i + b))), acc[ch], 0, 0, 0);
      }
    }
    STAMP(2 + 4 * job);
    {
      LAS char* pw = smem + wave * 8192 + lane * 16;
#pragma unroll
      for (int ch = 0; ch < 2; ++ch)
#pragma unroll
        for (int g = 0; g < 4; ++g) {
          const f32x4 v = {acc[ch][4 * g], acc[ch][4 * g + 1], acc[ch][4 * g + 2], acc[ch][4 * g + 3]};
          *(LAS f32x4*)(pw + (ch * 4 + g) * 1024) = v;
        }
    }
    WAITL(); RAWBAR();
    {
      typedef float f32x2 __attribute__((ext_vector_type(2)));
      typedef unsigned u32x2 __attribute__((ext_vector_type(2)));
      const int g = wave & 3, hf = wave >> 2;
      const LAS char* pr = smem + g * 1024 + lane * 16 + hf * 8;
      f32x2 r0[8], r1[8];
#pragma unroll
      for (int s = 0; s < 8; ++s) { r0[s] = *(const LAS f32x2*)(pr + s * 8192); r1[s] = *(const LAS f32x2*)(pr + s * 8192 + 4096); }
      __builtin_amdgcn_sched_barrier(0);
      const f32x2 a0 = ((r0[0] + r0[1]) + (r0[2] + r0[3])) + ((r0[4] + r0[5]) + (r0[6] + r0[7]));
      const f32x2 a1 = ((r1[0] + r1[1]) + (r1[2] + r1[3])) + ((r1[4] + r1[5]) + (r1[6] + r1[7]));
      u32x2 y;
      y[0] = pack_bf16(silu(a0[0]), silu(a1[0])); y[1] = pack_bf16(silu(a0[1]), silu(a1[1]));
      *(LAS u32x2*)(smem + L_IMG + (lane & 31) * 144 + (2 * g + (lane >> 5)) * 16 + hf * 8) = y;
    }
    const u32 tv = ld_agt(tmo);
    WAITL(); RAWBAR();
    STAMP(3 + 4 * job);
    if (wave == 0) {
      u32 O[16];
      const bool bad = rfl(tv) == call;
#pragma unroll
      for (int cc = 0; cc < 4; ++cc) {
        const u32x4 t = *(const LAS u32x4*)(smem + L_IMG + (cc * 8 + (lane >> 3)) * 144 + (lane & 7) * 16);
#pragma unroll
        for (int e = 0; e < 4; ++e) O[4 * cc + e] = bad ? 0x7FC07FC0u : t[e];
      }
      u32 P[12];
      const u32 hdr = pack_unit(O, P, lane);
      char* d = myws + OFF_OB + ht * UNIT;
      if ((hdr & 127u) == 127u) {
#pragma unroll
        for (int pl = 0; pl < 4; ++pl) st16(d, voff + pl * 1024, mk4(O + 4 * pl));
      } else {
#pragma unroll
        for (int pl = 0; pl < 3; ++pl) st16(d, voff + pl * 1024, mk4(P + 4 * pl));
      }
      char* dd = outp + ((size_t)((rank >> 2) * 1024 + tm * 64 + rh * 32) * N_ + (rank & 3) * 512 + tn * 64) * 2;
#pragma unroll
      for (int cc = 0; cc < 4; ++cc) st16(dd + cc * 8 * N_ * 2, ovo, mk4(O + 4 * cc));
      if (BARC && job == 0) RAWBAR();
      WAITV(4);
      if (lane == 0) st_sys64(myws + OFF_FL + (ht >> 1) * 128 + rh * 8, ((u64)call << 32) | hdr);
      STAMP(4 + 4 * job);
    } else if (BARC && job == 0) RAWBAR();
  }
  if (wave != rank) {
#pragma unroll 1
    for (int job = 0; job < 2; ++job) {
      const int o = grp + 4 * job;
      const int tm = 4 * (o >> 1) + (ul >> 2), tn = 4 * (o & 1) + (ul & 3);
      const int ht = (o * 16 + ul) * 2 + rh;
      const char* f = lane == 0 ? myws + OFF_FO + (wave * 256 + ht) * 8 : nullptr;
      bool ok2;
      const u32 h0 = rl(wave_wait64(f, call, tmo, ok2), 0);
      STAMP(9 + job);
      u32 O[16];
      const u32 si = OFF_IN + (wave * 256 + ht) * UNIT;
      {
        Unit u0;
        load_unit(u0, rs, voff, si);
        decode_unit(u0, rs, myws, voff, si, h0, lane, call, tmo, O, nrO);
      }
      const u32 tv = ld_agt(tmo);
      if (!ok2 || rfl(tv) == call) {
#pragma unroll
        for (int i = 0; i < 16; ++i) O[i] = 0x7FC07FC0u;
      }
      char* d = outp + ((size_t)((wave >> 2) * 1024 + tm * 64 + rh * 32) * N_ + (wave & 3) * 512 + tn * 64) * 2;
#pragma unroll
      for (int cc = 0; cc < 4; ++cc) st16(d + cc * 8 * N_ * 2, ovo, mk4(O + 4 * cc));
      const u32x4 z = {0, 0, 0, 0};
#pragma unroll
      for (int pl = 0; pl < 3; ++pl) st16(myws + si + pl * 1024, voff, z);
    }
  }
  if (tid == 0) st_agt(myws + OFF_CNT + wg * 128, call);
  STAMP(11);
}

extern "C" void kernel_launch(int rank, int nranks, ncclComm_t comm, hipStream_t stream, void* const* d_in, const int* in_sizes, int n_in,
                              void* d_out, int out_size, const int* in_dtypes, int out_dtype, void* const* d_out_all, void* const* d_ws_all, size_t ws_bytes) {
  if (nranks != NR || ws_bytes < WS_NEED || (unsigned)rank >= (unsigned)NR) { fprintf(stderr, "bad config\n"); return; }
  Args a;
  for (int p = 0; p < NR; ++p) a.ws[p] = (char*)d_ws_all[p];
  a.out = (char*)d_out; a.x = (const char*)d_in[0]; a.w = (const char*)d_in[1]; a.myws = a.ws[rank]; a.rank = rank; a.pad = 0;
  fused_gemm_ar<<<NWG, NTHR, 0, stream>>>(a);
}
```

```cpp
#include <hip/hip_runtime.h>
#include <rccl/rccl.h>
#include <cstdio>

#define DIAG 0
#define WIN 1
#define FORCE_RAW 0
#define ZDELAY 0
#define EARLY 1
#define UNR 0
#define ORD 3
#define HFIRST 1
#define DEFER2 0
#define STAG 0
#define FIRSTD 1
#define ND 0
#define NDM 0
#define TMOX 3
#define IBO 0
#define SPOLL 15
#define SSLEEP 4
#define GATE 2
#define GSLEEP 4
#define SWAP02 0
#define OPERM 0
#define G0 1
#define SWAPROLE 1
#define PACEW 9
#define NOZERO 0
#define PATH1 0
#define BARC 1

typedef unsigned u32;
typedef unsigned long long u64;
typedef unsigned u32x4 __attribute__((ext_vector_type(4)));
typedef unsigned short u16x2 __attribute__((ext_vector_type(2)));
typedef short bf16x8 __attribute__((ext_vector_type(8)));
typedef float f32x16 __attribute__((ext_vector_type(16)));
typedef float f32x4 __attribute__((ext_vector_type(4)));
#define LAS __attribute__((address_space(3)))
#define GAS __attribute__((address_space(1)))

constexpr int NR = 8;
constexpr int N_ = 2048;
constexpr int NWG = 256, NTHR = 512;
constexpr u64 DEADLINE = 20000000ull;
constexpr u32 UNIT = 4096;

constexpr u32 OFF_CNT = 0;
constexpr u32 OFF_TMO = 64 * 1024;
constexpr u32 OFF_FA = 80 * 1024;
constexpr u32 OFF_FB = OFF_FA + 16 * 8 * 8 * 8;
constexpr u32 OFF_FO = 96 * 1024;
constexpr u32 OFF_FL = 128 * 1024;
constexpr u32 OFF_PC = 160 * 1024;
constexpr u32 OFF_DIAG = 256 * 1024;
constexpr u32 OFF_A = 1u << 20;
constexpr u32 OFF_B = OFF_A + 16 * 8 * 8 * UNIT;
constexpr u32 AB_BYTES = (16 + 8) * 8 * 8 * UNIT;
constexpr u32 OFF_IN = OFF_A + 2 * AB_BYTES;
constexpr u32 OFF_OB = OFF_IN + 8 * 128 * 2 * UNIT;
constexpr u32 OB_BYTES = 128 * 2 * UNIT;
constexpr u32 WS_NEED = OFF_OB + 2 * OB_BYTES;

constexpr u32 L_IMG = 64 * 1024;
constexpr u32 L_HDR = 28 * UNIT;
constexpr u32 LDS_BYTES = L_HDR + 128;

struct Args { const char* x; const char* w; char* out; char* myws; int rank; int pad; char* ws[NR]; };

__device__ __forceinline__ void st16(char* base, u32 voff, u32x4 v) {
  asm volatile("s_nop 4\n\tglobal_store_dwordx4 %0, %1, %2 sc0 sc1\n\ts_nop 1" ::"v"(voff), "v"(v), "s"(base) : "memory");
}
__device__ __forceinline__ u64 ld_sys64(const char* p) { return __hip_atomic_load((const GAS u64*)p, __ATOMIC_RELAXED, __HIP_MEMORY_SCOPE_SYSTEM); }
__device__ __forceinline__ u32 ld_agt(const char* p) { return __hip_atomic_load((const GAS u32*)p, __ATOMIC_RELAXED, __HIP_MEMORY_SCOPE_AGENT); }
__device__ __forceinline__ void st_sys64(char* p, u64 v) { __hip_atomic_store((GAS u64*)p, v, __ATOMIC_RELAXED, __HIP_MEMORY_SCOPE_SYSTEM); }
__device__ __forceinline__ void st_agt(char* p, u32 v) { __hip_atomic_store((GAS u32*)p, v, __ATOMIC_RELAXED, __HIP_MEMORY_SCOPE_AGENT); }
#define WAITV(n) asm volatile("s_waitcnt vmcnt(" #n ")" ::: "memory")
#define WAITL() asm volatile("s_waitcnt lgkmcnt(0)" ::: "memory")
#define RAWBAR() do { asm volatile("" ::: "memory"); __builtin_amdgcn_s_barrier(); asm volatile("" ::: "memory"); } while (0)
__device__ __forceinline__ u32 rfl(u32 v) { return __builtin_amdgcn_readfirstlane(v); }
__device__ __forceinline__ u32 rl(u32 v, int l) { return __builtin_amdgcn_readlane(v, l); }
#define STAMP(i) do {} while (0)
#define LSTAMP(i) do {} while (0)

__device__ __forceinline__ u32 wave_wait64(const char* flag, u32 call, char* tmo, bool& ok) {
  const u64 t0 = __builtin_amdgcn_s_memrealtime();
  u32 data = 0;
  for (;;) {
    u64 v = (u64)call << 32;
    if (flag) v = ld_sys64(flag);
    const u32 tv = ld_agt(tmo);
    data = (u32)v;
    if (__all((u32)(v >> 32) >= call)) { ok = true; break; }
    if (tv == call || __builtin_amdgcn_s_memrealtime() - t0 > DEADLINE) { st_agt(tmo, call); ok = false; break; }
    __builtin_amdgcn_s_sleep(1);
  }
  asm volatile("" ::: "memory");
  return data;
}

__device__ __forceinline__ bool swait2(const char* p, u32 call, char* tmo, u32& h0, u32& h1) {
  const u64 t0 = __builtin_amdgcn_s_memrealtime();
  for (u32 it = 1;; ++it) {
    u32x4 v;
    asm volatile("s_load_dwordx4 %0, %1, 0x0 glc\n\ts_waitcnt lgkmcnt(0)" : "=&s"(v) : "s"(p) : "memory");
    h0 = v[0]; h1 = v[2];
    if (v[1] >= call && v[3] >= call) return true;
    if ((it & 63) == 0) {
      if (rfl(ld_agt(tmo)) == call || __builtin_amdgcn_s_memrealtime() - t0 > DEADLINE) { st_agt(tmo, call); return false; }
    }
    __builtin_amdgcn_s_sleep(SSLEEP);
  }
}

typedef unsigned u32x16 __attribute__((ext_vector_type(16)));
typedef unsigned u32x2 __attribute__((ext_vector_type(2)));
__device__ __forceinline__ u32 minu(u32 a, u32 b) { return a < b ? a : b; }
#define SPOLL_TAIL \
    if ((it & 63) == 0) { \
      if (rfl(ld_agt(tmo)) == call || __builtin_amdgcn_s_memrealtime() - t0 > DEADLINE) { st_agt(tmo, call); return false; } \
    } \
    __builtin_amdgcn_s_sleep(GSLEEP);
__device__ __forceinline__ bool swait8(const char* p, u32 call, char* tmo, u32 (&h)[8]) {
  const u64 t0 = __builtin_amdgcn_s_memrealtime();
  for (u32 it = 1;; ++it) {
    u32x16 v;
    asm volatile("s_load_dwordx16 %0, %1, 0x0 glc\n\ts_waitcnt lgkmcnt(0)" : "=&s"(v) : "s"(p) : "memory");
#pragma unroll
    for (int i = 0; i < 8; ++i) h[i] = v[2 * i];
    if (minu(minu(minu(v[1], v[3]), minu(v[5], v[7])), minu(minu(v[9], v[11]), minu(v[13], v[15]))) >= call) return true;
    SPOLL_TAIL
  }
}
__device__ __forceinline__ bool swait4(const char* p, u32 call, char* tmo, u32 (&h)[4]) {
  const u64 t0 = __builtin_amdgcn_s_memrealtime();
  for (u32 it = 1;; ++it) {
    u32x2 a, b, c, d;
    asm volatile("s_load_dwordx2 %0, %4, 0x0 glc\n\ts_load_dwordx2 %1, %4, 0x10 glc\n\ts_load_dwordx2 %2, %4, 0x20 glc\n\ts_load_dwordx2 %3, %4, 0x30 glc\n\ts_waitcnt lgkmcnt(0)"
                 : "=&s"(a), "=&s"(b), "=&s"(c), "=&s"(d) : "s"(p) : "memory");
    h[0] = a[0]; h[1] = b[0]; h[2] = c[0]; h[3] = d[0];
    if (minu(minu(a[1], b[1]), minu(c[1], d[1])) >= call) return true;
    SPOLL_TAIL
  }
}
__device__ __forceinline__ bool swait1(const char* p, u32 call, char* tmo, u32& h) {
  const u64 t0 = __builtin_amdgcn_s_memrealtime();
  for (u32 it = 1;; ++it) {
    u32x2 a;
    asm volatile("s_load_dwordx2 %0, %1, 0x0 glc\n\ts_waitcnt lgkmcnt(0)" : "=&s"(a) : "s"(p) : "memory");
    h = a[0];
    if (a[1] >= call) return true;
    SPOLL_TAIL
  }
}

__device__ __forceinline__ u32 perm(u32 hi, u32 lo, u32 sel) { return __builtin_amdgcn_perm(hi, lo, sel); }
__device__ __forceinline__ u32 rotr(u32 x, u32 n) { return __builtin_amdgcn_alignbit(x, x, n); }
__device__ __forceinline__ u32 bfi(u32 m, u32 a, u32 b) { return (a & m) | (b & ~m); }
__device__ __forceinline__ u32 pkmax(u32 a, u32 b) { return __builtin_bit_cast(u32, __builtin_elementwise_max(__builtin_bit_cast(u16x2, a), __builtin_bit_cast(u16x2, b))); }
template <int CTRL, int RM> __device__ __forceinline__ u32 dppmax(u32 m) { u32 t = __builtin_amdgcn_update_dpp(0u, m, CTRL, RM, 0xf, false); return t > m ? t : m; }

constexpr u32 HDR_RAW = 127u | (63u << 13) | (63u << 25);
constexpr u32 KX = 0x80008080u;
__device__ __forceinline__ u32 min3u(u32 a, u32 b, u32 c) { const u32 m = a < b ? a : b; return m < c ? m : c; }

__device__ __forceinline__ u32 pack_unit(const u32 (&O)[16], u32 (&P)[12], int lane) {
  u32 H[8], U[8], L[8];
#pragma unroll
  for (int m = 0; m < 8; ++m) { L[m] = perm(O[2 * m + 1], O[2 * m], 0x06040200u); H[m] = perm(O[2 * m + 1], O[2 * m], 0x07050301u); }
  u32 e = 0, o = 0;
#pragma unroll
  for (int m = 0; m < 8; ++m) { const u32 h7 = H[m] & 0x7F7F7F7Fu; e = pkmax(e, h7); o = pkmax(o, h7 << 8); }
  u32 mx = pkmax(e, o);
  mx = pkmax(mx, mx >> 16) & 0xFF00u;
  mx = dppmax<0x111, 0xf>(mx); mx = dppmax<0x112, 0xf>(mx); mx = dppmax<0x114, 0xf>(mx); mx = dppmax<0x118, 0xf>(mx);
  mx = dppmax<0x142, 0xa>(mx); mx = dppmax<0x143, 0xc>(mx);
  const u32 mc = rl(mx, 63) >> 8;
  const u32 base = mc > 7 ? mc - 7 : 0;
  const u32 baseB = base * 0x01010101u;
  u32 orr = 0;
#pragma unroll
  for (int m = 0; m < 8; ++m) { U[m] = (H[m] | 0x80808080u) - baseB; orr |= U[m]; }
  const bool bad = (orr & 0x78787878u) != 0;
  const u64 mask = __ballot(bad);
  u32 hdr = base | (63u << 13) | (63u << 25);
  if (mask) {
    u32 eo = 0, ea = 0xFFFFFFFFu, z = 0;
#pragma unroll
    for (int m = 0; m < 8; ++m) {
      const u32 bl = ~U[m] & 0x80808080u;
      const u32 mk = bl | (bl - (bl >> 7));
      eo |= U[m] & mk; ea &= U[m] | ~mk;
      const u32 t = U[m] ^ 0x80808080u;
      z |= (t - 0x01010101u) & ~t & 0x80808080u;
      U[m] = bfi(mk, 0x80808080u, U[m]);
    }
    eo |= eo >> 16; eo |= eo >> 8; eo &= 0xFFu;
    ea &= ea >> 16; ea &= ea >> 8; ea &= 0xFFu;
    const bool lane_ok = z == 0 && eo == ea && eo >= 0x41u;
    const u64 fail = __ballot(bad && !lane_ok);
    if (fail || __popcll(mask) > 2) hdr = HDR_RAW;
    else {
      const int l0 = __ffsll((long long)mask) - 1, l1 = 63 - __clzll((long long)mask);
      const u32 d0 = 0x7Fu - rl(eo, l0), d1 = 0x7Fu - rl(eo, l1);
      hdr = base | (l0 << 7) | (d0 << 13) | (l1 << 19) | (d1 << 25);
    }
  }
  u32 C[4];
#pragma unroll
  for (int t = 0; t < 4; ++t) {
    const u32 ya = bfi(0x80808080u, H[2 * t], U[2 * t]), yb = bfi(0x80808080u, H[2 * t + 1], U[2 * t + 1]);
    C[t] = (ya ^ KX) + rotr(yb, 28);
  }
  if (__ballot(min3u(C[0], C[1], C[2]) == 0)) hdr = HDR_RAW;
  P[0] = L[0]; P[1] = L[1]; P[2] = L[2]; P[3] = C[0];
  P[4] = L[3]; P[5] = L[4]; P[6] = L[5]; P[7] = C[1];
  P[8] = L[6]; P[9] = L[7]; P[10] = C[2]; P[11] = C[3];
  return hdr;
}

__device__ __forceinline__ void merge_lo_hi(u32x4 p0, u32x4 p1, u32x4 p2, int t, u32 h0, u32 h1, u32 (&O)[16]) {
  const u32 l0 = t == 0 ? p0[0] : t == 1 ? p0[2] : t == 2 ? p1[1] : p2[0], l1 = t == 0 ? p0[1] : t == 1 ? p1[0] : t == 2 ? p1[2] : p2[1];
  O[4 * t + 0] = perm(h0, l0, 0x05010400u); O[4 * t + 1] = perm(h0, l0, 0x07030602u);
  O[4 * t + 2] = perm(h1, l1, 0x05010400u); O[4 * t + 3] = perm(h1, l1, 0x07030602u);
}
__device__ __forceinline__ void unpack_unit(u32x4 p0, u32x4 p1, u32x4 p2, u32 hdr, int lane, u32 (&O)[16]) {
  const u32 base = hdr & 127u;
  const u32 baseB = base * 0x01010101u;
  const u32 d0 = (hdr >> 13) & 63u;
  if (d0 == 63u) {
#pragma unroll
    for (int t = 0; t < 4; ++t) {
      const u32 n = t == 0 ? p0[3] : t == 1 ? p1[3] : t == 2 ? p2[2] : p2[3];
      merge_lo_hi(p0, p1, p2, t, ((n & 0x87878787u) ^ KX) + baseB, (rotr(n, 4) & 0x87878787u) + baseB, O);
    }
  } else {
    const u32 T0 = baseB + 0x03020100u, T1 = T0 + 0x04040404u;
    const u32 d1 = (hdr >> 25) & 63u;
    const u32 Ta = (T0 & ~0xFFu) | (base - 1u - d0), Tb = (T0 & ~0xFFu) | (base - 1u - d1);
    u32 Tv = lane == (int)((hdr >> 19) & 63u) ? Tb : T0;
    Tv = lane == (int)((hdr >> 7) & 63u) ? Ta : Tv;
#pragma unroll
    for (int t = 0; t < 4; ++t) {
      const u32 n = t == 0 ? p0[3] : t == 1 ? p1[3] : t == 2 ? p2[2] : p2[3], r = rotr(n, 4);
      merge_lo_hi(p0, p1, p2, t, perm(T1, Tv, n & 0x07070707u) | ((n & 0x80808080u) ^ KX), perm(T1, Tv, r & 0x07070707u) | (r & 0x80808080u), O);
    }
  }
}

__device__ __forceinline__ float silu(float y) {
  const float c = fminf(fmaxf(y, -60.f), 60.f);
  return y * __builtin_amdgcn_rcpf(1.f + __expf(-c));
}
__device__ __forceinline__ u32 pack_bf16(float lo, float hi) {
  typedef __bf16 bf2 __attribute__((ext_vector_type(2)));
  bf2 r; r.x = (__bf16)lo; r.y = (__bf16)hi; return __builtin_bit_cast(u32, r);
}
__device__ __forceinline__ u32x4 mk4(const u32* p) { u32x4 v = {p[0], p[1], p[2], p[3]}; return v; }

struct Unit { u32x4 p[3]; };
__device__ __forceinline__ void load_unit(Unit& u, __amdgpu_buffer_rsrc_t rs, u32 voff, u32 soff) {
  u.p[0] = __builtin_amdgcn_raw_buffer_load_b128(rs, voff, soff, 17);
  u.p[1] = __builtin_amdgcn_raw_buffer_load_b128(rs, voff + 1024, soff, 17);
  u.p[2] = __builtin_amdgcn_raw_buffer_load_b128(rs, voff + 2048, soff, 17);
}
__device__ __forceinline__ void decode_unit(Unit& u, __amdgpu_buffer_rsrc_t rs, const char* ws, u32 voff, u32 soff, u32 hdr, int lane, u32 call, char* tmo, u32 (&O)[16], u32& nretry, bool& bad) {
  if ((hdr & 127u) == 127u) {
    u32x4 p3;
    asm volatile("s_nop 4\n\tglobal_load_dwordx4 %0, %1, %2 offset:3072 sc0 sc1\n\ts_waitcnt vmcnt(0)" : "=&v"(p3) : "v"(voff), "s"(ws + soff) : "memory");
#pragma unroll
    for (int i = 0; i < 12; ++i) O[i] = u.p[i >> 2][i & 3];
#pragma unroll
    for (int i = 0; i < 4; ++i) O[12 + i] = p3[i];
  } else {
    if (__builtin_expect(__ballot(min3u(u.p[0][3], u.p[1][3], u.p[2][2]) == 0) != 0, 0)) {
      const u64 t0 = __builtin_amdgcn_s_memrealtime();
      for (;;) {
        load_unit(u, rs, voff, soff);
        const u32 tv = rfl(ld_agt(tmo));
        if (!__ballot(min3u(u.p[0][3], u.p[1][3], u.p[2][2]) == 0)) break;
        if (tv == call || __builtin_amdgcn_s_memrealtime() - t0 > DEADLINE) { st_agt(tmo, call); bad = true; break; }
      }
    }
    unpack_unit(u.p[0], u.p[1], u.p[2], hdr, lane, O);
  }
}

__device__ __forceinline__ int second_tile(int grp) { return SWAP02 == 1 ? (grp == 0 ? 6 : grp == 2 ? 4 : grp + 4) : SWAP02 == 2 ? (grp ^ 2) + 4 : grp + 4; }
__device__ __forceinline__ int out_blk(int i) { return OPERM && (i == 1 || i == 2) ? 3 - i : i; }
__device__ __forceinline__ bool round_is_w(int r) { return r == 0 || r == 5; }
__device__ __forceinline__ int round_idx(int r) { return r == 0 ? 0 : r == 5 ? 1 : r - 1; }
__device__ __forceinline__ int tile_m(int o) { return o & 3; }
__device__ __forceinline__ int tile_n(int o) { return o >> 2; }
__global__ __launch_bounds__(NTHR, 2) void fused_gemm_ar(Args a) {
  __shared__ __attribute__((aligned(16))) char smem_[LDS_BYTES];
  LAS char* smem = (LAS char*)smem_;
  const int wg = blockIdx.x ^ (SWAPROLE ? 128 : 0), tid = threadIdx.x, lane = tid & 63;
  const int wave = rfl(tid >> 6);
  const char* xin = a.x; const char* win = a.w; char* outp = a.out; char* myws = a.myws; int rank = a.rank; char* dws = a.ws[wg & 7];
  asm volatile("" : "+s"(xin), "+s"(win), "+s"(outp), "+s"(myws), "+s"(rank), "+s"(dws));
  char* tmo = myws + OFF_TMO;
  STAMP(0);
  const char* cntp = myws + OFF_CNT + wg * 128;

  if (wg < 128) {
    const int p = wg & 7, j = wg >> 3;
    const int q = j & 3, blk = j >> 2;
    const bool self = p == rank;
    LAS u32* hdrs = (LAS u32*)(smem + L_HDR);
    if (wave == 0) {
      const u32 callv = ld_agt(cntp);
      if (lane < 32) hdrs[lane] = 0;
      WAITL(); RAWBAR();
      const u32 call = rfl(callv) + 1;
      const int nr = self ? 6 : 14;
      const u32 par = (call & 1) * AB_BYTES;
      if (G0) {
        bool ok;
        (void)wave_wait64(lane < 2 ? myws + OFF_PC + wg * 128 + lane * 8 : nullptr, call, tmo, ok);
        LSTAMP(1);
      }
#pragma unroll 1
      for (int r = G0 ? 1 : 0; r < nr; ++r) {
        if (r >= 6 && ((NDM >> (r - 6)) & 1)) continue;
        u32 h0, h1;
        {
          u64 hv;
          for (;;) {
            asm volatile("ds_read_b64 %0, %1\n\ts_waitcnt lgkmcnt(0)" : "=v"(hv) : "v"((u32)(uintptr_t)(hdrs + 2 * r)) : "memory");
            if (rfl((u32)hv) >> 31 & rfl((u32)(hv >> 32)) >> 31) break;
            __builtin_amdgcn_s_sleep(1);
          }
          h0 = rfl((u32)hv) & 0x7FFFFFFFu; h1 = rfl((u32)(hv >> 32)) & 0x7FFFFFFFu;
        }
        if (FIRSTD && !G0 && r == 0) { LSTAMP(1); continue; }
        LSTAMP(2 * r);
        u32 dslot, fslot;
        if (r >= 6) { const u32 u = (rank * 128 + (r - 6) * 16 + j) * 2; dslot = OFF_IN + u * UNIT; fslot = OFF_FO + u * 8; }
        else if (round_is_w(r)) { const u32 u = ((round_idx(r) * 4 + blk) * 8 + rank) * 8 + q * 2; dslot = OFF_B + par + u * UNIT; fslot = OFF_FB + u * 8; }
        else { const u32 u = ((round_idx(r) * 4 + blk) * 8 + rank) * 8 + q * 2; dslot = OFF_A + par + u * UNIT; fslot = OFF_FA + u * 8; }
        const LAS char* src = smem + r * 2 * UNIT + lane * 16;
        char* dst = dws + dslot;
        const u32 vo = lane * 16;
        const u32x4 v0 = *(const LAS u32x4*)(src), v1 = *(const LAS u32x4*)(src + 1024), v2 = *(const LAS u32x4*)(src + 2048);
        const u32x4 v4 = *(const LAS u32x4*)(src + UNIT), v5 = *(const LAS u32x4*)(src + UNIT + 1024), v6 = *(const LAS u32x4*)(src + UNIT + 2048);
        const bool raw0 = (h0 & 127u) == 127u, raw1 = (h1 & 127u) == 127u;
        const bool hfirst = HFIRST && EARLY && (r >= 6 || HFIRST == 2 || (HFIRST == 3 && !round_is_w(r)) || (HFIRST == 4 && r != 0)) && !raw0 && !raw1;
        if (hfirst && lane < 2) st_sys64(dws + fslot + lane * 8, ((u64)call << 32) | (lane == 0 ? h0 : h1));
        if (raw0) st16(dst, vo + 3072, *(const LAS u32x4*)(src + 3072));
        if (raw1) st16(dst, vo + UNIT + 3072, *(const LAS u32x4*)(src + UNIT + 3072));
        st16(dst, vo, v0); st16(dst, vo + 1024, v1); st16(dst, vo + 2048, v2);
        st16(dst, vo + UNIT, v4); st16(dst, vo + UNIT + 1024, v5); st16(dst, vo + UNIT + 2048, v6);
        if (!EARLY || raw0 || raw1) WAITV(0);
        if (!hfirst && lane < 2) st_sys64(dws + fslot + lane * 8, ((u64)call << 32) | (lane == 0 ? h0 : h1));
        if (GATE < 9 && lane == 0) hdrs[28] = r;
        WAITV(0);
        LSTAMP(2 * r + 1);
      }
      if (lane == 0) st_agt(myws + OFF_CNT + wg * 128, call);
      STAMP(6);
      return;
    }
    const int k = wave - 1;
    const int g1 = k + (G0 ? 2 : 0), r1 = g1 >> 1, i1 = g1 & 1;
    const bool w1 = round_is_w(r1);
    const int c = lane & 31, h = lane >> 5;
    const __amdgpu_buffer_rsrc_t rx = __builtin_amdgcn_make_buffer_rsrc((void*)xin, 0, 2048 * 512, 0x00020000);
    const __amdgpu_buffer_rsrc_t rw = __builtin_amdgcn_make_buffer_rsrc((void*)win, 0, 256 * 4096, 0x00020000);
    const u32 xv = c * 512 + h * 32, xs0 = ((p >> 2) * 1024 + blk * 64) * 512 + q * 128;
    const u32 wv = h * 16 * 4096 + c * 4, ws0 = q * 64 * 4096 + ((p & 3) * 512 + blk * 64) * 2;
    u32 D1[16], D2[16];
    auto loadw = [&](u32 (&D)[16], int r, int i) {
      const u32 so = ws0 + i * 32 * 4096 + round_idx(r) * 512;
#pragma unroll
      for (int rr = 0; rr < 16; ++rr) D[rr] = __builtin_amdgcn_raw_buffer_load_b32(rw, wv + (rr & 0) , so + rr * 4096, 0);
    };
    auto loadx = [&](u32 (&D)[16], int r, int i) {
      const u32 so = xs0 + round_idx(r) * 256 * 512 + i * 32 * 512;
#pragma unroll
      for (int cc = 0; cc < 4; ++cc) { const u32x4 t = __builtin_amdgcn_raw_buffer_load_b128(rx, xv + (cc >> 1) * 64 + (cc & 1) * 16, so, 0); D[4 * cc] = t[0]; D[4 * cc + 1] = t[1]; D[4 * cc + 2] = t[2]; D[4 * cc + 3] = t[3]; }
    };
    if (STAG && k >= 2) __builtin_amdgcn_s_sleep(STAG);
    if (w1) loadw(D1, r1, i1); else loadx(D1, r1, i1);
    const int g2 = g1 + 7, r2 = g2 >> 1, i2 = g2 & 1;
    const bool two = g2 < 12;
    const bool w2 = round_is_w(r2);
    auto load2 = [&]() { if (two) { if (w2) loadw(D2, r2, i2); else loadx(D2, r2, i2); } };
    if (!DEFER2) load2();
    const u32 callv = ld_agt(cntp);
    STAMP(4);
    RAWBAR();
    STAMP(5);
#pragma unroll
    for (int job = 0; job < 2; ++job) {
      if (job == 1 && !two) break;
      u32 O[16];
      if (job == 1 ? w2 : w1) {
        const u32* D = job == 1 ? D2 : D1;
#pragma unroll
        for (int cb = 0; cb < 4; ++cb)
#pragma unroll
          for (int e = 0; e < 4; ++e) O[4 * cb + e] = perm(D[8 * (cb & 1) + 2 * e + 1], D[8 * (cb & 1) + 2 * e], (cb >> 1) ? 0x07060302u : 0x05040100u);
      } else {
#pragma unroll
        for (int i = 0; i < 16; ++i) O[i] = job == 1 ? D2[i] : D1[i];
      }
      if (DEFER2 && job == 0) { asm volatile("" :: "v"(O[0]), "v"(O[15]) : "memory"); load2(); __builtin_amdgcn_sched_barrier(0); }
      const int slot = job == 0 ? g1 : g2;
      u32 P[12];
      const u32 hdr = pack_unit(O, P, lane);
      if (FIRSTD && !G0 && job == 0 && k < 2) {
        const u32 u = ((round_idx(0) * 4 + blk) * 8 + rank) * 8 + q * 2 + k;
        const u32 call0 = rfl(callv) + 1;
        char* dst = dws + OFF_B + (call0 & 1) * AB_BYTES + u * UNIT;
        char* fl = dws + OFF_FB + u * 8;
        if ((hdr & 127u) == 127u) {
#pragma unroll
          for (int pl = 0; pl < 4; ++pl) st16(dst, lane * 16 + pl * 1024, mk4(O + 4 * pl));
          WAITV(0);
        } else {
#pragma unroll
          for (int pl = 0; pl < 3; ++pl) st16(dst, lane * 16 + pl * 1024, mk4(P + 4 * pl));
        }
        if (lane == 0) st_sys64(fl, ((u64)call0 << 32) | hdr);
        if (k == 0) LSTAMP(0);
        WAITV(0);
        if (lane == 0) hdrs[k] = 0x80000000u;
        continue;
      }
      LAS char* d = smem + slot * UNIT + lane * 16;
      if ((hdr & 127u) == 127u) {
#pragma unroll
        for (int pl = 0; pl < 4; ++pl) *(LAS u32x4*)(d + pl * 1024) = mk4(O + 4 * pl);
      } else {
#pragma unroll
        for (int pl = 0; pl < 3; ++pl) *(LAS u32x4*)(d + pl * 1024) = mk4(P + 4 * pl);
      }
      WAITL();
      if (lane == 0) hdrs[slot] = hdr | 0x80000000u;
    }
    const u32 call = rfl(callv) + 1;
    STAMP(1);
    if (!self) {
      for (int o = k; o < 8; o += 7) {
        if ((NDM >> o) & 1) continue;
        const int tile = o * 16 + j;
        bool ok;
        u32 h0, h1;
        if (GATE < 9) {
          for (;;) {
            u32 pr;
            asm volatile("ds_read_b32 %0, %1\n\ts_waitcnt lgkmcnt(0)" : "=v"(pr) : "v"((u32)(uintptr_t)(hdrs + 28)) : "memory");
            if ((int)rfl(pr) >= 6 + o - GATE) break;
            __builtin_amdgcn_s_sleep(4);
          }
        }
        if (SPOLL & 1) ok = swait2(myws + OFF_FL + tile * 128, call, tmo, h0, h1);
        else { const u32 hv = wave_wait64(lane < 2 ? myws + OFF_FL + tile * 128 + lane * 8 : nullptr, call, tmo, ok); h0 = rl(hv, 0); h1 = rl(hv, 1); }
        const char* src = myws + OFF_OB + (IBO ? (call & 1) * OB_BYTES : 0) + tile * 2 * UNIT + lane * 16;
        LAS char* d = smem + (12 + 2 * o) * UNIT;
        if (ok) {
          const bool raw0 = (h0 & 127u) == 127u, raw1 = (h1 & 127u) == 127u;
          const u64 t0 = __builtin_amdgcn_s_memrealtime();
          for (;;) {
#pragma unroll
            for (int pl = 0; pl < 3; ++pl) {
              __builtin_amdgcn_global_load_lds((const GAS void*)(src + pl * 1024), (LAS void*)(d + pl * 1024), 16, 0, 17);
              __builtin_amdgcn_global_load_lds((const GAS void*)(src + UNIT + pl * 1024), (LAS void*)(d + UNIT + pl * 1024), 16, 0, 17);
            }
            if (raw0) __builtin_amdgcn_global_load_lds((const GAS void*)(src + 3072), (LAS void*)(d + 3072), 16, 0, 17);
            if (raw1) __builtin_amdgcn_global_load_lds((const GAS void*)(src + UNIT + 3072), (LAS void*)(d + UNIT + 3072), 16, 0, 17);
            WAITV(0);
            if (!IBO) break;
            const LAS char* cw = d + lane * 16;
            const u32 m0 = min3u(*(const LAS u32*)(cw + 12), *(const LAS u32*)(cw + 1024 + 12), *(const LAS u32*)(cw + 2048 + 8));
            const u32 m1 = min3u(*(const LAS u32*)(cw + UNIT + 12), *(const LAS u32*)(cw + UNIT + 1024 + 12), *(const LAS u32*)(cw + UNIT + 2048 + 8));
            if (!__ballot((!raw0 && m0 == 0) || (!raw1 && m1 == 0))) break;
            if (rfl(ld_agt(tmo)) == call || __builtin_amdgcn_s_memrealtime() - t0 > DEADLINE) { st_agt(tmo, call); ok = false; break; }
          }
        }
        if (!ok) {
          h0 = HDR_RAW; h1 = HDR_RAW;
          const u32x4 nan4 = {0x7FC07FC0u, 0x7FC07FC0u, 0x7FC07FC0u, 0x7FC07FC0u};
#pragma unroll
          for (int pl = 0; pl < 8; ++pl) *(LAS u32x4*)(d + pl * 1024 + lane * 16) = nan4;
          WAITL();
        }
        if (lane < 2) hdrs[12 + 2 * o + lane] = (lane == 0 ? h0 : h1) | 0x80000000u;
        STAMP(2);
      }
    }
    return;
  }

  const int gw = wg - 128;
  const int x = gw & 7, slot = gw >> 3, grp = slot >> 2, m = slot & 3;
  const int rh = m & 1, ul = (x >> 1) * 4 + (x & 1) * 2 + (m >> 1);
  const __amdgpu_buffer_rsrc_t rs = __builtin_amdgcn_make_buffer_rsrc((void*)myws, 0, (int)WS_NEED, 0x00020000);
  const u32 voff = lane * 16;
  u32 call;
  if (G0 && wave < 2) {
    const int q = slot & 3, blk = slot >> 2;
    const int c = lane & 31, h = lane >> 5;
    const __amdgpu_buffer_rsrc_t rw = __builtin_amdgcn_make_buffer_rsrc((void*)win, 0, 256 * 4096, 0x00020000);
    const u32 wv = h * 16 * 4096 + c * 4, so = (q * 64 + wave * 32) * 4096 + ((x & 3) * 512 + blk * 64) * 2;
    u32 D[16];
#pragma unroll
    for (int rr = 0; rr < 16; ++rr) D[rr] = __builtin_amdgcn_raw_buffer_load_b32(rw, wv, so + rr * 4096, 0);
    const u32 callv = ld_agt(cntp);
    u32 O[16], P[12];
#pragma unroll
    for (int cb = 0; cb < 4; ++cb)
#pragma unroll
      for (int e = 0; e < 4; ++e) O[4 * cb + e] = perm(D[8 * (cb & 1) + 2 * e + 1], D[8 * (cb & 1) + 2 * e], (cb >> 1) ? 0x07060302u : 0x05040100u);
    const u32 hdr = pack_unit(O, P, lane);
    call = rfl(callv) + 1;
    const u32 u = (blk * 8 + rank) * 8 + q * 2 + wave;
    char* dst = dws + OFF_B + (call & 1) * AB_BYTES + u * UNIT;
    if ((hdr & 127u) == 127u) {
#pragma unroll
      for (int pl = 0; pl < 4; ++pl) st16(dst, voff + pl * 1024, mk4(O + 4 * pl));
      WAITV(0);
    } else {
#pragma unroll
      for (int pl = 0; pl < 3; ++pl) st16(dst, voff + pl * 1024, mk4(P + 4 * pl));
    }
    if (lane == 0) st_sys64(dws + OFF_FB + u * 8, ((u64)call << 32) | hdr);
    if (lane == 0) st_sys64(myws + OFF_PC + gw * 128 + wave * 8, (u64)call << 32);
  } else {
    call = rfl(ld_agt(cntp)) + 1;
    const u32x4 z = {0, 0, 0, 0};
    char* zb = myws + OFF_A + (AB_BYTES - (call & 1) * AB_BYTES) + gw * 12 * UNIT;
#pragma unroll
    for (int i = 0; i < 6; ++i) {
      const u32 pl = G0 ? wave - 2 + 6 * i : wave + 8 * i, un = (pl * 43) >> 7;
      if (!NOZERO && pl < 36) st16(zb + un * UNIT + (pl - 3 * un) * 1024, voff, z);
    }
    if (IBO && wave >= 2) {
      const int jb = (wave - 2) >= 3, pl = wave - 2 - 3 * jb;
      const int ht2 = ((grp + 4 * jb) * 16 + ul) * 2 + rh;
      if (!NOZERO) st16(myws + OFF_OB + (OB_BYTES - (call & 1) * OB_BYTES) + ht2 * UNIT + pl * 1024, voff, z);
    }
  }
  const u32 par = (call & 1) * AB_BYTES;
  const u32 ovo = (lane >> 3) * (N_ * 2) + (lane & 7) * 16;
  u32 OA[4][16];
  u32 nrB = 0, nrA = 0, nrO = 0;
  bool bad = false;
#pragma unroll 1
  for (int job = 0; job < 2; ++job) {
    const int o = grp + 4 * job;
    const int tm = 4 * tile_m(o) + (ul >> 2), tn = 4 * tile_n(o) + (ul & 3);
    const int ht = (o * 16 + ul) * 2 + rh;
    f32x16 acc[2] = {};
    {
      const u32 sa = OFF_A + par + ((tm * 8 + wave) * 8 + rh) * UNIT, sb = OFF_B + par + (tn * 8 + wave) * 8 * UNIT;
      bool ok;
      u32 hb[8];
      if (SPOLL & 2) ok = swait8(myws + OFF_FB + (tn * 8 + wave) * 64, call, tmo, hb);
      else {
        const u32 hv = wave_wait64(lane < 8 ? myws + OFF_FB + ((tn * 8 + wave) * 8 + lane) * 8 : nullptr, call, tmo, ok);
#pragma unroll
        for (int u = 0; u < 8; ++u) hb[u] = rl(hv, u);
      }
      bad |= !ok;
      STAMP(job == 0 ? 12 : 5);
      Unit ub[8];
#pragma unroll
      for (int u = 0; u < 8; ++u) { load_unit(ub[u], rs, voff, sb + u * UNIT); __builtin_amdgcn_sched_barrier(0); }
      if (job == 0) {
        u32 OB[8][16];
#pragma unroll
        for (int u = 0; u < 8; ++u) decode_unit(ub[u], rs, myws, voff, sb + u * UNIT, hb[u], lane, call, tmo, OB[u], nrB, bad);
        u32 ha[4];
        if (SPOLL & 4) ok = swait4(myws + OFF_FA + (tm * 8 + wave) * 64 + rh * 8, call, tmo, ha);
        else {
          const u32 hv = wave_wait64(lane < 4 ? myws + OFF_FA + ((tm * 8 + wave) * 8 + lane * 2 + rh) * 8 : nullptr, call, tmo, ok);
#pragma unroll
          for (int q = 0; q < 4; ++q) ha[q] = rl(hv, q);
        }
        bad |= !ok;
        STAMP(1);
        Unit ua[4];
#pragma unroll
        for (int q = 0; q < 4; ++q) { load_unit(ua[q], rs, voff, sa + 2 * q * UNIT); __builtin_amdgcn_sched_barrier(0); }
#pragma unroll
        for (int q = 0; q < 4; ++q) {
          decode_unit(ua[q], rs, myws, voff, sa + 2 * q * UNIT, ha[q], lane, call, tmo, OA[q], nrA, bad);
#pragma unroll
          for (int i = 0; i < 2; ++i)
#pragma unroll
            for (int b = 0; b < 2; ++b)
#pragma unroll
              for (int ch = 0; ch < 2; ++ch)
                acc[ch] = __builtin_amdgcn_mfma_f32_32x32x16_bf16(__builtin_bit_cast(bf16x8, mk4(OB[2 * q + i] + 4 * (2 * ch + b))), __builtin_bit_cast(bf16x8, mk4(OA[q] + 4 * (2 * i + b))), acc[ch], 0, 0, 0);
        }
      } else {
#pragma unroll
        for (int u = 0; u < 8; ++u) {
          u32 OB[16];
          decode_unit(ub[u], rs, myws, voff, sb + u * UNIT, hb[u], lane, call, tmo, OB, nrB, bad);
#pragma unroll
          for (int b = 0; b < 2; ++b)
#pragma unroll
            for (int ch = 0; ch < 2; ++ch)
              acc[ch] = __builtin_amdgcn_mfma_f32_32x32x16_bf16(__builtin_bit_cast(bf16x8, mk4(OB + 4 * (2 * ch + b))), __builtin_bit_cast(bf16x8, mk4(OA[u >> 1] + 4 * (2 * (u & 1) + b))), acc[ch], 0, 0, 0);
        }
      }
    }
    STAMP(2 + 4 * job);
    if (__builtin_expect(bad, 0)) {
#pragma unroll
      for (int i = 0; i < 16; ++i) { acc[0][i] = __builtin_nanf(""); acc[1][i] = __builtin_nanf(""); }
    }
    WAITV(0);
    {
      LAS char* pw = smem + wave * 8192 + lane * 16;
#pragma unroll
      for (int ch = 0; ch < 2; ++ch)
#pragma unroll
        for (int g = 0; g < 4; ++g) {
          const f32x4 v = {acc[ch][4 * g], acc[ch][4 * g + 1], acc[ch][4 * g + 2], acc[ch][4 * g + 3]};
          *(LAS f32x4*)(pw + (ch * 4 + g) * 1024) = v;
        }
    }
    WAITL(); RAWBAR();
    {
      typedef float f32x2 __attribute__((ext_vector_type(2)));
      const int g = wave & 3, hf = wave >> 2;
      const LAS char* pr = smem + g * 1024 + lane * 16 + hf * 8;
      f32x2 r0[8], r1[8];
#pragma unroll
      for (int s = 0; s < 8; ++s) { r0[s] = *(const LAS f32x2*)(pr + s * 8192); r1[s] = *(const LAS f32x2*)(pr + s * 8192 + 4096); }
      __builtin_amdgcn_sched_barrier(0);
      const f32x2 a0 = ((r0[0] + r0[1]) + (r0[2] + r0[3])) + ((r0[4] + r0[5]) + (r0[6] + r0[7]));
      const f32x2 a1 = ((r1[0] + r1[1]) + (r1[2] + r1[3])) + ((r1[4] + r1[5]) + (r1[6] + r1[7]));
      u32x2 y;
      y[0] = pack_bf16(silu(a0[0]), silu(a1[0])); y[1] = pack_bf16(silu(a0[1]), silu(a1[1]));
      *(LAS u32x2*)(smem + L_IMG + (lane & 31) * 144 + (2 * g + (lane >> 5)) * 16 + hf * 8) = y;
    }
#if !(TMOX & 1)
    const u32 tv = ld_agt(tmo);
#endif
    WAITL(); RAWBAR();
    STAMP(3 + 4 * job);
    if (wave == 0) {
      u32 O[16];
      const bool badt = false;
#pragma unroll
      for (int cc = 0; cc < 4; ++cc) {
        const u32x4 t = *(const LAS u32x4*)(smem + L_IMG + (cc * 8 + (lane >> 3)) * 144 + (lane & 7) * 16);
#pragma unroll
        for (int e = 0; e < 4; ++e) O[4 * cc + e] = badt ? 0x7FC07FC0u : t[e];
      }
      u32 P[12];
      const u32 hdr = pack_unit(O, P, lane);
      const bool raw = (hdr & 127u) == 127u;
      if ((NDM >> o) & 1) {
        const u32 islot = OFF_IN + (rank * 256 + ht) * UNIT;
        char* pw = a.ws[0];
#pragma unroll
        for (int d = 1; d < 8; ++d) pw = lane == d ? a.ws[d] : pw;
        char* fo = pw + OFF_FO + (rank * 256 + ht) * 8;
        if (!raw && lane < 8 && lane != rank) st_sys64(fo, ((u64)call << 32) | hdr);
#pragma unroll
        for (int d = 0; d < 7; ++d) {
          int dd7 = d + (x + 2 * m) % 7; dd7 = dd7 >= 7 ? dd7 - 7 : dd7;
          char* dst = a.ws[(rank + 1 + dd7) & 7] + islot;
          if (raw) {
#pragma unroll
            for (int pl = 0; pl < 4; ++pl) st16(dst, voff + pl * 1024, mk4(O + 4 * pl));
          } else {
#pragma unroll
            for (int pl = 0; pl < 3; ++pl) st16(dst, voff + pl * 1024, mk4(P + 4 * pl));
          }
        }
        if (raw) { WAITV(0); if (lane < 8 && lane != rank) st_sys64(fo, ((u64)call << 32) | hdr); }
      } else {
        char* d = myws + OFF_OB + (IBO ? (call & 1) * OB_BYTES : 0) + ht * UNIT;
        if (raw) {
#pragma unroll
          for (int pl = 0; pl < 4; ++pl) st16(d, voff + pl * 1024, mk4(O + 4 * pl));
        } else {
#pragma unroll
          for (int pl = 0; pl < 3; ++pl) st16(d, voff + pl * 1024, mk4(P + 4 * pl));
        }
      }
      if (IBO && !raw && !((NDM >> o) & 1) && lane == 0) st_sys64(myws + OFF_FL + (ht >> 1) * 128 + rh * 8, ((u64)call << 32) | hdr);
      char* dd = outp + ((size_t)((rank >> 2) * 1024 + tm * 64 + rh * 32) * N_ + (rank & 3) * 512 + tn * 64) * 2;
#pragma unroll
      for (int cc = 0; cc < 4; ++cc) st16(dd + cc * 8 * N_ * 2, ovo, mk4(O + 4 * cc));
      if (BARC && job == 0) RAWBAR();
      if (!((NDM >> o) & 1) && (!IBO || raw)) {
        WAITV(4);
        if (lane == 0) st_sys64(myws + OFF_FL + (ht >> 1) * 128 + rh * 8, ((u64)call << 32) | hdr);
      }
      STAMP(4 + 4 * job);
    } else if (BARC && job == 0) RAWBAR();
  }
  if (wave != rank) {
#pragma unroll 1
    for (int job = 0; job < 2; ++job) {
      const int o = grp + 4 * job;
      const int tm = 4 * tile_m(o) + (ul >> 2), tn = 4 * tile_n(o) + (ul & 3);
      const int ht = (o * 16 + ul) * 2 + rh;
      bool ok2;
      u32 h0;
      if (SPOLL & 8) ok2 = swait1(myws + OFF_FO + (wave * 256 + ht) * 8, call, tmo, h0);
      else h0 = rl(wave_wait64(lane == 0 ? myws + OFF_FO + (wave * 256 + ht) * 8 : nullptr, call, tmo, ok2), 0);
      STAMP(9 + job);
      u32 O[16];
      const u32 si = OFF_IN + (wave * 256 + ht) * UNIT;
      bool bad2 = !ok2;
      {
        Unit u0;
        load_unit(u0, rs, voff, si);
        decode_unit(u0, rs, myws, voff, si, h0, lane, call, tmo, O, nrO, bad2);
      }
#if !(TMOX & 2)
      bad2 |= rfl(ld_agt(tmo)) == call;
#endif
      if (__builtin_expect(bad2, 0)) {
#pragma unroll
        for (int i = 0; i < 16; ++i) O[i] = 0x7FC07FC0u;
      }
      char* d = outp + ((size_t)((wave >> 2) * 1024 + tm * 64 + rh * 32) * N_ + (wave & 3) * 512 + tn * 64) * 2;
#pragma unroll
      for (int cc = 0; cc < 4; ++cc) st16(d + cc * 8 * N_ * 2, ovo, mk4(O + 4 * cc));
      const u32x4 z = {0, 0, 0, 0};
#pragma unroll
      for (int pl = 0; pl < 3; ++pl) st16(myws + si + pl * 1024, voff, z);
    }
  }
  if (tid == 0) st_agt(myws + OFF_CNT + wg * 128, call);
  STAMP(11);
}

extern "C" void kernel_launch(int rank, int nranks, ncclComm_t comm, hipStream_t stream, void* const* d_in, const int* in_sizes, int n_in,
                              void* d_out, int out_size, const int* in_dtypes, int out_dtype, void* const* d_out_all, void* const* d_ws_all, size_t ws_bytes) {
  if (nranks != NR || ws_bytes < WS_NEED || (unsigned)rank >= (unsigned)NR) { fprintf(stderr, "bad config\n"); return; }
  Args a;
  for (int p = 0; p < NR; ++p) a.ws[p] = (char*)d_ws_all[p];
  a.out = (char*)d_out; a.x = (const char*)d_in[0]; a.w = (const char*)d_in[1]; a.myws = a.ws[rank]; a.rank = rank; a.pad = 0;
  fused_gemm_ar<<<NWG, NTHR, 0, stream>>>(a);
}
```

```cpp
#include <hip/hip_runtime.h>
#include <rccl/rccl.h>
#include <cstdio>

#define DIAG 0
#define WIN 13
#define FORCE_RAW 0
#define ZDELAY 0
#define EARLY 1
#define UNR 0
#define ORD 3
#define HFIRST 1
#define DEFER2 0
#define STAG 0
#define FIRSTD 1
#define ND 0
#define NDM 0
#define TMOX 2
#define IBO 0
#define SPOLL 15
#define SSLEEP 4
#define GATE 3
#define GSLEEP 4
#define PCT 0
#define LMAP 0
#define LMAPSHOW 0
#define BLA 8
#define PIPA 0
#define IBOD 0
#define ZLINK 1
#define ZG 3
#define SWAP02 0
#define OPERM 0
#define G0 1
#define SWAPROLE 1
#define PACEW 9
#define NOZERO 0
#define PATH1 0
#define BARC 1

typedef unsigned u32;
typedef unsigned long long u64;
typedef unsigned u32x4 __attribute__((ext_vector_type(4)));
typedef unsigned short u16x2 __attribute__((ext_vector_type(2)));
typedef short bf16x8 __attribute__((ext_vector_type(8)));
typedef float f32x16 __attribute__((ext_vector_type(16)));
typedef float f32x4 __attribute__((ext_vector_type(4)));
#define LAS __attribute__((address_space(3)))
#define GAS __attribute__((address_space(1)))

constexpr int NR = 8;
constexpr int N_ = 2048;
constexpr int NWG = 256, NTHR = 512;
constexpr u64 DEADLINE = 20000000ull;
constexpr u32 UNIT = 4096;

constexpr u32 OFF_CNT = 0;
constexpr u32 OFF_TMO = 64 * 1024;
constexpr u32 OFF_FA = 80 * 1024;
constexpr u32 OFF_FB = OFF_FA + 16 * 8 * 8 * 8;
constexpr u32 OFF_FO = 96 * 1024;
constexpr u32 OFF_FL = 128 * 1024;
constexpr u32 OFF_PC = 160 * 1024;
constexpr u32 OFF_DIAG = 256 * 1024;
constexpr u32 OFF_A = 1u << 20;
constexpr u32 OFF_B = OFF_A + 16 * 8 * 8 * UNIT;
constexpr u32 AB_BYTES = (16 + 8) * 8 * 8 * UNIT;
constexpr u32 OFF_IN = OFF_A + 2 * AB_BYTES;
constexpr u32 OFF_OB = OFF_IN + 8 * 128 * 2 * UNIT;
constexpr u32 OB_BYTES = 128 * 2 * UNIT;
constexpr u32 WS_NEED = OFF_OB + 2 * OB_BYTES;

constexpr u32 L_IMG = 64 * 1024;
constexpr u32 L_HDR = 28 * UNIT;
constexpr u32 LDS_BYTES = L_HDR + 128;

struct Args { const char* x; const char* w; char* out; char* myws; int rank; u32 pmap; char* ws[NR]; };
static inline int link_dest(int rank, int i) {
  return i;
}

__device__ __forceinline__ void st16(char* base, u32 voff, u32x4 v) {
  asm volatile("s_nop 4\n\tglobal_store_dwordx4 %0, %1, %2 sc0 sc1\n\ts_nop 1" ::"v"(voff), "v"(v), "s"(base) : "memory");
}
__device__ __forceinline__ u64 ld_sys64(const char* p) { return __hip_atomic_load((const GAS u64*)p, __ATOMIC_RELAXED, __HIP_MEMORY_SCOPE_SYSTEM); }
__device__ __forceinline__ u32 ld_agt(const char* p) { return __hip_atomic_load((const GAS u32*)p, __ATOMIC_RELAXED, __HIP_MEMORY_SCOPE_AGENT); }
__device__ __forceinline__ void st_sys64(char* p, u64 v) { __hip_atomic_store((GAS u64*)p, v, __ATOMIC_RELAXED, __HIP_MEMORY_SCOPE_SYSTEM); }
__device__ __forceinline__ void st_agt(char* p, u32 v) { __hip_atomic_store((GAS u32*)p, v, __ATOMIC_RELAXED, __HIP_MEMORY_SCOPE_AGENT); }
#define WAITV(n) asm volatile("s_waitcnt vmcnt(" #n ")" ::: "memory")
#define WAITL() asm volatile("s_waitcnt lgkmcnt(0)" ::: "memory")
#define RAWBAR() do { asm volatile("" ::: "memory"); __builtin_amdgcn_s_barrier(); asm volatile("" ::: "memory"); } while (0)
__device__ __forceinline__ u32 rfl(u32 v) { return __builtin_amdgcn_readfirstlane(v); }
__device__ __forceinline__ u32 rl(u32 v, int l) { return __builtin_amdgcn_readlane(v, l); }
#define STAMP(i) do {} while (0)
#define LSTAMP(i) do {} while (0)

__device__ __forceinline__ u32 wave_wait64(const char* flag, u32 call, char* tmo, bool& ok) {
  const u64 t0 = __builtin_amdgcn_s_memrealtime();
  u32 data = 0;
  for (;;) {
    u64 v = (u64)call << 32;
    if (flag) v = ld_sys64(flag);
    const u32 tv = ld_agt(tmo);
    data = (u32)v;
    if (__all((u32)(v >> 32) >= call)) { ok = true; break; }
    if (tv == call || __builtin_amdgcn_s_memrealtime() - t0 > DEADLINE) { st_agt(tmo, call); ok = false; break; }
    __builtin_amdgcn_s_sleep(1);
  }
  asm volatile("" ::: "memory");
  return data;
}

__device__ __forceinline__ bool swait2(const char* p, u32 call, char* tmo, u32& h0, u32& h1) {
  const u64 t0 = __builtin_amdgcn_s_memrealtime();
  for (u32 it = 1;; ++it) {
    u32x4 v;
    asm volatile("s_load_dwordx4 %0, %1, 0x0 glc\n\ts_waitcnt lgkmcnt(0)" : "=&s"(v) : "s"(p) : "memory");
    h0 = v[0]; h1 = v[2];
    if (v[1] >= call && v[3] >= call) return true;
    if ((it & 63) == 0) {
      if (rfl(ld_agt(tmo)) == call || __builtin_amdgcn_s_memrealtime() - t0 > DEADLINE) { st_agt(tmo, call); return false; }
    }
    __builtin_amdgcn_s_sleep(SSLEEP);
  }
}

typedef unsigned u32x16 __attribute__((ext_vector_type(16)));
typedef unsigned u32x2 __attribute__((ext_vector_type(2)));
__device__ __forceinline__ u32 minu(u32 a, u32 b) { return a < b ? a : b; }
#define SPOLL_TAIL \
    if ((it & 63) == 0) { \
      if (rfl(ld_agt(tmo)) == call || __builtin_amdgcn_s_memrealtime() - t0 > DEADLINE) { st_agt(tmo, call); return false; } \
    } \
    __builtin_amdgcn_s_sleep(GSLEEP);
__device__ __forceinline__ bool swait8(const char* p, u32 call, char* tmo, u32 (&h)[8]) {
  const u64 t0 = __builtin_amdgcn_s_memrealtime();
  for (u32 it = 1;; ++it) {
    u32x16 v;
    asm volatile("s_load_dwordx16 %0, %1, 0x0 glc\n\ts_waitcnt lgkmcnt(0)" : "=&s"(v) : "s"(p) : "memory");
#pragma unroll
    for (int i = 0; i < 8; ++i) h[i] = v[2 * i];
    if (minu(minu(minu(v[1], v[3]), minu(v[5], v[7])), minu(minu(v[9], v[11]), minu(v[13], v[15]))) >= call) return true;
    SPOLL_TAIL
  }
}
__device__ __forceinline__ bool swait4(const char* p, u32 call, char* tmo, u32 (&h)[4]) {
  const u64 t0 = __builtin_amdgcn_s_memrealtime();
  for (u32 it = 1;; ++it) {
    u32x2 a, b, c, d;
    asm volatile("s_load_dwordx2 %0, %4, 0x0 glc\n\ts_load_dwordx2 %1, %4, 0x10 glc\n\ts_load_dwordx2 %2, %4, 0x20 glc\n\ts_load_dwordx2 %3, %4, 0x30 glc\n\ts_waitcnt lgkmcnt(0)"
                 : "=&s"(a), "=&s"(b), "=&s"(c), "=&s"(d) : "s"(p) : "memory");
    h[0] = a[0]; h[1] = b[0]; h[2] = c[0]; h[3] = d[0];
    if (minu(minu(a[1], b[1]), minu(c[1], d[1])) >= call) return true;
    SPOLL_TAIL
  }
}
__device__ __forceinline__ bool swait1(const char* p, u32 call, char* tmo, u32& h) {
  const u64 t0 = __builtin_amdgcn_s_memrealtime();
  for (u32 it = 1;; ++it) {
    u32x2 a;
    asm volatile("s_load_dwordx2 %0, %1, 0x0 glc\n\ts_waitcnt lgkmcnt(0)" : "=&s"(a) : "s"(p) : "memory");
    h = a[0];
    if (a[1] >= call) return true;
    SPOLL_TAIL
  }
}

__device__ __forceinline__ u32 perm(u32 hi, u32 lo, u32 sel) { return __builtin_amdgcn_perm(hi, lo, sel); }
__device__ __forceinline__ u32 rotr(u32 x, u32 n) { return __builtin_amdgcn_alignbit(x, x, n); }
__device__ __forceinline__ u32 bfi(u32 m, u32 a, u32 b) { return (a & m) | (b & ~m); }
__device__ __forceinline__ u32 pkmax(u32 a, u32 b) { return __builtin_bit_cast(u32, __builtin_elementwise_max(__builtin_bit_cast(u16x2, a), __builtin_bit_cast(u16x2, b))); }
template <int CTRL, int RM> __device__ __forceinline__ u32 dppmax(u32 m) { u32 t = __builtin_amdgcn_update_dpp(0u, m, CTRL, RM, 0xf, false); return t > m ? t : m; }

constexpr u32 HDR_RAW = 127u | (63u << 13) | (63u << 25);
constexpr u32 KX = 0x80008080u;
__device__ __forceinline__ u32 min3u(u32 a, u32 b, u32 c) { const u32 m = a < b ? a : b; return m < c ? m : c; }

__device__ __forceinline__ u32 pack_unit(const u32 (&O)[16], u32 (&P)[12], int lane) {
  u32 H[8], U[8], L[8];
#pragma unroll
  for (int m = 0; m < 8; ++m) { L[m] = perm(O[2 * m + 1], O[2 * m], 0x06040200u); H[m] = perm(O[2 * m + 1], O[2 * m], 0x07050301u); }
  u32 e = 0, o = 0;
#pragma unroll
  for (int m = 0; m < 8; ++m) { const u32 h7 = H[m] & 0x7F7F7F7Fu; e = pkmax(e, h7); o = pkmax(o, h7 << 8); }
  u32 mx = pkmax(e, o);
  mx = pkmax(mx, mx >> 16) & 0xFF00u;
  mx = dppmax<0x111, 0xf>(mx); mx = dppmax<0x112, 0xf>(mx); mx = dppmax<0x114, 0xf>(mx); mx = dppmax<0x118, 0xf>(mx);
  mx = dppmax<0x142, 0xa>(mx); mx = dppmax<0x143, 0xc>(mx);
  const u32 mc = rl(mx, 63) >> 8;
  const u32 base = mc > 7 ? mc - 7 : 0;
  const u32 baseB = base * 0x01010101u;
  u32 orr = 0;
#pragma unroll
  for (int m = 0; m < 8; ++m) { U[m] = (H[m] | 0x80808080u) - baseB; orr |= U[m]; }
  const bool bad = (orr & 0x78787878u) != 0;
  const u64 mask = __ballot(bad);
  u32 hdr = base | (63u << 13) | (63u << 25);
  if (mask) {
    u32 eo = 0, ea = 0xFFFFFFFFu, z = 0;
#pragma unroll
    for (int m = 0; m < 8; ++m) {
      const u32 bl = ~U[m] & 0x80808080u;
      const u32 mk = bl | (bl - (bl >> 7));
      eo |= U[m] & mk; ea &= U[m] | ~mk;
      const u32 t = U[m] ^ 0x80808080u;
      z |= (t - 0x01010101u) & ~t & 0x80808080u;
      U[m] = bfi(mk, 0x80808080u, U[m]);
    }
    eo |= eo >> 16; eo |= eo >> 8; eo &= 0xFFu;
    ea &= ea >> 16; ea &= ea >> 8; ea &= 0xFFu;
    const bool lane_ok = z == 0 && eo == ea && eo >= 0x41u;
    const u64 fail = __ballot(bad && !lane_ok);
    if (fail || __popcll(mask) > 2) hdr = HDR_RAW;
    else {
      const int l0 = __ffsll((long long)mask) - 1, l1 = 63 - __clzll((long long)mask);
      const u32 d0 = 0x7Fu - rl(eo, l0), d1 = 0x7Fu - rl(eo, l1);
      hdr = base | (l0 << 7) | (d0 << 13) | (l1 << 19) | (d1 << 25);
    }
  }
  u32 C[4];
#pragma unroll
  for (int t = 0; t < 4; ++t) {
    const u32 ya = bfi(0x80808080u, H[2 * t], U[2 * t]), yb = bfi(0x80808080u, H[2 * t + 1], U[2 * t + 1]);
    C[t] = (ya ^ KX) + rotr(yb, 28);
  }
  if (__ballot(min3u(C[0], C[1], C[2]) == 0)) hdr = HDR_RAW;
  P[0] = L[0]; P[1] = L[1]; P[2] = L[2]; P[3] = C[0];
  P[4] = L[3]; P[5] = L[4]; P[6] = L[5]; P[7] = C[1];
  P[8] = L[6]; P[9] = L[7]; P[10] = C[2]; P[11] = C[3];
  return hdr;
}

__device__ __forceinline__ void merge_lo_hi(u32x4 p0, u32x4 p1, u32x4 p2, int t, u32 h0, u32 h1, u32 (&O)[16]) {
  const u32 l0 = t == 0 ? p0[0] : t == 1 ? p0[2] : t == 2 ? p1[1] : p2[0], l1 = t == 0 ? p0[1] : t == 1 ? p1[0] : t == 2 ? p1[2] : p2[1];
  O[4 * t + 0] = perm(h0, l0, 0x05010400u); O[4 * t + 1] = perm(h0, l0, 0x07030602u);
  O[4 * t + 2] = perm(h1, l1, 0x05010400u); O[4 * t + 3] = perm(h1, l1, 0x07030602u);
}
__device__ __forceinline__ void unpack_unit(u32x4 p0, u32x4 p1, u32x4 p2, u32 hdr, int lane, u32 (&O)[16]) {
  const u32 base = hdr & 127u;
  const u32 baseB = base * 0x01010101u;
  const u32 d0 = (hdr >> 13) & 63u;
  if (d0 == 63u) {
#pragma unroll
    for (int t = 0; t < 4; ++t) {
      const u32 n = t == 0 ? p0[3] : t == 1 ? p1[3] : t == 2 ? p2[2] : p2[3];
      merge_lo_hi(p0, p1, p2, t, ((n & 0x87878787u) ^ KX) + baseB, (rotr(n, 4) & 0x87878787u) + baseB, O);
    }
  } else {
    const u32 T0 = baseB + 0x03020100u, T1 = T0 + 0x04040404u;
    const u32 d1 = (hdr >> 25) & 63u;
    const u32 Ta = (T0 & ~0xFFu) | (base - 1u - d0), Tb = (T0 & ~0xFFu) | (base - 1u - d1);
    u32 Tv = lane == (int)((hdr >> 19) & 63u) ? Tb : T0;
    Tv = lane == (int)((hdr >> 7) & 63u) ? Ta : Tv;
#pragma unroll
    for (int t = 0; t < 4; ++t) {
      const u32 n = t == 0 ? p0[3] : t == 1 ? p1[3] : t == 2 ? p2[2] : p2[3], r = rotr(n, 4);
      merge_lo_hi(p0, p1, p2, t, perm(T1, Tv, n & 0x07070707u) | ((n & 0x80808080u) ^ KX), perm(T1, Tv, r & 0x07070707u) | (r & 0x80808080u), O);
    }
  }
}

__device__ __forceinline__ float silu(float y) {
  const float c = fminf(fmaxf(y, -60.f), 60.f);
  return y * __builtin_amdgcn_rcpf(1.f + __expf(-c));
}
__device__ __forceinline__ u32 pack_bf16(float lo, float hi) {
  typedef __bf16 bf2 __attribute__((ext_vector_type(2)));
  bf2 r; r.x = (__bf16)lo; r.y = (__bf16)hi; return __builtin_bit_cast(u32, r);
}
__device__ __forceinline__ u32x4 mk4(const u32* p) { u32x4 v = {p[0], p[1], p[2], p[3]}; return v; }

struct Unit { u32x4 p[3]; };
__device__ __forceinline__ void load_unit(Unit& u, __amdgpu_buffer_rsrc_t rs, u32 voff, u32 soff) {
  u.p[0] = __builtin_amdgcn_raw_buffer_load_b128(rs, voff, soff, 17);
  u.p[1] = __builtin_amdgcn_raw_buffer_load_b128(rs, voff + 1024, soff, 17);
  u.p[2] = __builtin_amdgcn_raw_buffer_load_b128(rs, voff + 2048, soff, 17);
}
__device__ __forceinline__ void ensure_unit(Unit& u, const char* ws, u32 voff, u32 soff, u32 call, char* tmo, u32& nretry, bool& bad) {
  if (__builtin_expect(__ballot(min3u(u.p[0][3], u.p[1][3], u.p[2][2]) == 0) != 0, 0)) {
    const u64 t0 = __builtin_amdgcn_s_memrealtime();
    for (;;) {
      asm volatile("s_nop 4\n\tglobal_load_dwordx4 %0, %3, %4 sc0 sc1\n\tglobal_load_dwordx4 %1, %3, %4 offset:1024 sc0 sc1\n\t"
                   "global_load_dwordx4 %2, %3, %4 offset:2048 sc0 sc1\n\ts_waitcnt vmcnt(0)"
                   : "+v"(u.p[0]), "+v"(u.p[1]), "+v"(u.p[2]) : "v"(voff), "s"(ws + soff) : "memory");
      if (!__ballot(min3u(u.p[0][3], u.p[1][3], u.p[2][2]) == 0)) break;
      if (rfl(ld_agt(tmo)) == call || __builtin_amdgcn_s_memrealtime() - t0 > DEADLINE) { st_agt(tmo, call); bad = true; break; }
    }
  }
}
__device__ __forceinline__ void decode_unit(Unit& u, __amdgpu_buffer_rsrc_t rs, const char* ws, u32 voff, u32 soff, u32 hdr, int lane, u32 call, char* tmo, u32 (&O)[16], u32& nretry, bool& bad) {
  if ((hdr & 127u) == 127u) {
    u32x4 p3;
    asm volatile("s_nop 4\n\tglobal_load_dwordx4 %0, %1, %2 offset:3072 sc0 sc1\n\ts_waitcnt vmcnt(0)" : "=&v"(p3) : "v"(voff), "s"(ws + soff) : "memory");
#pragma unroll
    for (int i = 0; i < 12; ++i) O[i] = u.p[i >> 2][i & 3];
#pragma unroll
    for (int i = 0; i < 4; ++i) O[12 + i] = p3[i];
  } else {
    ensure_unit(u, ws, voff, soff, call, tmo, nretry, bad);
    unpack_unit(u.p[0], u.p[1], u.p[2], hdr, lane, O);
  }
}
__device__ __forceinline__ void unpack_tab(const Unit& u, u32 hdr, int lane, u32 (&O)[16]) {
  const u32 base = hdr & 127u, baseB = base * 0x01010101u;
  const u32 T0 = baseB + 0x03020100u, T1 = T0 + 0x04040404u;
  const u32 d0 = (hdr >> 13) & 63u, d1 = (hdr >> 25) & 63u;
  const int l0 = d0 != 63u ? (int)((hdr >> 7) & 63u) : 255, l1 = d0 != 63u ? (int)((hdr >> 19) & 63u) : 255;
  const u32 Ta = (T0 & ~0xFFu) | ((base - 1u - d0) & 0xFFu), Tb = (T0 & ~0xFFu) | ((base - 1u - d1) & 0xFFu);
  u32 Tv = lane == l1 ? Tb : T0;
  Tv = lane == l0 ? Ta : Tv;
#pragma unroll
  for (int t = 0; t < 4; ++t) {
    const u32 n = t == 0 ? u.p[0][3] : t == 1 ? u.p[1][3] : t == 2 ? u.p[2][2] : u.p[2][3], r = rotr(n, 4);
    merge_lo_hi(u.p[0], u.p[1], u.p[2], t, perm(T1, Tv, n & 0x07070707u) | ((n & 0x80808080u) ^ KX), perm(T1, Tv, r & 0x07070707u) | (r & 0x80808080u), O);
  }
}

__device__ __forceinline__ int second_tile(int grp) { return SWAP02 == 1 ? (grp == 0 ? 6 : grp == 2 ? 4 : grp + 4) : SWAP02 == 2 ? (grp ^ 2) + 4 : grp + 4; }
__device__ __forceinline__ int out_blk(int i) { return OPERM && (i == 1 || i == 2) ? 3 - i : i; }
__device__ __forceinline__ bool round_is_w(int r) { return r == 0 || r == 5; }
__device__ __forceinline__ int round_idx(int r) { return r == 0 ? 0 : r == 5 ? 1 : r - 1; }
__device__ __forceinline__ int tile_m(int o) { return o & 3; }
__device__ __forceinline__ int tile_n(int o) { return o >> 2; }
__global__ __launch_bounds__(NTHR, 2) void fused_gemm_ar(Args a) {
  __shared__ __attribute__((aligned(16))) char smem_[LDS_BYTES];
  LAS char* smem = (LAS char*)smem_;
  const int wg = blockIdx.x ^ (SWAPROLE ? 128 : 0), tid = threadIdx.x, lane = tid & 63;
  const int wave = rfl(tid >> 6);
  const char* xin = a.x; const char* win = a.w; char* outp = a.out; char* myws = a.myws; int rank = a.rank; char* dws = a.ws[wg & 7]; u32 pmap = a.pmap;
  asm volatile("" : "+s"(xin), "+s"(win), "+s"(outp), "+s"(myws), "+s"(rank), "+s"(dws), "+s"(pmap));
  const int pdst = (pmap >> (4 * (wg & 7))) & 7;
  char* tmo = myws + OFF_TMO;
  const u64 tstart = (PCT || ZDELAY) ? __builtin_amdgcn_s_memrealtime() : 0;
  STAMP(0);
  const char* cntp = myws + OFF_CNT + wg * 128;

  if (wg < 128) {
    const int p = pdst, j = wg >> 3;
    const int q = j & 3, blk = j >> 2;
    const bool self = p == rank;
    LAS u32* hdrs = (LAS u32*)(smem + L_HDR);
    if (wave == 0) {
      const u32 callv = ld_agt(cntp);
      if (lane < 32) hdrs[lane] = 0;
      WAITL(); RAWBAR();
      const u32 call = rfl(callv) + 1;
      const int nr = self ? 6 : 14;
      const u32 par = (call & 1) * AB_BYTES;
      if (G0 && PCT == 0) {
        bool ok;
        (void)wave_wait64(lane < 2 ? myws + OFF_PC + wg * 128 + lane * 8 : nullptr, call, tmo, ok);
        LSTAMP(1);
      } else if (G0) {
        while (__builtin_amdgcn_s_memrealtime() - tstart < PCT) __builtin_amdgcn_s_sleep(1);
        LSTAMP(1);
      }
#pragma unroll 1
      for (int r = G0 ? 1 : 0; r < nr; ++r) {
        if (r >= 6 && ((NDM >> (r - 6)) & 1)) continue;
        u32 h0, h1;
        {
          u64 hv;
          for (;;) {
            asm volatile("ds_read_b64 %0, %1\n\ts_waitcnt lgkmcnt(0)" : "=v"(hv) : "v"((u32)(uintptr_t)(hdrs + 2 * r)) : "memory");
            if (rfl((u32)hv) >> 31 & rfl((u32)(hv >> 32)) >> 31) break;
            __builtin_amdgcn_s_sleep(1);
          }
          h0 = rfl((u32)hv) & 0x7FFFFFFFu; h1 = rfl((u32)(hv >> 32)) & 0x7FFFFFFFu;
        }
        if (FIRSTD && !G0 && r == 0) { LSTAMP(1); continue; }
        LSTAMP(2 * r);
        u32 dslot, fslot;
        if (r >= 6) { const u32 u = (rank * 128 + (r - 6) * 16 + j) * 2; dslot = OFF_IN + u * UNIT; fslot = OFF_FO + u * 8; }
        else if (round_is_w(r)) { const u32 u = ((round_idx(r) * 4 + blk) * 8 + rank) * 8 + q * 2; dslot = OFF_B + par + u * UNIT; fslot = OFF_FB + u * 8; }
        else { const u32 u = ((round_idx(r) * 4 + blk) * 8 + rank) * 8 + q * 2; dslot = OFF_A + par + u * UNIT; fslot = OFF_FA + u * 8; }
        const LAS char* src = smem + r * 2 * UNIT + lane * 16;
        char* dst = dws + dslot;
        const u32 vo = lane * 16;
        const u32x4 v0 = *(const LAS u32x4*)(src), v1 = *(const LAS u32x4*)(src + 1024), v2 = *(const LAS u32x4*)(src + 2048);
        const u32x4 v4 = *(const LAS u32x4*)(src + UNIT), v5 = *(const LAS u32x4*)(src + UNIT + 1024), v6 = *(const LAS u32x4*)(src + UNIT + 2048);
        const bool raw0 = (h0 & 127u) == 127u, raw1 = (h1 & 127u) == 127u;
        const bool hfirst = HFIRST && EARLY && (r >= 6 || HFIRST == 2 || (HFIRST == 3 && !round_is_w(r)) || (HFIRST == 4 && r != 0)) && !raw0 && !raw1;
        if (hfirst && lane < 2) st_sys64(dws + fslot + lane * 8, ((u64)call << 32) | (lane == 0 ? h0 : h1));
        if (raw0) st16(dst, vo + 3072, *(const LAS u32x4*)(src + 3072));
        if (raw1) st16(dst, vo + UNIT + 3072, *(const LAS u32x4*)(src + UNIT + 3072));
        st16(dst, vo, v0); st16(dst, vo + 1024, v1); st16(dst, vo + 2048, v2);
        st16(dst, vo + UNIT, v4); st16(dst, vo + UNIT + 1024, v5); st16(dst, vo + UNIT + 2048, v6);
        if (!EARLY || raw0 || raw1) WAITV(0);
        if (!hfirst && lane < 2) st_sys64(dws + fslot + lane * 8, ((u64)call << 32) | (lane == 0 ? h0 : h1));
        if (GATE < 9 && lane == 0) hdrs[28] = r;
        WAITV(3);
        LSTAMP(2 * r + 1);
      }
      if (lane == 0) st_agt(myws + OFF_CNT + wg * 128, call);
      STAMP(6);
      return;
    }
    const int k = wave - 1;
    const int g1 = k + (G0 ? 2 : 0), r1 = g1 >> 1, i1 = g1 & 1;
    const bool w1 = round_is_w(r1);
    const int c = lane & 31, h = lane >> 5;
    const __amdgpu_buffer_rsrc_t rx = __builtin_amdgcn_make_buffer_rsrc((void*)xin, 0, 2048 * 512, 0x00020000);
    const __amdgpu_buffer_rsrc_t rw = __builtin_amdgcn_make_buffer_rsrc((void*)win, 0, 256 * 4096, 0x00020000);
    const u32 xv = c * 512 + h * 32, xs0 = ((p >> 2) * 1024 + blk * 64) * 512 + q * 128;
    const u32 wv = h * 16 * 4096 + c * 4, ws0 = q * 64 * 4096 + ((p & 3) * 512 + blk * 64) * 2;
    u32 D1[16], D2[16];
    auto loadw = [&](u32 (&D)[16], int r, int i) {
      const u32 so = ws0 + i * 32 * 4096 + round_idx(r) * 512;
#pragma unroll
      for (int rr = 0; rr < 16; ++rr) D[rr] = __builtin_amdgcn_raw_buffer_load_b32(rw, wv + (rr & 0) , so + rr * 4096, 0);
    };
    auto loadx = [&](u32 (&D)[16], int r, int i) {
      const u32 so = xs0 + round_idx(r) * 256 * 512 + i * 32 * 512;
#pragma unroll
      for (int cc = 0; cc < 4; ++cc) { const u32x4 t = __builtin_amdgcn_raw_buffer_load_b128(rx, xv + (cc >> 1) * 64 + (cc & 1) * 16, so, 0); D[4 * cc] = t[0]; D[4 * cc + 1] = t[1]; D[4 * cc + 2] = t[2]; D[4 * cc + 3] = t[3]; }
    };
    if (STAG && k >= 2) __builtin_amdgcn_s_sleep(STAG);
    if (w1) loadw(D1, r1, i1); else loadx(D1, r1, i1);
    const int g2 = g1 + 7, r2 = g2 >> 1, i2 = g2 & 1;
    const bool two = g2 < 12;
    const bool w2 = round_is_w(r2);
    auto load2 = [&]() { if (two) { if (w2) loadw(D2, r2, i2); else loadx(D2, r2, i2); } };
    if (!DEFER2) load2();
    const u32 callv = ld_agt(cntp);
    STAMP(4);
    RAWBAR();
    STAMP(5);
#pragma unroll
    for (int job = 0; job < 2; ++job) {
      if (job == 1 && !two) break;
      u32 O[16];
      if (job == 1 ? w2 : w1) {
        const u32* D = job == 1 ? D2 : D1;
#pragma unroll
        for (int cb = 0; cb < 4; ++cb)
#pragma unroll
          for (int e = 0; e < 4; ++e) O[4 * cb + e] = perm(D[8 * (cb & 1) + 2 * e + 1], D[8 * (cb & 1) + 2 * e], (cb >> 1) ? 0x07060302u : 0x05040100u);
      } else {
#pragma unroll
        for (int i = 0; i < 16; ++i) O[i] = job == 1 ? D2[i] : D1[i];
      }
      if (DEFER2 && job == 0) { asm volatile("" :: "v"(O[0]), "v"(O[15]) : "memory"); load2(); __builtin_amdgcn_sched_barrier(0); }
      const int slot = job == 0 ? g1 : g2;
      u32 P[12];
      const u32 hdr = pack_unit(O, P, lane);
      if (FIRSTD && !G0 && job == 0 && k < 2) {
        const u32 u = ((round_idx(0) * 4 + blk) * 8 + rank) * 8 + q * 2 + k;
        const u32 call0 = rfl(callv) + 1;
        char* dst = dws + OFF_B + (call0 & 1) * AB_BYTES + u * UNIT;
        char* fl = dws + OFF_FB + u * 8;
        if ((hdr & 127u) == 127u) {
#pragma unroll
          for (int pl = 0; pl < 4; ++pl) st16(dst, lane * 16 + pl * 1024, mk4(O + 4 * pl));
          WAITV(0);
        } else {
#pragma unroll
          for (int pl = 0; pl < 3; ++pl) st16(dst, lane * 16 + pl * 1024, mk4(P + 4 * pl));
        }
        if (lane == 0) st_sys64(fl, ((u64)call0 << 32) | hdr);
        if (k == 0) LSTAMP(0);
        WAITV(0);
        if (lane == 0) hdrs[k] = 0x80000000u;
        continue;
      }
      LAS char* d = smem + slot * UNIT + lane * 16;
      if ((hdr & 127u) == 127u) {
#pragma unroll
        for (int pl = 0; pl < 4; ++pl) *(LAS u32x4*)(d + pl * 1024) = mk4(O + 4 * pl);
      } else {
#pragma unroll
        for (int pl = 0; pl < 3; ++pl) *(LAS u32x4*)(d + pl * 1024) = mk4(P + 4 * pl);
      }
      WAITL();
      if (lane == 0) hdrs[slot] = hdr | 0x80000000u;
    }
    const u32 call = rfl(callv) + 1;
    STAMP(1);
    if (ZLINK) {
      for (;;) {
        u32 pr;
        asm volatile("ds_read_b32 %0, %1\n\ts_waitcnt lgkmcnt(0)" : "=v"(pr) : "v"((u32)(uintptr_t)(hdrs + 28)) : "memory");
        if ((int)rfl(pr) >= ZG) break;
        __builtin_amdgcn_s_sleep(4);
      }
      const u32x4 z = {0, 0, 0, 0};
      const u32 par1 = AB_BYTES - (call & 1) * AB_BYTES;
#pragma unroll
      for (int i = 0; i < 6; ++i) {
        const u32 s = k + 7 * i, un = (s * 43) >> 7;
        const u32 slot = un < 8 ? OFF_A + ((j * 8 + p) * 8 + un) * UNIT : OFF_B + (((j >> 1) * 8 + p) * 8 + (j & 1) * 4 + un - 8) * UNIT;
        if (!NOZERO && s < 36) st16(myws + par1 + slot + (s - 3 * un) * 1024, lane * 16, z);
      }
      WAITV(0);
    }
    if (!self) {
      for (int o = k; o < 8; o += 7) {
        if ((NDM >> o) & 1) continue;
        const int tile = o * 16 + j;
        bool ok;
        u32 h0, h1;
        if (GATE < 9) {
          for (;;) {
            u32 pr;
            asm volatile("ds_read_b32 %0, %1\n\ts_waitcnt lgkmcnt(0)" : "=v"(pr) : "v"((u32)(uintptr_t)(hdrs + 28)) : "memory");
            if ((int)rfl(pr) >= 6 + o - GATE) break;
            __builtin_amdgcn_s_sleep(4);
          }
        }
        if (SPOLL & 1) ok = swait2(myws + OFF_FL + tile * 128, call, tmo, h0, h1);
        else { const u32 hv = wave_wait64(lane < 2 ? myws + OFF_FL + tile * 128 + lane * 8 : nullptr, call, tmo, ok); h0 = rl(hv, 0); h1 = rl(hv, 1); }
        const char* src = myws + OFF_OB + (IBO ? (call & 1) * OB_BYTES : 0) + tile * 2 * UNIT + lane * 16;
        LAS char* d = smem + (12 + 2 * o) * UNIT;
        if (ok) {
          const bool raw0 = (h0 & 127u) == 127u, raw1 = (h1 & 127u) == 127u;
          const u64 t0 = __builtin_amdgcn_s_memrealtime();
          if (IBO && IBOD) __builtin_amdgcn_s_sleep(IBOD);
          for (;;) {
#pragma unroll
            for (int pl = 0; pl < 3; ++pl) {
              __builtin_amdgcn_global_load_lds((const GAS void*)(src + pl * 1024), (LAS void*)(d + pl * 1024), 16, 0, 17);
              __builtin_amdgcn_global_load_lds((const GAS void*)(src + UNIT + pl * 1024), (LAS void*)(d + UNIT + pl * 1024), 16, 0, 17);
            }
            if (raw0) __builtin_amdgcn_global_load_lds((const GAS void*)(src + 3072), (LAS void*)(d + 3072), 16, 0, 17);
            if (raw1) __builtin_amdgcn_global_load_lds((const GAS void*)(src + UNIT + 3072), (LAS void*)(d + UNIT + 3072), 16, 0, 17);
            WAITV(0);
            if (!IBO) break;
            const LAS char* cw = d + lane * 16;
            const u32 m0 = min3u(*(const LAS u32*)(cw + 12), *(const LAS u32*)(cw + 1024 + 12), *(const LAS u32*)(cw + 2048 + 8));
            const u32 m1 = min3u(*(const LAS u32*)(cw + UNIT + 12), *(const LAS u32*)(cw + UNIT + 1024 + 12), *(const LAS u32*)(cw + UNIT + 2048 + 8));
            if (!__ballot((!raw0 && m0 == 0) || (!raw1 && m1 == 0))) break;
            if (rfl(ld_agt(tmo)) == call || __builtin_amdgcn_s_memrealtime() - t0 > DEADLINE) { st_agt(tmo, call); ok = false; break; }
          }
        }
        if (!ok) {
          h0 = HDR_RAW; h1 = HDR_RAW;
          const u32x4 nan4 = {0x7FC07FC0u, 0x7FC07FC0u, 0x7FC07FC0u, 0x7FC07FC0u};
#pragma unroll
          for (int pl = 0; pl < 8; ++pl) *(LAS u32x4*)(d + pl * 1024 + lane * 16) = nan4;
          WAITL();
        }
        if (lane < 2) hdrs[12 + 2 * o + lane] = (lane == 0 ? h0 : h1) | 0x80000000u;
        STAMP(2);
      }
    }
    return;
  }

  const int gw = wg - 128;
  const int x = gw & 7, slot = gw >> 3, grp = slot >> 2, m = slot & 3;
  const int rh = m & 1, ul = (x >> 1) * 4 + (x & 1) * 2 + (m >> 1);
  const __amdgpu_buffer_rsrc_t rs = __builtin_amdgcn_make_buffer_rsrc((void*)myws, 0, (int)WS_NEED, 0x00020000);
  const u32 voff = lane * 16;
  u32 call;
  if (G0 && wave < 2) {
    const int q = slot & 3, blk = slot >> 2;
    const int c = lane & 31, h = lane >> 5;
    const __amdgpu_buffer_rsrc_t rw = __builtin_amdgcn_make_buffer_rsrc((void*)win, 0, 256 * 4096, 0x00020000);
    const u32 wv = h * 16 * 4096 + c * 4, so = (q * 64 + wave * 32) * 4096 + ((pdst & 3) * 512 + blk * 64) * 2;
    u32 D[16];
#pragma unroll
    for (int rr = 0; rr < 16; ++rr) D[rr] = __builtin_amdgcn_raw_buffer_load_b32(rw, wv, so + rr * 4096, 0);
    const u32 callv = ld_agt(cntp);
    u32 O[16], P[12];
#pragma unroll
    for (int cb = 0; cb < 4; ++cb)
#pragma unroll
      for (int e = 0; e < 4; ++e) O[4 * cb + e] = perm(D[8 * (cb & 1) + 2 * e + 1], D[8 * (cb & 1) + 2 * e], (cb >> 1) ? 0x07060302u : 0x05040100u);
    const u32 hdr = pack_unit(O, P, lane);
    call = rfl(callv) + 1;
    const u32 u = (blk * 8 + rank) * 8 + q * 2 + wave;
    char* dst = dws + OFF_B + (call & 1) * AB_BYTES + u * UNIT;
    if ((hdr & 127u) == 127u) {
#pragma unroll
      for (int pl = 0; pl < 4; ++pl) st16(dst, voff + pl * 1024, mk4(O + 4 * pl));
      WAITV(0);
    } else {
#pragma unroll
      for (int pl = 0; pl < 3; ++pl) st16(dst, voff + pl * 1024, mk4(P + 4 * pl));
    }
    if (lane == 0) st_sys64(dws + OFF_FB + u * 8, ((u64)call << 32) | hdr);
    if (PCT == 0 && lane == 0) st_sys64(myws + OFF_PC + gw * 128 + wave * 8, (u64)call << 32);
  } else {
    call = rfl(ld_agt(cntp)) + 1;
    if (ZDELAY) { while (__builtin_amdgcn_s_memrealtime() - tstart < ZDELAY) __builtin_amdgcn_s_sleep(2); }
    const u32x4 z = {0, 0, 0, 0};
    char* zb = myws + OFF_A + (AB_BYTES - (call & 1) * AB_BYTES) + gw * 12 * UNIT;
#pragma unroll
    for (int i = 0; i < 6; ++i) {
      const u32 pl = G0 ? wave - 2 + 6 * i : wave + 8 * i, un = (pl * 43) >> 7;
      if (!ZLINK && !NOZERO && pl < 36) st16(zb + un * UNIT + (pl - 3 * un) * 1024, voff, z);
    }
    if (IBO && wave >= 2) {
      const int jb = (wave - 2) >= 3, pl = wave - 2 - 3 * jb;
      const int ht2 = ((grp + 4 * jb) * 16 + ul) * 2 + rh;
      if (!NOZERO) st16(myws + OFF_OB + (OB_BYTES - (call & 1) * OB_BYTES) + ht2 * UNIT + pl * 1024, voff, z);
    }
  }
  const u32 par = (call & 1) * AB_BYTES;
  const u32 ovo = (lane >> 3) * (N_ * 2) + (lane & 7) * 16;
  u32 OA[4][16];
  u32 nrB = 0, nrA = 0, nrO = 0;
  bool bad = false;
#pragma unroll 1
  for (int job = 0; job < 2; ++job) {
    const int o = grp + 4 * job;
    const int tm = 4 * tile_m(o) + (ul >> 2), tn = 4 * tile_n(o) + (ul & 3);
    const int ht = (o * 16 + ul) * 2 + rh;
    f32x16 acc[2] = {};
    {
      const u32 sa = OFF_A + par + ((tm * 8 + wave) * 8 + rh) * UNIT, sb = OFF_B + par + (tn * 8 + wave) * 8 * UNIT;
      bool ok;
      u32 hb[8];
      if (SPOLL & 2) ok = swait8(myws + OFF_FB + (tn * 8 + wave) * 64, call, tmo, hb);
      else {
        const u32 hv = wave_wait64(lane < 8 ? myws + OFF_FB + ((tn * 8 + wave) * 8 + lane) * 8 : nullptr, call, tmo, ok);
#pragma unroll
        for (int u = 0; u < 8; ++u) hb[u] = rl(hv, u);
      }
      bad |= !ok;
      STAMP(job == 0 ? 12 : 5);
      Unit ub[8];
#pragma unroll
      for (int u = 0; u < 8; ++u) { load_unit(ub[u], rs, voff, sb + u * UNIT); __builtin_amdgcn_sched_barrier(0); }
      if (job == 0) {
        u32 OB[8][16];
        Unit ua[4];
#pragma unroll
        for (int u = 0; u < 8; ++u) {
          if (u == BLA) {
#pragma unroll
            for (int q = 0; q < 4; ++q) { load_unit(ua[q], rs, voff, sa + 2 * q * UNIT); __builtin_amdgcn_sched_barrier(0); }
          }
          decode_unit(ub[u], rs, myws, voff, sb + u * UNIT, hb[u], lane, call, tmo, OB[u], nrB, bad);
        }
        u32 ha[4];
        if (SPOLL & 4) ok = swait4(myws + OFF_FA + (tm * 8 + wave) * 64 + rh * 8, call, tmo, ha);
        else {
          const u32 hv = wave_wait64(lane < 4 ? myws + OFF_FA + ((tm * 8 + wave) * 8 + lane * 2 + rh) * 8 : nullptr, call, tmo, ok);
#pragma unroll
          for (int q = 0; q < 4; ++q) ha[q] = rl(hv, q);
        }
        bad |= !ok;
        STAMP(1);
        bool again = BLA >= 8;
        if (BLA < 8) {
          u32 mn = 0xFFFFFFFFu;
#pragma unroll
          for (int q = 0; q < 4; ++q) { mn = minu(mn, min3u(ua[q].p[0][3], ua[q].p[1][3], ua[q].p[2][2])); again |= (ha[q] & 127u) == 127u; }
          again |= __ballot(mn == 0) != 0;
        }
        if (again) {
#pragma unroll
          for (int q = 0; q < 4; ++q) { load_unit(ua[q], rs, voff, sa + 2 * q * UNIT); __builtin_amdgcn_sched_barrier(0); }
        }
#define MFMA_Q(q) \
  _Pragma("unroll") for (int i = 0; i < 2; ++i) _Pragma("unroll") for (int b = 0; b < 2; ++b) _Pragma("unroll") for (int ch = 0; ch < 2; ++ch) \
    acc[ch] = __builtin_amdgcn_mfma_f32_32x32x16_bf16(__builtin_bit_cast(bf16x8, mk4(OB[2 * (q) + i] + 4 * (2 * ch + b))), __builtin_bit_cast(bf16x8, mk4(OA[q] + 4 * (2 * i + b))), acc[ch], 0, 0, 0);
        auto fix_raw = [&](int q) {
          if (__builtin_expect((ha[q] & 127u) == 127u, 0)) {
            u32x4 p3;
            asm volatile("s_nop 4\n\tglobal_load_dwordx4 %0, %1, %2 offset:3072 sc0 sc1\n\ts_waitcnt vmcnt(0)" : "=&v"(p3) : "v"(voff), "s"(myws + sa + 2 * q * UNIT) : "memory");
#pragma unroll
            for (int i = 0; i < 12; ++i) OA[q][i] = ua[q].p[i >> 2][i & 3];
#pragma unroll
            for (int i = 0; i < 4; ++i) OA[q][12 + i] = p3[i];
          }
        };
        if (PIPA) {
          if ((ha[0] & 127u) != 127u) ensure_unit(ua[0], myws, voff, sa, call, tmo, nrA, bad);
          unpack_tab(ua[0], ha[0], lane, OA[0]);
          fix_raw(0);
#pragma unroll
          for (int q = 0; q < 4; ++q) {
            if (q < 3 && (ha[q + 1] & 127u) != 127u) ensure_unit(ua[q + 1], myws, voff, sa + 2 * (q + 1) * UNIT, call, tmo, nrA, bad);
            __builtin_amdgcn_sched_barrier(0);
            if (q < 3) unpack_tab(ua[q + 1], ha[q + 1], lane, OA[q + 1]);
            MFMA_Q(q)
            if (q < 3) {
              asm volatile("" : "+v"(OA[q + 1][0]), "+v"(OA[q + 1][1]), "+v"(OA[q + 1][2]), "+v"(OA[q + 1][3]), "+v"(OA[q + 1][4]), "+v"(OA[q + 1][5]), "+v"(OA[q + 1][6]), "+v"(OA[q + 1][7]),
                                "+v"(OA[q + 1][8]), "+v"(OA[q + 1][9]), "+v"(OA[q + 1][10]), "+v"(OA[q + 1][11]), "+v"(OA[q + 1][12]), "+v"(OA[q + 1][13]), "+v"(OA[q + 1][14]), "+v"(OA[q + 1][15]));
#pragma unroll
              for (int g = 0; g < 8; ++g) { __builtin_amdgcn_sched_group_barrier(0x8, 1, 0); __builtin_amdgcn_sched_group_barrier(0x2, PIPA, 0); }
            }
            __builtin_amdgcn_sched_barrier(0);
            if (q < 3) fix_raw(q + 1);
          }
        } else {
#pragma unroll
          for (int q = 0; q < 4; ++q) {
            decode_unit(ua[q], rs, myws, voff, sa + 2 * q * UNIT, ha[q], lane, call, tmo, OA[q], nrA, bad);
            MFMA_Q(q)
          }
        }
      } else {
#pragma unroll
        for (int u = 0; u < 8; ++u) {
          u32 OB[16];
          decode_unit(ub[u], rs, myws, voff, sb + u * UNIT, hb[u], lane, call, tmo, OB, nrB, bad);
#pragma unroll
          for (int b = 0; b < 2; ++b)
#pragma unroll
            for (int ch = 0; ch < 2; ++ch)
              acc[ch] = __builtin_amdgcn_mfma_f32_32x32x16_bf16(__builtin_bit_cast(bf16x8, mk4(OB + 4 * (2 * ch + b))), __builtin_bit_cast(bf16x8, mk4(OA[u >> 1] + 4 * (2 * (u & 1) + b))), acc[ch], 0, 0, 0);
        }
      }
    }
    STAMP(2 + 4 * job);
    WAITV(0);
    {
      LAS char* pw = smem + wave * 8192 + lane * 16;
#pragma unroll
      for (int ch = 0; ch < 2; ++ch)
#pragma unroll
        for (int g = 0; g < 4; ++g) {
          const f32x4 v = {acc[ch][4 * g], acc[ch][4 * g + 1], acc[ch][4 * g + 2], acc[ch][4 * g + 3]};
          *(LAS f32x4*)(pw + (ch * 4 + g) * 1024) = v;
        }
    }
    WAITL(); RAWBAR();
    {
      typedef float f32x2 __attribute__((ext_vector_type(2)));
      const int g = wave & 3, hf = wave >> 2;
      const LAS char* pr = smem + g * 1024 + lane * 16 + hf * 8;
      f32x2 r0[8], r1[8];
#pragma unroll
      for (int s = 0; s < 8; ++s) { r0[s] = *(const LAS f32x2*)(pr + s * 8192); r1[s] = *(const LAS f32x2*)(pr + s * 8192 + 4096); }
      __builtin_amdgcn_sched_barrier(0);
      const f32x2 a0 = ((r0[0] + r0[1]) + (r0[2] + r0[3])) + ((r0[4] + r0[5]) + (r0[6] + r0[7]));
      const f32x2 a1 = ((r1[0] + r1[1]) + (r1[2] + r1[3])) + ((r1[4] + r1[5]) + (r1[6] + r1[7]));
      u32x2 y;
      y[0] = pack_bf16(silu(a0[0]), silu(a1[0])); y[1] = pack_bf16(silu(a0[1]), silu(a1[1]));
      *(LAS u32x2*)(smem + L_IMG + (lane & 31) * 144 + (2 * g + (lane >> 5)) * 16 + hf * 8) = y;
    }
#if !(TMOX & 1)
    const u32 tv = ld_agt(tmo);
#endif
    WAITL(); RAWBAR();
    STAMP(3 + 4 * job);
    if (wave == 0) {
      u32 O[16];
      const bool badt = rfl(tv) == call;
#pragma unroll
      for (int cc = 0; cc < 4; ++cc) {
        const u32x4 t = *(const LAS u32x4*)(smem + L_IMG + (cc * 8 + (lane >> 3)) * 144 + (lane & 7) * 16);
#pragma unroll
        for (int e = 0; e < 4; ++e) O[4 * cc + e] = badt ? 0x7FC07FC0u : t[e];
      }
      u32 P[12];
      const u32 hdr = pack_unit(O, P, lane);
      const bool raw = (hdr & 127u) == 127u;
      if ((NDM >> o) & 1) {
        const u32 islot = OFF_IN + (rank * 256 + ht) * UNIT;
        char* pw = a.ws[0];
#pragma unroll
        for (int d = 1; d < 8; ++d) pw = lane == d ? a.ws[d] : pw;
        char* fo = pw + OFF_FO + (rank * 256 + ht) * 8;
        if (!raw && lane < 8 && lane != rank) st_sys64(fo, ((u64)call << 32) | hdr);
#pragma unroll
        for (int d = 0; d < 7; ++d) {
          int dd7 = d + (x + 2 * m) % 7; dd7 = dd7 >= 7 ? dd7 - 7 : dd7;
          char* dst = a.ws[(rank + 1 + dd7) & 7] + islot;
          if (raw) {
#pragma unroll
            for (int pl = 0; pl < 4; ++pl) st16(dst, voff + pl * 1024, mk4(O + 4 * pl));
          } else {
#pragma unroll
            for (int pl = 0; pl < 3; ++pl) st16(dst, voff + pl * 1024, mk4(P + 4 * pl));
          }
        }
        if (raw) { WAITV(0); if (lane < 8 && lane != rank) st_sys64(fo, ((u64)call << 32) | hdr); }
      } else {
        char* d = myws + OFF_OB + (IBO ? (call & 1) * OB_BYTES : 0) + ht * UNIT;
        if (raw) {
#pragma unroll
          for (int pl = 0; pl < 4; ++pl) st16(d, voff + pl * 1024, mk4(O + 4 * pl));
        } else {
#pragma unroll
          for (int pl = 0; pl < 3; ++pl) st16(d, voff + pl * 1024, mk4(P + 4 * pl));
        }
      }
      if (IBO && !raw && !((NDM >> o) & 1) && lane == 0) st_sys64(myws + OFF_FL + (ht >> 1) * 128 + rh * 8, ((u64)call << 32) | hdr);
      char* dd = outp + ((size_t)((rank >> 2) * 1024 + tm * 64 + rh * 32) * N_ + (rank & 3) * 512 + tn * 64) * 2;
#pragma unroll
      for (int cc = 0; cc < 4; ++cc) st16(dd + cc * 8 * N_ * 2, ovo, mk4(O + 4 * cc));
      if (BARC && job == 0) RAWBAR();
      if (!((NDM >> o) & 1) && (!IBO || raw)) {
        WAITV(4);
        if (lane == 0) st_sys64(myws + OFF_FL + (ht >> 1) * 128 + rh * 8, ((u64)call << 32) | hdr);
      }
      STAMP(4 + 4 * job);
    } else if (BARC && job == 0) RAWBAR();
  }
  if (wave != rank) {
#pragma unroll 1
    for (int job = 0; job < 2; ++job) {
      const int o = grp + 4 * job;
      const int tm = 4 * tile_m(o) + (ul >> 2), tn = 4 * tile_n(o) + (ul & 3);
      const int ht = (o * 16 + ul) * 2 + rh;
      bool ok2;
      u32 h0;
      if (SPOLL & 8) ok2 = swait1(myws + OFF_FO + (wave * 256 + ht) * 8, call, tmo, h0);
      else h0 = rl(wave_wait64(lane == 0 ? myws + OFF_FO + (wave * 256 + ht) * 8 : nullptr, call, tmo, ok2), 0);
      STAMP(9 + job);
      u32 O[16];
      const u32 si = OFF_IN + (wave * 256 + ht) * UNIT;
      bool bad2 = !ok2;
      {
        Unit u0;
        load_unit(u0, rs, voff, si);
        decode_unit(u0, rs, myws, voff, si, h0, lane, call, tmo, O, nrO, bad2);
      }
#if !(TMOX & 2)
      bad2 |= rfl(ld_agt(tmo)) == call;
#endif
      if (__builtin_expect(bad2, 0)) {
#pragma unroll
        for (int i = 0; i < 16; ++i) O[i] = 0x7FC07FC0u;
      }
      char* d = outp + ((size_t)((wave >> 2) * 1024 + tm * 64 + rh * 32) * N_ + (wave & 3) * 512 + tn * 64) * 2;
#pragma unroll
      for (int cc = 0; cc < 4; ++cc) st16(d + cc * 8 * N_ * 2, ovo, mk4(O + 4 * cc));
      const u32x4 z = {0, 0, 0, 0};
#pragma unroll
      for (int pl = 0; pl < 3; ++pl) st16(myws + si + pl * 1024, voff, z);
    }
  }
  if (tid == 0) st_agt(myws + OFF_CNT + wg * 128, call);
  STAMP(11);
}

extern "C" void kernel_launch(int rank, int nranks, ncclComm_t comm, hipStream_t stream, void* const* d_in, const int* in_sizes, int n_in,
                              void* d_out, int out_size, const int* in_dtypes, int out_dtype, void* const* d_out_all, void* const* d_ws_all, size_t ws_bytes) {
  if (nranks != NR || ws_bytes < WS_NEED || (unsigned)rank >= (unsigned)NR) { fprintf(stderr, "bad config\n"); return; }
  Args a;
  a.pmap = 0;
  for (int i = 0; i < NR; ++i) { const int p = link_dest(rank, i); a.ws[i] = (char*)d_ws_all[p]; a.pmap |= (u32)p << (4 * i); }
  a.out = (char*)d_out; a.x = (const char*)d_in[0]; a.w = (const char*)d_in[1]; a.myws = (char*)d_ws_all[rank]; a.rank = rank;
  fused_gemm_ar<<<NWG, NTHR, 0, stream>>>(a);
}
```

```cpp
#include <hip/hip_runtime.h>
#include <rccl/rccl.h>
#include <cstdio>

#define DIAG 0
#define WIN 13
#define FORCE_RAW 0
#define ZDELAY 0
#define EARLY 1
#define UNR 0
#define ORD 3
#define HFIRST 1
#define DEFER2 0
#define STAG 0
#define FIRSTD 1
#define ND 0
#define NDM 0
#define TMOX 2
#define IBO 0
#define SPOLL 15
#define SSLEEP 4
#define GATE 3
#define GSLEEP 4
#define PCT 0
#define LMAP 1
#define LMAPSHOW 0
#define BLA 8
#define PIPA 0
#define IBOD 0
#define ZLINK 1
#define ZG 3
#define SWAP02 0
#define OPERM 0
#define G0 1
#define SWAPROLE 1
#define PACEW 9
#define NOZERO 0
#define RTMO 8
#define RSLEEP 0
#define PATH1 0
#define BARC 1

typedef unsigned u32;
typedef unsigned long long u64;
typedef unsigned u32x4 __attribute__((ext_vector_type(4)));
typedef unsigned short u16x2 __attribute__((ext_vector_type(2)));
typedef short bf16x8 __attribute__((ext_vector_type(8)));
typedef float f32x16 __attribute__((ext_vector_type(16)));
typedef float f32x4 __attribute__((ext_vector_type(4)));
#define LAS __attribute__((address_space(3)))
#define GAS __attribute__((address_space(1)))

constexpr int NR = 8;
constexpr int N_ = 2048;
constexpr int NWG = 256, NTHR = 512;
constexpr u64 DEADLINE = 20000000ull;
constexpr u32 UNIT = 4096;

constexpr u32 OFF_CNT = 0;
constexpr u32 OFF_TMO = 64 * 1024;
constexpr u32 OFF_FA = 80 * 1024;
constexpr u32 OFF_FB = OFF_FA + 16 * 8 * 8 * 8;
constexpr u32 OFF_FO = 96 * 1024;
constexpr u32 OFF_FL = 128 * 1024;
constexpr u32 OFF_PC = 160 * 1024;
constexpr u32 OFF_DIAG = 256 * 1024;
constexpr u32 OFF_A = 1u << 20;
constexpr u32 OFF_B = OFF_A + 16 * 8 * 8 * UNIT;
constexpr u32 AB_BYTES = (16 + 8) * 8 * 8 * UNIT;
constexpr u32 OFF_IN = OFF_A + 2 * AB_BYTES;
constexpr u32 OFF_OB = OFF_IN + 8 * 128 * 2 * UNIT;
constexpr u32 OB_BYTES = 128 * 2 * UNIT;
constexpr u32 WS_NEED = OFF_OB + 2 * OB_BYTES;

constexpr u32 L_IMG = 64 * 1024;
constexpr u32 L_HDR = 28 * UNIT;
constexpr u32 LDS_BYTES = L_HDR + 128;

struct Args { const char* x; const char* w; char* out; char* myws; int rank; u32 pmap; char* ws[NR]; };
static inline int link_dest(int rank, int i) {
  return i ^ rank;
}

__device__ __forceinline__ void st16(char* base, u32 voff, u32x4 v) {
  asm volatile("s_nop 4\n\tglobal_store_dwordx4 %0, %1, %2 sc0 sc1\n\ts_nop 1" ::"v"(voff), "v"(v), "s"(base) : "memory");
}
__device__ __forceinline__ u64 ld_sys64(const char* p) { return __hip_atomic_load((const GAS u64*)p, __ATOMIC_RELAXED, __HIP_MEMORY_SCOPE_SYSTEM); }
__device__ __forceinline__ u32 ld_agt(const char* p) { return __hip_atomic_load((const GAS u32*)p, __ATOMIC_RELAXED, __HIP_MEMORY_SCOPE_AGENT); }
__device__ __forceinline__ void st_sys64(char* p, u64 v) { __hip_atomic_store((GAS u64*)p, v, __ATOMIC_RELAXED, __HIP_MEMORY_SCOPE_SYSTEM); }
__device__ __forceinline__ void st_agt(char* p, u32 v) { __hip_atomic_store((GAS u32*)p, v, __ATOMIC_RELAXED, __HIP_MEMORY_SCOPE_AGENT); }
#define WAITV(n) asm volatile("s_waitcnt vmcnt(" #n ")" ::: "memory")
#define WAITL() asm volatile("s_waitcnt lgkmcnt(0)" ::: "memory")
#define RAWBAR() do { asm volatile("" ::: "memory"); __builtin_amdgcn_s_barrier(); asm volatile("" ::: "memory"); } while (0)
__device__ __forceinline__ u32 rfl(u32 v) { return __builtin_amdgcn_readfirstlane(v); }
__device__ __forceinline__ u32 rl(u32 v, int l) { return __builtin_amdgcn_readlane(v, l); }
#define STAMP(i) do {} while (0)
#define LSTAMP(i) do {} while (0)

__device__ __forceinline__ u32 wave_wait64(const char* flag, u32 call, char* tmo, bool& ok) {
  const u64 t0 = __builtin_amdgcn_s_memrealtime();
  u32 data = 0;
  for (;;) {
    u64 v = (u64)call << 32;
    if (flag) v = ld_sys64(flag);
    const u32 tv = ld_agt(tmo);
    data = (u32)v;
    if (__all((u32)(v >> 32) >= call)) { ok = true; break; }
    if (tv == call || __builtin_amdgcn_s_memrealtime() - t0 > DEADLINE) { st_agt(tmo, call); ok = false; break; }
    __builtin_amdgcn_s_sleep(1);
  }
  asm volatile("" ::: "memory");
  return data;
}

__device__ __forceinline__ bool swait2(const char* p, u32 call, char* tmo, u32& h0, u32& h1) {
  const u64 t0 = __builtin_amdgcn_s_memrealtime();
  for (u32 it = 1;; ++it) {
    u32x4 v;
    asm volatile("s_load_dwordx4 %0, %1, 0x0 glc\n\ts_waitcnt lgkmcnt(0)" : "=&s"(v) : "s"(p) : "memory");
    h0 = v[0]; h1 = v[2];
    if (v[1] >= call && v[3] >= call) return true;
    if ((it & 63) == 0) {
      if (rfl(ld_agt(tmo)) == call || __builtin_amdgcn_s_memrealtime() - t0 > DEADLINE) { st_agt(tmo, call); return false; }
    }
    __builtin_amdgcn_s_sleep(SSLEEP);
  }
}

typedef unsigned u32x16 __attribute__((ext_vector_type(16)));
typedef unsigned u32x2 __attribute__((ext_vector_type(2)));
__device__ __forceinline__ u32 minu(u32 a, u32 b) { return a < b ? a : b; }
#define SPOLL_TAIL \
    if ((it & 63) == 0) { \
      if (rfl(ld_agt(tmo)) == call || __builtin_amdgcn_s_memrealtime() - t0 > DEADLINE) { st_agt(tmo, call); return false; } \
    } \
    __builtin_amdgcn_s_sleep(GSLEEP);
__device__ __forceinline__ bool swait8(const char* p, u32 call, char* tmo, u32 (&h)[8]) {
  const u64 t0 = __builtin_amdgcn_s_memrealtime();
  for (u32 it = 1;; ++it) {
    u32x16 v;
    asm volatile("s_load_dwordx16 %0, %1, 0x0 glc\n\ts_waitcnt lgkmcnt(0)" : "=&s"(v) : "s"(p) : "memory");
#pragma unroll
    for (int i = 0; i < 8; ++i) h[i] = v[2 * i];
    if (minu(minu(minu(v[1], v[3]), minu(v[5], v[7])), minu(minu(v[9], v[11]), minu(v[13], v[15]))) >= call) return true;
    SPOLL_TAIL
  }
}
__device__ __forceinline__ bool swait4(const char* p, u32 call, char* tmo, u32 (&h)[4]) {
  const u64 t0 = __builtin_amdgcn_s_memrealtime();
  for (u32 it = 1;; ++it) {
    u32x2 a, b, c, d;
    asm volatile("s_load_dwordx2 %0, %4, 0x0 glc\n\ts_load_dwordx2 %1, %4, 0x10 glc\n\ts_load_dwordx2 %2, %4, 0x20 glc\n\ts_load_dwordx2 %3, %4, 0x30 glc\n\ts_waitcnt lgkmcnt(0)"
                 : "=&s"(a), "=&s"(b), "=&s"(c), "=&s"(d) : "s"(p) : "memory");
    h[0] = a[0]; h[1] = b[0]; h[2] = c[0]; h[3] = d[0];
    if (minu(minu(a[1], b[1]), minu(c[1], d[1])) >= call) return true;
    SPOLL_TAIL
  }
}
__device__ __forceinline__ bool swait1(const char* p, u32 call, char* tmo, u32& h) {
  const u64 t0 = __builtin_amdgcn_s_memrealtime();
  for (u32 it = 1;; ++it) {
    u32x2 a;
    asm volatile("s_load_dwordx2 %0, %1, 0x0 glc\n\ts_waitcnt lgkmcnt(0)" : "=&s"(a) : "s"(p) : "memory");
    h = a[0];
    if (a[1] >= call) return true;
    SPOLL_TAIL
  }
}

__device__ __forceinline__ u32 perm(u32 hi, u32 lo, u32 sel) { return __builtin_amdgcn_perm(hi, lo, sel); }
__device__ __forceinline__ u32 rotr(u32 x, u32 n) { return __builtin_amdgcn_alignbit(x, x, n); }
__device__ __forceinline__ u32 bfi(u32 m, u32 a, u32 b) { return (a & m) | (b & ~m); }
__device__ __forceinline__ u32 pkmax(u32 a, u32 b) { return __builtin_bit_cast(u32, __builtin_elementwise_max(__builtin_bit_cast(u16x2, a), __builtin_bit_cast(u16x2, b))); }
template <int CTRL, int RM> __device__ __forceinline__ u32 dppmax(u32 m) { u32 t = __builtin_amdgcn_update_dpp(0u, m, CTRL, RM, 0xf, false); return t > m ? t : m; }

constexpr u32 HDR_RAW = 127u | (63u << 13) | (63u << 25);
constexpr u32 KX = 0x80008080u;
__device__ __forceinline__ u32 min3u(u32 a, u32 b, u32 c) { const u32 m = a < b ? a : b; return m < c ? m : c; }

__device__ __forceinline__ u32 pack_unit(const u32 (&O)[16], u32 (&P)[12], int lane) {
  u32 H[8], U[8], L[8];
#pragma unroll
  for (int m = 0; m < 8; ++m) { L[m] = perm(O[2 * m + 1], O[2 * m], 0x06040200u); H[m] = perm(O[2 * m + 1], O[2 * m], 0x07050301u); }
  u32 e = 0, o = 0;
#pragma unroll
  for (int m = 0; m < 8; ++m) { const u32 h7 = H[m] & 0x7F7F7F7Fu; e = pkmax(e, h7); o = pkmax(o, h7 << 8); }
  u32 mx = pkmax(e, o);
  mx = pkmax(mx, mx >> 16) & 0xFF00u;
  mx = dppmax<0x111, 0xf>(mx); mx = dppmax<0x112, 0xf>(mx); mx = dppmax<0x114, 0xf>(mx); mx = dppmax<0x118, 0xf>(mx);
  mx = dppmax<0x142, 0xa>(mx); mx = dppmax<0x143, 0xc>(mx);
  const u32 mc = rl(mx, 63) >> 8;
  const u32 base = mc > 7 ? mc - 7 : 0;
  const u32 baseB = base * 0x01010101u;
  u32 orr = 0;
#pragma unroll
  for (int m = 0; m < 8; ++m) { U[m] = (H[m] | 0x80808080u) - baseB; orr |= U[m]; }
  const bool bad = (orr & 0x78787878u) != 0;
  const u64 mask = __ballot(bad);
  u32 hdr = base | (63u << 13) | (63u << 25);
  if (mask) {
    u32 eo = 0, ea = 0xFFFFFFFFu, z = 0;
#pragma unroll
    for (int m = 0; m < 8; ++m) {
      const u32 bl = ~U[m] & 0x80808080u;
      const u32 mk = bl | (bl - (bl >> 7));
      eo |= U[m] & mk; ea &= U[m] | ~mk;
      const u32 t = U[m] ^ 0x80808080u;
      z |= (t - 0x01010101u) & ~t & 0x80808080u;
      U[m] = bfi(mk, 0x80808080u, U[m]);
    }
    eo |= eo >> 16; eo |= eo >> 8; eo &= 0xFFu;
    ea &= ea >> 16; ea &= ea >> 8; ea &= 0xFFu;
    const bool lane_ok = z == 0 && eo == ea && eo >= 0x41u;
    const u64 fail = __ballot(bad && !lane_ok);
    if (fail || __popcll(mask) > 2) hdr = HDR_RAW;
    else {
      const int l0 = __ffsll((long long)mask) - 1, l1 = 63 - __clzll((long long)mask);
      const u32 d0 = 0x7Fu - rl(eo, l0), d1 = 0x7Fu - rl(eo, l1);
      hdr = base | (l0 << 7) | (d0 << 13) | (l1 << 19) | (d1 << 25);
    }
  }
  u32 C[4];
#pragma unroll
  for (int t = 0; t < 4; ++t) {
    const u32 ya = bfi(0x80808080u, H[2 * t], U[2 * t]), yb = bfi(0x80808080u, H[2 * t + 1], U[2 * t + 1]);
    C[t] = (ya ^ KX) + rotr(yb, 28);
  }
  if (__ballot(min3u(C[0], C[1], C[2]) == 0)) hdr = HDR_RAW;
  P[0] = L[0]; P[1] = L[1]; P[2] = L[2]; P[3] = C[0];
  P[4] = L[3]; P[5] = L[4]; P[6] = L[5]; P[7] = C[1];
  P[8] = L[6]; P[9] = L[7]; P[10] = C[2]; P[11] = C[3];
  return hdr;
}

__device__ __forceinline__ void merge_lo_hi(u32x4 p0, u32x4 p1, u32x4 p2, int t, u32 h0, u32 h1, u32 (&O)[16]) {
  const u32 l0 = t == 0 ? p0[0] : t == 1 ? p0[2] : t == 2 ? p1[1] : p2[0], l1 = t == 0 ? p0[1] : t == 1 ? p1[0] : t == 2 ? p1[2] : p2[1];
  O[4 * t + 0] = perm(h0, l0, 0x05010400u); O[4 * t + 1] = perm(h0, l0, 0x07030602u);
  O[4 * t + 2] = perm(h1, l1, 0x05010400u); O[4 * t + 3] = perm(h1, l1, 0x07030602u);
}
__device__ __forceinline__ void unpack_unit(u32x4 p0, u32x4 p1, u32x4 p2, u32 hdr, int lane, u32 (&O)[16]) {
  const u32 base = hdr & 127u;
  const u32 baseB = base * 0x01010101u;
  const u32 d0 = (hdr >> 13) & 63u;
  if (d0 == 63u) {
#pragma unroll
    for (int t = 0; t < 4; ++t) {
      const u32 n = t == 0 ? p0[3] : t == 1 ? p1[3] : t == 2 ? p2[2] : p2[3];
      merge_lo_hi(p0, p1, p2, t, ((n & 0x87878787u) ^ KX) + baseB, (rotr(n, 4) & 0x87878787u) + baseB, O);
    }
  } else {
    const u32 T0 = baseB + 0x03020100u, T1 = T0 + 0x04040404u;
    const u32 d1 = (hdr >> 25) & 63u;
    const u32 Ta = (T0 & ~0xFFu) | (base - 1u - d0), Tb = (T0 & ~0xFFu) | (base - 1u - d1);
    u32 Tv = lane == (int)((hdr >> 19) & 63u) ? Tb : T0;
    Tv = lane == (int)((hdr >> 7) & 63u) ? Ta : Tv;
#pragma unroll
    for (int t = 0; t < 4; ++t) {
      const u32 n = t == 0 ? p0[3] : t == 1 ? p1[3] : t == 2 ? p2[2] : p2[3], r = rotr(n, 4);
      merge_lo_hi(p0, p1, p2, t, perm(T1, Tv, n & 0x07070707u) | ((n & 0x80808080u) ^ KX), perm(T1, Tv, r & 0x07070707u) | (r & 0x80808080u), O);
    }
  }
}

__device__ __forceinline__ float silu(float y) {
  const float c = fminf(fmaxf(y, -60.f), 60.f);
  return y * __builtin_amdgcn_rcpf(1.f + __expf(-c));
}
__device__ __forceinline__ u32 pack_bf16(float lo, float hi) {
  typedef __bf16 bf2 __attribute__((ext_vector_type(2)));
  bf2 r; r.x = (__bf16)lo; r.y = (__bf16)hi; return __builtin_bit_cast(u32, r);
}
__device__ __forceinline__ u32x4 mk4(const u32* p) { u32x4 v = {p[0], p[1], p[2], p[3]}; return v; }

struct Unit { u32x4 p[3]; };
__device__ __forceinline__ void load_unit(Unit& u, __amdgpu_buffer_rsrc_t rs, u32 voff, u32 soff) {
  u.p[0] = __builtin_amdgcn_raw_buffer_load_b128(rs, voff, soff, 17);
  u.p[1] = __builtin_amdgcn_raw_buffer_load_b128(rs, voff + 1024, soff, 17);
  u.p[2] = __builtin_amdgcn_raw_buffer_load_b128(rs, voff + 2048, soff, 17);
}
__device__ __forceinline__ void ensure_unit(Unit& u, const char* ws, u32 voff, u32 soff, u32 call, char* tmo, u32& nretry, bool& bad) {
  if (__builtin_expect(__ballot(min3u(u.p[0][3], u.p[1][3], u.p[2][2]) == 0) != 0, 0)) {
    const u64 t0 = __builtin_amdgcn_s_memrealtime();
    for (u32 it = 1;; ++it) {
      asm volatile("s_nop 4\n\tglobal_load_dwordx4 %0, %3, %4 sc0 sc1\n\tglobal_load_dwordx4 %1, %3, %4 offset:1024 sc0 sc1\n\t"
                   "global_load_dwordx4 %2, %3, %4 offset:2048 sc0 sc1\n\ts_waitcnt vmcnt(0)"
                   : "+v"(u.p[0]), "+v"(u.p[1]), "+v"(u.p[2]) : "v"(voff), "s"(ws + soff) : "memory");
      if (!__ballot(min3u(u.p[0][3], u.p[1][3], u.p[2][2]) == 0)) break;
      if ((it & (RTMO - 1)) == 0) {
        if (rfl(ld_agt(tmo)) == call || __builtin_amdgcn_s_memrealtime() - t0 > DEADLINE) { st_agt(tmo, call); bad = true; break; }
      }
      if (RSLEEP) __builtin_amdgcn_s_sleep(RSLEEP);
    }
  }
}
__device__ __forceinline__ void decode_unit(Unit& u, __amdgpu_buffer_rsrc_t rs, const char* ws, u32 voff, u32 soff, u32 hdr, int lane, u32 call, char* tmo, u32 (&O)[16], u32& nretry, bool& bad) {
  if ((hdr & 127u) == 127u) {
    u32x4 p3;
    asm volatile("s_nop 4\n\tglobal_load_dwordx4 %0, %1, %2 offset:3072 sc0 sc1\n\ts_waitcnt vmcnt(0)" : "=&v"(p3) : "v"(voff), "s"(ws + soff) : "memory");
#pragma unroll
    for (int i = 0; i < 12; ++i) O[i] = u.p[i >> 2][i & 3];
#pragma unroll
    for (int i = 0; i < 4; ++i) O[12 + i] = p3[i];
  } else {
    ensure_unit(u, ws, voff, soff, call, tmo, nretry, bad);
    unpack_unit(u.p[0], u.p[1], u.p[2], hdr, lane, O);
  }
}
__device__ __forceinline__ void unpack_tab(const Unit& u, u32 hdr, int lane, u32 (&O)[16]) {
  const u32 base = hdr & 127u, baseB = base * 0x01010101u;
  const u32 T0 = baseB + 0x03020100u, T1 = T0 + 0x04040404u;
  const u32 d0 = (hdr >> 13) & 63u, d1 = (hdr >> 25) & 63u;
  const int l0 = d0 != 63u ? (int)((hdr >> 7) & 63u) : 255, l1 = d0 != 63u ? (int)((hdr >> 19) & 63u) : 255;
  const u32 Ta = (T0 & ~0xFFu) | ((base - 1u - d0) & 0xFFu), Tb = (T0 & ~0xFFu) | ((base - 1u - d1) & 0xFFu);
  u32 Tv = lane == l1 ? Tb : T0;
  Tv = lane == l0 ? Ta : Tv;
#pragma unroll
  for (int t = 0; t < 4; ++t) {
    const u32 n = t == 0 ? u.p[0][3] : t == 1 ? u.p[1][3] : t == 2 ? u.p[2][2] : u.p[2][3], r = rotr(n, 4);
    merge_lo_hi(u.p[0], u.p[1], u.p[2], t, perm(T1, Tv, n & 0x07070707u) | ((n & 0x80808080u) ^ KX), perm(T1, Tv, r & 0x07070707u) | (r & 0x80808080u), O);
  }
}

__device__ __forceinline__ int second_tile(int grp) { return SWAP02 == 1 ? (grp == 0 ? 6 : grp == 2 ? 4 : grp + 4) : SWAP02 == 2 ? (grp ^ 2) + 4 : grp + 4; }
__device__ __forceinline__ int out_blk(int i) { return OPERM && (i == 1 || i == 2) ? 3 - i : i; }
__device__ __forceinline__ bool round_is_w(int r) { return r == 0 || r == 5; }
__device__ __forceinline__ int round_idx(int r) { return r == 0 ? 0 : r == 5 ? 1 : r - 1; }
__device__ __forceinline__ int tile_m(int o) { return o & 3; }
__device__ __forceinline__ int tile_n(int o) { return o >> 2; }
__global__ __launch_bounds__(NTHR, 2) void fused_gemm_ar(Args a) {
  __shared__ __attribute__((aligned(16))) char smem_[LDS_BYTES];
  LAS char* smem = (LAS char*)smem_;
  const int wg = blockIdx.x ^ (SWAPROLE ? 128 : 0), tid = threadIdx.x, lane = tid & 63;
  const int wave = rfl(tid >> 6);
  const char* xin = a.x; const char* win = a.w; char* outp = a.out; char* myws = a.myws; int rank = a.rank; char* dws = a.ws[wg & 7]; u32 pmap = a.pmap;
  asm volatile("" : "+s"(xin), "+s"(win), "+s"(outp), "+s"(myws), "+s"(rank), "+s"(dws), "+s"(pmap));
  const int pdst = (pmap >> (4 * (wg & 7))) & 7;
  char* tmo = myws + OFF_TMO;
  const u64 tstart = (PCT || ZDELAY) ? __builtin_amdgcn_s_memrealtime() : 0;
  STAMP(0);
  const char* cntp = myws + OFF_CNT + wg * 128;

  if (wg < 128) {
    const int p = pdst, j = wg >> 3;
    const int q = j & 3, blk = j >> 2;
    const bool self = p == rank;
    LAS u32* hdrs = (LAS u32*)(smem + L_HDR);
    if (wave == 0) {
      const u32 callv = ld_agt(cntp);
      if (lane < 32) hdrs[lane] = 0;
      WAITL(); RAWBAR();
      const u32 call = rfl(callv) + 1;
      const int nr = self ? 6 : 14;
      const u32 par = (call & 1) * AB_BYTES;
      if (G0 && PCT == 0) {
        bool ok;
        (void)wave_wait64(lane < 2 ? myws + OFF_PC + wg * 128 + lane * 8 : nullptr, call, tmo, ok);
        LSTAMP(1);
      } else if (G0) {
        while (__builtin_amdgcn_s_memrealtime() - tstart < PCT) __builtin_amdgcn_s_sleep(1);
        LSTAMP(1);
      }
#pragma unroll 1
      for (int r = G0 ? 1 : 0; r < nr; ++r) {
        if (r >= 6 && ((NDM >> (r - 6)) & 1)) continue;
        u32 h0, h1;
        {
          u64 hv;
          for (;;) {
            asm volatile("ds_read_b64 %0, %1\n\ts_waitcnt lgkmcnt(0)" : "=v"(hv) : "v"((u32)(uintptr_t)(hdrs + 2 * r)) : "memory");
            if (rfl((u32)hv) >> 31 & rfl((u32)(hv >> 32)) >> 31) break;
            __builtin_amdgcn_s_sleep(1);
          }
          h0 = rfl((u32)hv) & 0x7FFFFFFFu; h1 = rfl((u32)(hv >> 32)) & 0x7FFFFFFFu;
        }
        if (FIRSTD && !G0 && r == 0) { LSTAMP(1); continue; }
        LSTAMP(2 * r);
        u32 dslot, fslot;
        if (r >= 6) { const u32 u = (rank * 128 + (r - 6) * 16 + j) * 2; dslot = OFF_IN + u * UNIT; fslot = OFF_FO + u * 8; }
        else if (round_is_w(r)) { const u32 u = ((round_idx(r) * 4 + blk) * 8 + rank) * 8 + q * 2; dslot = OFF_B + par + u * UNIT; fslot = OFF_FB + u * 8; }
        else { const u32 u = ((round_idx(r) * 4 + blk) * 8 + rank) * 8 + q * 2; dslot = OFF_A + par + u * UNIT; fslot = OFF_FA + u * 8; }
        const LAS char* src = smem + r * 2 * UNIT + lane * 16;
        char* dst = dws + dslot;
        const u32 vo = lane * 16;
        const u32x4 v0 = *(const LAS u32x4*)(src), v1 = *(const LAS u32x4*)(src + 1024), v2 = *(const LAS u32x4*)(src + 2048);
        const u32x4 v4 = *(const LAS u32x4*)(src + UNIT), v5 = *(const LAS u32x4*)(src + UNIT + 1024), v6 = *(const LAS u32x4*)(src + UNIT + 2048);
        const bool raw0 = (h0 & 127u) == 127u, raw1 = (h1 & 127u) == 127u;
        const bool hfirst = HFIRST && EARLY && (r >= 6 || HFIRST == 2 || (HFIRST == 3 && !round_is_w(r)) || (HFIRST == 4 && r != 0)) && !raw0 && !raw1;
        if (hfirst && lane < 2) st_sys64(dws + fslot + lane * 8, ((u64)call << 32) | (lane == 0 ? h0 : h1));
        if (raw0) st16(dst, vo + 3072, *(const LAS u32x4*)(src + 3072));
        if (raw1) st16(dst, vo + UNIT + 3072, *(const LAS u32x4*)(src + UNIT + 3072));
        st16(dst, vo, v0); st16(dst, vo + 1024, v1); st16(dst, vo + 2048, v2);
        st16(dst, vo + UNIT, v4); st16(dst, vo + UNIT + 1024, v5); st16(dst, vo + UNIT + 2048, v6);
        if (!EARLY || raw0 || raw1) WAITV(0);
        if (!hfirst && lane < 2) st_sys64(dws + fslot + lane * 8, ((u64)call << 32) | (lane == 0 ? h0 : h1));
        if (GATE < 9 && lane == 0) hdrs[28] = r;
        WAITV(3);
        LSTAMP(2 * r + 1);
      }
      if (lane == 0) st_agt(myws + OFF_CNT + wg * 128, call);
      STAMP(6);
      return;
    }
    const int k = wave - 1;
    const int g1 = k + (G0 ? 2 : 0), r1 = g1 >> 1, i1 = g1 & 1;
    const bool w1 = round_is_w(r1);
    const int c = lane & 31, h = lane >> 5;
    const __amdgpu_buffer_rsrc_t rx = __builtin_amdgcn_make_buffer_rsrc((void*)xin, 0, 2048 * 512, 0x00020000);
    const __amdgpu_buffer_rsrc_t rw = __builtin_amdgcn_make_buffer_rsrc((void*)win, 0, 256 * 4096, 0x00020000);
    const u32 xv = c * 512 + h * 32, xs0 = ((p >> 2) * 1024 + blk * 64) * 512 + q * 128;
    const u32 wv = h * 16 * 4096 + c * 4, ws0 = q * 64 * 4096 + ((p & 3) * 512 + blk * 64) * 2;
    u32 D1[16], D2[16];
    auto loadw = [&](u32 (&D)[16], int r, int i) {
      const u32 so = ws0 + i * 32 * 4096 + round_idx(r) * 512;
#pragma unroll
      for (int rr = 0; rr < 16; ++rr) D[rr] = __builtin_amdgcn_raw_buffer_load_b32(rw, wv + (rr & 0) , so + rr * 4096, 0);
    };
    auto loadx = [&](u32 (&D)[16], int r, int i) {
      const u32 so = xs0 + round_idx(r) * 256 * 512 + i * 32 * 512;
#pragma unroll
      for (int cc = 0; cc < 4; ++cc) { const u32x4 t = __builtin_amdgcn_raw_buffer_load_b128(rx, xv + (cc >> 1) * 64 + (cc & 1) * 16, so, 0); D[4 * cc] = t[0]; D[4 * cc + 1] = t[1]; D[4 * cc + 2] = t[2]; D[4 * cc + 3] = t[3]; }
    };
    if (STAG && (G0 || k >= 2)) __builtin_amdgcn_s_sleep(STAG);
    if (w1) loadw(D1, r1, i1); else loadx(D1, r1, i1);
    const int g2 = g1 + 7, r2 = g2 >> 1, i2 = g2 & 1;
    const bool two = g2 < 12;
    const bool w2 = round_is_w(r2);
    auto load2 = [&]() { if (two) { if (w2) loadw(D2, r2, i2); else loadx(D2, r2, i2); } };
    if (!DEFER2) load2();
    const u32 callv = ld_agt(cntp);
    STAMP(4);
    RAWBAR();
    STAMP(5);
#pragma unroll
    for (int job = 0; job < 2; ++job) {
      if (job == 1 && !two) break;
      u32 O[16];
      if (job == 1 ? w2 : w1) {
        const u32* D = job == 1 ? D2 : D1;
#pragma unroll
        for (int cb = 0; cb < 4; ++cb)
#pragma unroll
          for (int e = 0; e < 4; ++e) O[4 * cb + e] = perm(D[8 * (cb & 1) + 2 * e + 1], D[8 * (cb & 1) + 2 * e], (cb >> 1) ? 0x07060302u : 0x05040100u);
      } else {
#pragma unroll
        for (int i = 0; i < 16; ++i) O[i] = job == 1 ? D2[i] : D1[i];
      }
      if (DEFER2 && job == 0) { asm volatile("" :: "v"(O[0]), "v"(O[15]) : "memory"); load2(); __builtin_amdgcn_sched_barrier(0); }
      const int slot = job == 0 ? g1 : g2;
      u32 P[12];
      const u32 hdr = pack_unit(O, P, lane);
      if (FIRSTD && !G0 && job == 0 && k < 2) {
        const u32 u = ((round_idx(0) * 4 + blk) * 8 + rank) * 8 + q * 2 + k;
        const u32 call0 = rfl(callv) + 1;
        char* dst = dws + OFF_B + (call0 & 1) * AB_BYTES + u * UNIT;
        char* fl = dws + OFF_FB + u * 8;
        if ((hdr & 127u) == 127u) {
#pragma unroll
          for (int pl = 0; pl < 4; ++pl) st16(dst, lane * 16 + pl * 1024, mk4(O + 4 * pl));
          WAITV(0);
        } else {
#pragma unroll
          for (int pl = 0; pl < 3; ++pl) st16(dst, lane * 16 + pl * 1024, mk4(P + 4 * pl));
        }
        if (lane == 0) st_sys64(fl, ((u64)call0 << 32) | hdr);
        if (k == 0) LSTAMP(0);
        WAITV(0);
        if (lane == 0) hdrs[k] = 0x80000000u;
        continue;
      }
      LAS char* d = smem + slot * UNIT + lane * 16;
      if ((hdr & 127u) == 127u) {
#pragma unroll
        for (int pl = 0; pl < 4; ++pl) *(LAS u32x4*)(d + pl * 1024) = mk4(O + 4 * pl);
      } else {
#pragma unroll
        for (int pl = 0; pl < 3; ++pl) *(LAS u32x4*)(d + pl * 1024) = mk4(P + 4 * pl);
      }
      WAITL();
      if (lane == 0) hdrs[slot] = hdr | 0x80000000u;
    }
    const u32 call = rfl(callv) + 1;
    STAMP(1);
    if (ZLINK) {
      for (;;) {
        u32 pr;
        asm volatile("ds_read_b32 %0, %1\n\ts_waitcnt lgkmcnt(0)" : "=v"(pr) : "v"((u32)(uintptr_t)(hdrs + 28)) : "memory");
        if ((int)rfl(pr) >= ZG) break;
        __builtin_amdgcn_s_sleep(4);
      }
      const u32x4 z = {0, 0, 0, 0};
      const u32 par1 = AB_BYTES - (call & 1) * AB_BYTES;
#pragma unroll
      for (int i = 0; i < 6; ++i) {
        const u32 s = k + 7 * i, un = (s * 43) >> 7;
        const u32 slot = un < 8 ? OFF_A + ((j * 8 + p) * 8 + un) * UNIT : OFF_B + (((j >> 1) * 8 + p) * 8 + (j & 1) * 4 + un - 8) * UNIT;
        if (!NOZERO && s < 36) st16(myws + par1 + slot + (s - 3 * un) * 1024, lane * 16, z);
      }
      WAITV(0);
    }
    if (!self) {
      for (int o = k; o < 8; o += 7) {
        if ((NDM >> o) & 1) continue;
        const int tile = o * 16 + j;
        bool ok;
        u32 h0, h1;
        if (GATE < 9) {
          for (;;) {
            u32 pr;
            asm volatile("ds_read_b32 %0, %1\n\ts_waitcnt lgkmcnt(0)" : "=v"(pr) : "v"((u32)(uintptr_t)(hdrs + 28)) : "memory");
            if ((int)rfl(pr) >= 6 + o - GATE) break;
            __builtin_amdgcn_s_sleep(4);
          }
        }
        if (SPOLL & 1) ok = swait2(myws + OFF_FL + tile * 128, call, tmo, h0, h1);
        else { const u32 hv = wave_wait64(lane < 2 ? myws + OFF_FL + tile * 128 + lane * 8 : nullptr, call, tmo, ok); h0 = rl(hv, 0); h1 = rl(hv, 1); }
        const char* src = myws + OFF_OB + (IBO ? (call & 1) * OB_BYTES : 0) + tile * 2 * UNIT + lane * 16;
        LAS char* d = smem + (12 + 2 * o) * UNIT;
        if (ok) {
          const bool raw0 = (h0 & 127u) == 127u, raw1 = (h1 & 127u) == 127u;
          const u64 t0 = __builtin_amdgcn_s_memrealtime();
          if (IBO && IBOD) __builtin_amdgcn_s_sleep(IBOD);
          for (;;) {
#pragma unroll
            for (int pl = 0; pl < 3; ++pl) {
              __builtin_amdgcn_global_load_lds((const GAS void*)(src + pl * 1024), (LAS void*)(d + pl * 1024), 16, 0, 17);
              __builtin_amdgcn_global_load_lds((const GAS void*)(src + UNIT + pl * 1024), (LAS void*)(d + UNIT + pl * 1024), 16, 0, 17);
            }
            if (raw0) __builtin_amdgcn_global_load_lds((const GAS void*)(src + 3072), (LAS void*)(d + 3072), 16, 0, 17);
            if (raw1) __builtin_amdgcn_global_load_lds((const GAS void*)(src + UNIT + 3072), (LAS void*)(d + UNIT + 3072), 16, 0, 17);
            WAITV(0);
            if (!IBO) break;
            const LAS char* cw = d + lane * 16;
            const u32 m0 = min3u(*(const LAS u32*)(cw + 12), *(const LAS u32*)(cw + 1024 + 12), *(const LAS u32*)(cw + 2048 + 8));
            const u32 m1 = min3u(*(const LAS u32*)(cw + UNIT + 12), *(const LAS u32*)(cw + UNIT + 1024 + 12), *(const LAS u32*)(cw + UNIT + 2048 + 8));
            if (!__ballot((!raw0 && m0 == 0) || (!raw1 && m1 == 0))) break;
            if (rfl(ld_agt(tmo)) == call || __builtin_amdgcn_s_memrealtime() - t0 > DEADLINE) { st_agt(tmo, call); ok = false; break; }
          }
        }
        if (!ok) {
          h0 = HDR_RAW; h1 = HDR_RAW;
          const u32x4 nan4 = {0x7FC07FC0u, 0x7FC07FC0u, 0x7FC07FC0u, 0x7FC07FC0u};
#pragma unroll
          for (int pl = 0; pl < 8; ++pl) *(LAS u32x4*)(d + pl * 1024 + lane * 16) = nan4;
          WAITL();
        }
        if (lane < 2) hdrs[12 + 2 * o + lane] = (lane == 0 ? h0 : h1) | 0x80000000u;
        STAMP(2);
      }
    }
    return;
  }

  const int gw = wg - 128;
  const int x = gw & 7, slot = gw >> 3, grp = slot >> 2, m = slot & 3;
  const int rh = m & 1, ul = (x >> 1) * 4 + (x & 1) * 2 + (m >> 1);
  const __amdgpu_buffer_rsrc_t rs = __builtin_amdgcn_make_buffer_rsrc((void*)myws, 0, (int)WS_NEED, 0x00020000);
  const u32 voff = lane * 16;
  u32 call;
  if (G0 && wave < 2) {
    const int q = slot & 3, blk = slot >> 2;
    const int c = lane & 31, h = lane >> 5;
    const __amdgpu_buffer_rsrc_t rw = __builtin_amdgcn_make_buffer_rsrc((void*)win, 0, 256 * 4096, 0x00020000);
    const u32 wv = h * 16 * 4096 + c * 4, so = (q * 64 + wave * 32) * 4096 + ((pdst & 3) * 512 + blk * 64) * 2;
    u32 D[16];
#pragma unroll
    for (int rr = 0; rr < 16; ++rr) D[rr] = __builtin_amdgcn_raw_buffer_load_b32(rw, wv, so + rr * 4096, 0);
    const u32 callv = ld_agt(cntp);
    u32 O[16], P[12];
#pragma unroll
    for (int cb = 0; cb < 4; ++cb)
#pragma unroll
      for (int e = 0; e < 4; ++e) O[4 * cb + e] = perm(D[8 * (cb & 1) + 2 * e + 1], D[8 * (cb & 1) + 2 * e], (cb >> 1) ? 0x07060302u : 0x05040100u);
    const u32 hdr = pack_unit(O, P, lane);
    call = rfl(callv) + 1;
    const u32 u = (blk * 8 + rank) * 8 + q * 2 + wave;
    char* dst = dws + OFF_B + (call & 1) * AB_BYTES + u * UNIT;
    if ((hdr & 127u) == 127u) {
#pragma unroll
      for (int pl = 0; pl < 4; ++pl) st16(dst, voff + pl * 1024, mk4(O + 4 * pl));
      WAITV(0);
    } else {
#pragma unroll
      for (int pl = 0; pl < 3; ++pl) st16(dst, voff + pl * 1024, mk4(P + 4 * pl));
    }
    if (lane == 0) st_sys64(dws + OFF_FB + u * 8, ((u64)call << 32) | hdr);
    if (PCT == 0 && lane == 0) st_sys64(myws + OFF_PC + gw * 128 + wave * 8, (u64)call << 32);
  } else {
    call = rfl(ld_agt(cntp)) + 1;
    if (ZDELAY) { while (__builtin_amdgcn_s_memrealtime() - tstart < ZDELAY) __builtin_amdgcn_s_sleep(2); }
    const u32x4 z = {0, 0, 0, 0};
    char* zb = myws + OFF_A + (AB_BYTES - (call & 1) * AB_BYTES) + gw * 12 * UNIT;
#pragma unroll
    for (int i = 0; i < 6; ++i) {
      const u32 pl = G0 ? wave - 2 + 6 * i : wave + 8 * i, un = (pl * 43) >> 7;
      if (!ZLINK && !NOZERO && pl < 36) st16(zb + un * UNIT + (pl - 3 * un) * 1024, voff, z);
    }
    if (IBO && wave >= 2) {
      const int jb = (wave - 2) >= 3, pl = wave - 2 - 3 * jb;
      const int ht2 = ((grp + 4 * jb) * 16 + ul) * 2 + rh;
      if (!NOZERO) st16(myws + OFF_OB + (OB_BYTES - (call & 1) * OB_BYTES) + ht2 * UNIT + pl * 1024, voff, z);
    }
  }
  const u32 par = (call & 1) * AB_BYTES;
  const u32 ovo = (lane >> 3) * (N_ * 2) + (lane & 7) * 16;
  u32 OA[4][16];
  u32 nrB = 0, nrA = 0, nrO = 0;
  bool bad = false;
#pragma unroll 1
  for (int job = 0; job < 2; ++job) {
    const int o = grp + 4 * job;
    const int tm = 4 * tile_m(o) + (ul >> 2), tn = 4 * tile_n(o) + (ul & 3);
    const int ht = (o * 16 + ul) * 2 + rh;
    f32x16 acc[2] = {};
    {
      const u32 sa = OFF_A + par + ((tm * 8 + wave) * 8 + rh) * UNIT, sb = OFF_B + par + (tn * 8 + wave) * 8 * UNIT;
      bool ok;
      u32 hb[8];
      if (SPOLL & 2) ok = swait8(myws + OFF_FB + (tn * 8 + wave) * 64, call, tmo, hb);
      else {
        const u32 hv = wave_wait64(lane < 8 ? myws + OFF_FB + ((tn * 8 + wave) * 8 + lane) * 8 : nullptr, call, tmo, ok);
#pragma unroll
        for (int u = 0; u < 8; ++u) hb[u] = rl(hv, u);
      }
      bad |= !ok;
      STAMP(job == 0 ? 12 : 5);
      Unit ub[8];
#pragma unroll
      for (int u = 0; u < 8; ++u) { load_unit(ub[u], rs, voff, sb + u * UNIT); __builtin_amdgcn_sched_barrier(0); }
      if (job == 0) {
        u32 OB[8][16];
        Unit ua[4];
#pragma unroll
        for (int u = 0; u < 8; ++u) {
          if (u == BLA) {
#pragma unroll
            for (int q = 0; q < 4; ++q) { load_unit(ua[q], rs, voff, sa + 2 * q * UNIT); __builtin_amdgcn_sched_barrier(0); }
          }
          decode_unit(ub[u], rs, myws, voff, sb + u * UNIT, hb[u], lane, call, tmo, OB[u], nrB, bad);
        }
        u32 ha[4];
        if (SPOLL & 4) ok = swait4(myws + OFF_FA + (tm * 8 + wave) * 64 + rh * 8, call, tmo, ha);
        else {
          const u32 hv = wave_wait64(lane < 4 ? myws + OFF_FA + ((tm * 8 + wave) * 8 + lane * 2 + rh) * 8 : nullptr, call, tmo, ok);
#pragma unroll
          for (int q = 0; q < 4; ++q) ha[q] = rl(hv, q);
        }
        bad |= !ok;
        STAMP(1);
        bool again = BLA >= 8;
        if (BLA < 8) {
          u32 mn = 0xFFFFFFFFu;
#pragma unroll
          for (int q = 0; q < 4; ++q) { mn = minu(mn, min3u(ua[q].p[0][3], ua[q].p[1][3], ua[q].p[2][2])); again |= (ha[q] & 127u) == 127u; }
          again |= __ballot(mn == 0) != 0;
        }
        if (again) {
#pragma unroll
          for (int q = 0; q < 4; ++q) { load_unit(ua[q], rs, voff, sa + 2 * q * UNIT); __builtin_amdgcn_sched_barrier(0); }
        }
#define MFMA_Q(q) \
  _Pragma("unroll") for (int i = 0; i < 2; ++i) _Pragma("unroll") for (int b = 0; b < 2; ++b) _Pragma("unroll") for (int ch = 0; ch < 2; ++ch) \
    acc[ch] = __builtin_amdgcn_mfma_f32_32x32x16_bf16(__builtin_bit_cast(bf16x8, mk4(OB[2 * (q) + i] + 4 * (2 * ch + b))), __builtin_bit_cast(bf16x8, mk4(OA[q] + 4 * (2 * i + b))), acc[ch], 0, 0, 0);
        auto fix_raw = [&](int q) {
          if (__builtin_expect((ha[q] & 127u) == 127u, 0)) {
            u32x4 p3;
            asm volatile("s_nop 4\n\tglobal_load_dwordx4 %0, %1, %2 offset:3072 sc0 sc1\n\ts_waitcnt vmcnt(0)" : "=&v"(p3) : "v"(voff), "s"(myws + sa + 2 * q * UNIT) : "memory");
#pragma unroll
            for (int i = 0; i < 12; ++i) OA[q][i] = ua[q].p[i >> 2][i & 3];
#pragma unroll
            for (int i = 0; i < 4; ++i) OA[q][12 + i] = p3[i];
          }
        };
        if (PIPA) {
          if ((ha[0] & 127u) != 127u) ensure_unit(ua[0], myws, voff, sa, call, tmo, nrA, bad);
          unpack_tab(ua[0], ha[0], lane, OA[0]);
          fix_raw(0);
#pragma unroll
          for (int q = 0; q < 4; ++q) {
            if (q < 3 && (ha[q + 1] & 127u) != 127u) ensure_unit(ua[q + 1], myws, voff, sa + 2 * (q + 1) * UNIT, call, tmo, nrA, bad);
            __builtin_amdgcn_sched_barrier(0);
            if (q < 3) unpack_tab(ua[q + 1], ha[q + 1], lane, OA[q + 1]);
            MFMA_Q(q)
            if (q < 3) {
              asm volatile("" : "+v"(OA[q + 1][0]), "+v"(OA[q + 1][1]), "+v"(OA[q + 1][2]), "+v"(OA[q + 1][3]), "+v"(OA[q + 1][4]), "+v"(OA[q + 1][5]), "+v"(OA[q + 1][6]), "+v"(OA[q + 1][7]),
                                "+v"(OA[q + 1][8]), "+v"(OA[q + 1][9]), "+v"(OA[q + 1][10]), "+v"(OA[q + 1][11]), "+v"(OA[q + 1][12]), "+v"(OA[q + 1][13]), "+v"(OA[q + 1][14]), "+v"(OA[q + 1][15]));
#pragma unroll
              for (int g = 0; g < 8; ++g) { __builtin_amdgcn_sched_group_barrier(0x8, 1, 0); __builtin_amdgcn_sched_group_barrier(0x2, PIPA, 0); }
            }
            __builtin_amdgcn_sched_barrier(0);
            if (q < 3) fix_raw(q + 1);
          }
        } else {
#pragma unroll
          for (int q = 0; q < 4; ++q) {
            decode_unit(ua[q], rs, myws, voff, sa + 2 * q * UNIT, ha[q], lane, call, tmo, OA[q], nrA, bad);
            MFMA_Q(q)
          }
        }
      } else {
#pragma unroll
        for (int u = 0; u < 8; ++u) {
          u32 OB[16];
          decode_unit(ub[u], rs, myws, voff, sb + u * UNIT, hb[u], lane, call, tmo, OB, nrB, bad);
#pragma unroll
          for (int b = 0; b < 2; ++b)
#pragma unroll
            for (int ch = 0; ch < 2; ++ch)
              acc[ch] = __builtin_amdgcn_mfma_f32_32x32x16_bf16(__builtin_bit_cast(bf16x8, mk4(OB + 4 * (2 * ch + b))), __builtin_bit_cast(bf16x8, mk4(OA[u >> 1] + 4 * (2 * (u & 1) + b))), acc[ch], 0, 0, 0);
        }
      }
    }
    STAMP(2 + 4 * job);
    WAITV(0);
    {
      LAS char* pw = smem + wave * 8192 + lane * 16;
#pragma unroll
      for (int ch = 0; ch < 2; ++ch)
#pragma unroll
        for (int g = 0; g < 4; ++g) {
          const f32x4 v = {acc[ch][4 * g], acc[ch][4 * g + 1], acc[ch][4 * g + 2], acc[ch][4 * g + 3]};
          *(LAS f32x4*)(pw + (ch * 4 + g) * 1024) = v;
        }
    }
    WAITL(); RAWBAR();
    {
      typedef float f32x2 __attribute__((ext_vector_type(2)));
      const int g = wave & 3, hf = wave >> 2;
      const LAS char* pr = smem + g * 1024 + lane * 16 + hf * 8;
      f32x2 r0[8], r1[8];
#pragma unroll
      for (int s = 0; s < 8; ++s) { r0[s] = *(const LAS f32x2*)(pr + s * 8192); r1[s] = *(const LAS f32x2*)(pr + s * 8192 + 4096); }
      __builtin_amdgcn_sched_barrier(0);
      const f32x2 a0 = ((r0[0] + r0[1]) + (r0[2] + r0[3])) + ((r0[4] + r0[5]) + (r0[6] + r0[7]));
      const f32x2 a1 = ((r1[0] + r1[1]) + (r1[2] + r1[3])) + ((r1[4] + r1[5]) + (r1[6] + r1[7]));
      u32x2 y;
      y[0] = pack_bf16(silu(a0[0]), silu(a1[0])); y[1] = pack_bf16(silu(a0[1]), silu(a1[1]));
      *(LAS u32x2*)(smem + L_IMG + (lane & 31) * 144 + (2 * g + (lane >> 5)) * 16 + hf * 8) = y;
    }
#if !(TMOX & 1)
    const u32 tv = ld_agt(tmo);
#endif
    WAITL(); RAWBAR();
    STAMP(3 + 4 * job);
    if (wave == 0) {
      u32 O[16];
      const bool badt = rfl(tv) == call;
#pragma unroll
      for (int cc = 0; cc < 4; ++cc) {
        const u32x4 t = *(const LAS u32x4*)(smem + L_IMG + (cc * 8 + (lane >> 3)) * 144 + (lane & 7) * 16);
#pragma unroll
        for (int e = 0; e < 4; ++e) O[4 * cc + e] = badt ? 0x7FC07FC0u : t[e];
      }
      u32 P[12];
      const u32 hdr = pack_unit(O, P, lane);
      const bool raw = (hdr & 127u) == 127u;
      if ((NDM >> o) & 1) {
        const u32 islot = OFF_IN + (rank * 256 + ht) * UNIT;
        char* pw = a.ws[0];
#pragma unroll
        for (int d = 1; d < 8; ++d) pw = lane == d ? a.ws[d] : pw;
        char* fo = pw + OFF_FO + (rank * 256 + ht) * 8;
        if (!raw && lane < 8 && lane != rank) st_sys64(fo, ((u64)call << 32) | hdr);
#pragma unroll
        for (int d = 0; d < 7; ++d) {
          int dd7 = d + (x + 2 * m) % 7; dd7 = dd7 >= 7 ? dd7 - 7 : dd7;
          char* dst = a.ws[(rank + 1 + dd7) & 7] + islot;
          if (raw) {
#pragma unroll
            for (int pl = 0; pl < 4; ++pl) st16(dst, voff + pl * 1024, mk4(O + 4 * pl));
          } else {
#pragma unroll
            for (int pl = 0; pl < 3; ++pl) st16(dst, voff + pl * 1024, mk4(P + 4 * pl));
          }
        }
        if (raw) { WAITV(0); if (lane < 8 && lane != rank) st_sys64(fo, ((u64)call << 32) | hdr); }
      } else {
        char* d = myws + OFF_OB + (IBO ? (call & 1) * OB_BYTES : 0) + ht * UNIT;
        if (raw) {
#pragma unroll
          for (int pl = 0; pl < 4; ++pl) st16(d, voff + pl * 1024, mk4(O + 4 * pl));
        } else {
#pragma unroll
          for (int pl = 0; pl < 3; ++pl) st16(d, voff + pl * 1024, mk4(P + 4 * pl));
        }
      }
      if (IBO && !raw && !((NDM >> o) & 1) && lane == 0) st_sys64(myws + OFF_FL + (ht >> 1) * 128 + rh * 8, ((u64)call << 32) | hdr);
      char* dd = outp + ((size_t)((rank >> 2) * 1024 + tm * 64 + rh * 32) * N_ + (rank & 3) * 512 + tn * 64) * 2;
#pragma unroll
      for (int cc = 0; cc < 4; ++cc) st16(dd + cc * 8 * N_ * 2, ovo, mk4(O + 4 * cc));
      if (BARC && job == 0) RAWBAR();
      if (!((NDM >> o) & 1) && (!IBO || raw)) {
        WAITV(4);
        if (lane == 0) st_sys64(myws + OFF_FL + (ht >> 1) * 128 + rh * 8, ((u64)call << 32) | hdr);
      }
      STAMP(4 + 4 * job);
    } else if (BARC && job == 0) RAWBAR();
  }
  if (wave != rank) {
#pragma unroll 1
    for (int job = 0; job < 2; ++job) {
      const int o = grp + 4 * job;
      const int tm = 4 * tile_m(o) + (ul >> 2), tn = 4 * tile_n(o) + (ul & 3);
      const int ht = (o * 16 + ul) * 2 + rh;
      bool ok2;
      u32 h0;
      if (SPOLL & 8) ok2 = swait1(myws + OFF_FO + (wave * 256 + ht) * 8, call, tmo, h0);
      else h0 = rl(wave_wait64(lane == 0 ? myws + OFF_FO + (wave * 256 + ht) * 8 : nullptr, call, tmo, ok2), 0);
      STAMP(9 + job);
      u32 O[16];
      const u32 si = OFF_IN + (wave * 256 + ht) * UNIT;
      bool bad2 = !ok2;
      {
        Unit u0;
        load_unit(u0, rs, voff, si);
        decode_unit(u0, rs, myws, voff, si, h0, lane, call, tmo, O, nrO, bad2);
      }
#if !(TMOX & 2)
      bad2 |= rfl(ld_agt(tmo)) == call;
#endif
      if (__builtin_expect(bad2, 0)) {
#pragma unroll
        for (int i = 0; i < 16; ++i) O[i] = 0x7FC07FC0u;
      }
      char* d = outp + ((size_t)((wave >> 2) * 1024 + tm * 64 + rh * 32) * N_ + (wave & 3) * 512 + tn * 64) * 2;
#pragma unroll
      for (int cc = 0; cc < 4; ++cc) st16(d + cc * 8 * N_ * 2, ovo, mk4(O + 4 * cc));
      const u32x4 z = {0, 0, 0, 0};
#pragma unroll
      for (int pl = 0; pl < 3; ++pl) st16(myws + si + pl * 1024, voff, z);
    }
  }
  if (tid == 0) st_agt(myws + OFF_CNT + wg * 128, call);
  STAMP(11);
}

extern "C" void kernel_launch(int rank, int nranks, ncclComm_t comm, hipStream_t stream, void* const* d_in, const int* in_sizes, int n_in,
                              void* d_out, int out_size, const int* in_dtypes, int out_dtype, void* const* d_out_all, void* const* d_ws_all, size_t ws_bytes) {
  if (nranks != NR || ws_bytes < WS_NEED || (unsigned)rank >= (unsigned)NR) { fprintf(stderr, "bad config\n"); return; }
  Args a;
  a.pmap = 0;
  for (int i = 0; i < NR; ++i) { const int p = link_dest(rank, i); a.ws[i] = (char*)d_ws_all[p]; a.pmap |= (u32)p << (4 * i); }
  a.out = (char*)d_out; a.x = (const char*)d_in[0]; a.w = (const char*)d_in[1]; a.myws = (char*)d_ws_all[rank]; a.rank = rank;
  fused_gemm_ar<<<NWG, NTHR, 0, stream>>>(a);
}
```
